```python
import numpy as np

kernel_name = "gpu_a2a_gemm_m8192_k8192_n8192_f32_none"


ABSMAX_RTOL = 0.001

COMPUTE_DTYPE = "float32"
EPILOGUE = "none"
IN_QUANT = None
OUT_QUANT = None
IN_DTYPES = {"x": "float32", "w_mat": "float32"}
WORLD_SIZE = 8

def setup_inputs():

    x = np.random.randn(8192, 8192).astype(np.float32)
    w_mat = (np.random.randn(8192, 8192) / np.sqrt(8192)).astype(np.float32)
    return {"x": x, "w_mat": w_mat}

def _epilogue(y):
    return y

def reference(x, w_mat):
    y = (x @ w_mat).astype(np.float32)
    return _epilogue(y)
```

```cpp
#include <hip/hip_runtime.h>
#include <rccl/rccl.h>
#include <cstdio>

typedef _Float16 h8 __attribute__((ext_vector_type(8)));
typedef float f4 __attribute__((ext_vector_type(4)));
typedef unsigned u4 __attribute__((ext_vector_type(4)));
typedef unsigned u2 __attribute__((ext_vector_type(2)));
typedef unsigned long long u64;
#define LAS __attribute__((address_space(3)))

#define PERIOD 6
#define WIN 2
#define E0 0
#define SW 4
#define NTSRC 0
#define LEAD 1
#define W0AUX ""
#define NTABS 0
#define NTX 0
#define C0X 0
#define NZ 0
#define NTLIM 128
#define UEARLY 0
#define CODEPF 0
#define EVEN 1
#define DBG_WHI 0
#define EV0 -1
#define TEARLY 0
#define WARM0 0
#define WARM1 0
#define WARM2 0
#define JROT 2
#define X0AUX ""
#define STAMPS 0
#define DBG_NOWAIT 0
#define DBG_NOPUSH 0
#define DBG_SELF 0
#define DBG_WFIX DBG_WHI
#define DBG_ABUF1 0
#define DBG_DUMMY 0

constexpr int NR = 8;
constexpr unsigned NN = 8192, ROWB = NN * 4;
constexpr unsigned X_BYTES = 8192u * 1024u * 4u, W_BYTES = 8192u * 8192u * 4u, C_BYTES = 1024u * 8192u * 4u;
constexpr unsigned WS_CNT = 0x000000, WS_TMO = 0x020000, WS_XFL = 0x040000, WS_FLG = 0x060000, WS_DBG = 0x2100000, WS_STG = 0x100000;
constexpr unsigned STG_BYTES = 16u << 20;
constexpr size_t WS_NEED = (size_t)WS_STG + 2 * (size_t)STG_BYTES;
#define DEADLINE "100000000"
#define RMASK 0x1ffffu

struct Params { const float* x; const float* W; float* C; char* ws; char* ws_all[NR]; int rank; };

#define FENCE() asm volatile("" ::: "memory")
#define BAR() do { FENCE(); __builtin_amdgcn_s_barrier(); FENCE(); } while (0)
#define RFL(x) __builtin_amdgcn_readfirstlane(x)

constexpr unsigned CE = 64;
__device__ __forceinline__ unsigned stg_base(unsigned h, unsigned mt) { return (h ? 260u + 126u * mt : 130u * mt) * 32768u; }

__global__ __launch_bounds__(512, 2) void a2a_gemm(Params p) {
  __shared__ __attribute__((aligned(1024))) char lds[131072 + 8192];
  const unsigned w = RFL(threadIdx.x >> 6);
  const unsigned lane = threadIdx.x & 63;
  const unsigned bid = blockIdx.x;
  const unsigned xcd = bid & 7, idx = bid >> 3;
  const unsigned h = xcd & 1, mt = idx & 1, nt = (xcd >> 1) * 16 + ((idx >> 1) ^ NTX);
  const unsigned rank = p.rank;

  const __amdgpu_buffer_rsrc_t srdW = __builtin_amdgcn_make_buffer_rsrc((void*)p.W, (short)0, W_BYTES, 0x00020000);
  const __amdgpu_buffer_rsrc_t srdX = __builtin_amdgcn_make_buffer_rsrc((void*)p.x, (short)0, X_BYTES, 0x00020000);
  const __amdgpu_buffer_rsrc_t srdF = __builtin_amdgcn_make_buffer_rsrc((void*)p.ws, (short)0, WS_STG, 0x00020000);
  const unsigned va = w * 4096 + lane * 16;
  const unsigned vw = (lane >> 4) * 4 * ROWB + (nt * 128 + (w & 1) * 64 + (lane & 15) * 4) * 4;
  const unsigned wo0 = (mt & LEAD) * 16 * ROWB, wo1 = wo0 + 2 * ROWB, wo2 = wo0 ^ (16 * ROWB), wo3 = wo2 + 2 * ROWB;
  unsigned vx = (lane >> 3) * 4096 + (lane & 7) * 16;
  unsigned vtw = 131072 + w * 1024 + ((lane & 3) * 16 + (lane >> 3)) * 16 + (((lane >> 2) ^ ((bid >> 3) & LEAD)) & 1) * 8, vtr = 131072 + w * 1024 + lane * 16;
  const unsigned ub = bid & 15, uh = w >> 2, um = ub >> 3, c0 = (bid >> 4) ^ C0X;
  const unsigned xw = (ub * 64 + (w & 3) * 16) * 4096 + uh * 128;
  const unsigned NSL = 32 - h;

  f4 XR[2];
  unsigned P[4][4];
  unsigned rdo = lane * 16, wro = va;
  unsigned vp, tab, vst = 0;

#define TS(k)
#define WAITV(n) asm volatile("s_waitcnt vmcnt(" #n ")" ::: "memory")
#define WAITL(n) asm volatile("s_waitcnt lgkmcnt(" #n ")" ::: "memory")
#define W_LD0(dst, so) asm volatile("buffer_load_dwordx4 %0, %1, %2, %3 offen " W0AUX : "=v"(dst) : "v"(vw), "s"(srdW), "s"(so) : "memory")
#define X_LD0(dst, so, aux) asm volatile("buffer_load_dwordx4 %0, %1, %2, %3 offen " aux : "=v"(dst) : "v"(vx), "s"(srdX), "s"(so) : "memory")
#define CVTP(d, a, b) asm volatile("v_cvt_pk_f16_f32 %0, %1, %2" : "=v"(d) : "v"(a), "v"(b))
#define CVT(pr, r0, r1) do { CVTP(P[0][pr], r0[0], r1[0]); CVTP(P[1][pr], r0[1], r1[1]); CVTP(P[2][pr], r0[2], r1[2]); CVTP(P[3][pr], r0[3], r1[3]); } while (0)
#define PARKX(r0, r1) do { unsigned a_, b_, c_, d_; CVTP(a_, r0[0], r0[1]); CVTP(b_, r0[2], r0[3]); CVTP(c_, r1[0], r1[1]); CVTP(d_, r1[2], r1[3]); \
    asm volatile("ds_write2_b64 %0, %1, %2 offset1:16" :: "v"(vtw), "v"((u2){a_, b_}), "v"((u2){c_, d_}) : "memory"); } while (0)
#define B_WR1(i) asm volatile("ds_write_b128 %0, %1 offset:%2" :: "v"(wro), "v"((u4){P[i][0], P[i][1], P[i][2], P[i][3]}), "n"((i) * 1024) : "memory")
#define SPIN_TAIL \
      "s_memrealtime s[92:93]\n\t" \
      "s_waitcnt lgkmcnt(0)\n\t" \
      "s_sub_u32 s92, s92, s90\n\t" \
      "s_cmp_lt_u32 s92, 20000\n\t" \
      "s_cbranch_scc1 4f\n\t" \
      "s_mov_b32 s89, 0x20000\n\t" \
      "buffer_load_dword %[vp], off, %[srdF], s89 sc1\n\t" \
      "s_waitcnt vmcnt(0)\n\t" \
      "v_readfirstlane_b32 s88, %[vp]\n\t" \
      "s_cmp_gt_u32 s92, " DEADLINE "\n\t" \
      "s_cbranch_scc1 3f\n\t" \
      "s_cmp_eq_u32 s88, %[call]\n\t" \
      "s_cbranch_scc1 3f\n" \
      "4:\n\t" \
      "s_sleep 1\n\t" \
      "s_branch 2b\n" \
      "3:\n\t" \
      "v_mov_b32 %[vp], %[call]\n\t" \
      "buffer_store_dword %[vp], off, %[srdF], s89 sc1\n\t" \
      "s_waitcnt vmcnt(0)\n\t"
  static_assert(WS_TMO == 0x20000, "SPIN_TAIL");
#define POLL_FOLD \
      "v_cmp_le_u32 vcc, %[call], %[vp]\n\t" \
      "s_nop 0\n\t" \
      "s_ff0_i32_b64 s88, vcc\n\t" \
      "s_min_u32 s88, s88, 64\n\t" \
      "s_lshr_b32 s88, s88, 3\n\t" \
      "s_add_u32 %[rdy], %[rdy], s88\n\t"
#define POLL_ISSUE \
      "s_lshl_b32 s88, %[rdy], 7\n\t" \
      "s_add_u32 s88, s88, %[fb]\n\t" \
      "buffer_load_dword %[vp], %[va], %[srdF], s88 offen sc0 sc1\n\t"
#define POLL_SPIN \
      "s_memrealtime s[90:91]\n" \
      "2:\n\t" \
      POLL_ISSUE \
      "s_waitcnt vmcnt(0) lgkmcnt(0)\n\t" \
      POLL_FOLD \
      "s_cmp_ge_u32 %[rdy], s94\n\t" \
      "s_cbranch_scc1 8f\n\t" \
      SPIN_TAIL \
      "s_movk_i32 %[rdy], 0x80\n" \
      "8:\n\t"

  unsigned call, par, parofs;
  unsigned evE = EV0 + ((bid * (PERIOD / 2)) >> 8); const unsigned fe0 = evE + 1;
  unsigned q0 = 1;
  u4 wa = {lane * 0x9E3779B1u & 0x3bff3bffu, (lane + 77) * 0x85EBCA6Bu & 0xbbff3bffu, (lane + 5) * 0xC2B2AE35u & 0x3bffbbffu, (lane + 31) * 0x27D4EB2Fu & 0xbbffbbffu};
  u4 wc = {wa[3], wa[0], wa[2], wa[1]};
  f4 z0 = {0.f, 0.f, 0.f, 0.f}, z1 = z0, z2 = z0, z3 = z0;
#define WARM(n) do { if (n) { unsigned k_ = n; asm volatile("s_nop 1\n1:\n\t" \
      "v_mfma_f32_16x16x32_f16 %0, %5, %6, %0\n\tv_mfma_f32_16x16x32_f16 %1, %6, %5, %1\n\tv_mfma_f32_16x16x32_f16 %2, %5, %5, %2\n\tv_mfma_f32_16x16x32_f16 %3, %6, %6, %3\n\t" \
      "v_mfma_f32_16x16x32_f16 %0, %6, %5, %0\n\tv_mfma_f32_16x16x32_f16 %1, %5, %6, %1\n\tv_mfma_f32_16x16x32_f16 %2, %6, %6, %2\n\tv_mfma_f32_16x16x32_f16 %3, %5, %5, %3\n\t" \
      "s_sub_u32 %4, %4, 1\n\ts_cmp_lg_u32 %4, 0\n\ts_cbranch_scc1 1b\n\ts_nop 7" \
      : "+v"(z0), "+v"(z1), "+v"(z2), "+v"(z3), "+s"(k_) : "v"(wa), "v"(wc) : "scc"); } } while (0)
  {
    f4 T[8], U[8], X0[2];
    unsigned cv;
    TS(0);
    const unsigned cofs = WS_CNT + (bid * 8 + w) * 64;
    asm volatile("buffer_load_dword %0, off, %1, %2 sc1" : "=v"(cv) : "s"(srdF), "s"(cofs) : "memory");
    const unsigned x0 = rank * (4u << 20) + c0 * 256 + xw;
    X_LD0(X0[0], x0, X0AUX); X_LD0(X0[1], x0 + 32768, X0AUX);
    const unsigned j1 = c0 / 7, ip1 = c0 - j1 * 7;
    unsigned x1 = ((rank - 1 - ip1) & 7) * (4u << 20) + ((j1 + JROT * ip1) & 15) * 256 + xw;
    asm volatile("" : "+s"(x1));
    X_LD0(XR[0], x1, "nt"); X_LD0(XR[1], x1 + 32768, "nt");
    const unsigned ks = w >> 1;
    unsigned w0 = w >= 4 ? (rank * 1024 + (ks - 2) * 64 + h * 32) * ROWB : 0x10000000u;
    unsigned w1 = (rank * 1024 + (ks + 2) * 64 + h * 32) * ROWB;
    unsigned off0 = WS_STG + stg_base(uh, um) + c0 * 32768u + ((w & 3) ^ (2 * uh)) * 1024u + (ub & 7) * 4096 - w * 4096;
    asm volatile("" : "+s"(w0), "+s"(w1), "+s"(off0));
#define WT_LD(T, base) do { W_LD0(T[0], (base) + wo0); W_LD0(T[1], (base) + wo0 + ROWB); W_LD0(T[2], (base) + wo1); W_LD0(T[3], (base) + wo1 + ROWB); \
                            W_LD0(T[4], (base) + wo2); W_LD0(T[5], (base) + wo2 + ROWB); W_LD0(T[6], (base) + wo3); W_LD0(T[7], (base) + wo3 + ROWB); } while (0)
#define WT_CVT(T) do { CVT(0, T[0], T[1]); CVT(1, T[2], T[3]); CVT(2, T[4], T[5]); CVT(3, T[6], T[7]); } while (0)
    if (TEARLY) WT_LD(T, w0);
    WARM(WARM0);
    asm volatile("s_waitcnt vmcnt(%3)" : "+v"(cv), "+v"(X0[0]), "+v"(X0[1]) : "n"(TEARLY ? 10 : 2) : "memory");
    TS(1);
    PARKX(X0[0], X0[1]);
    call = RFL(cv) + 1; par = call & 1; parofs = par * STG_BYTES;
    const char* dst0 = p.ws + (parofs + off0);
    asm volatile("ds_read_b128 %0, %2\n\ts_waitcnt lgkmcnt(0)\n\tglobal_store_dwordx4 %1, %0, %3 sc0 sc1\n\ts_nop 1"
                 : "=&v"(X0[0]) : "v"(va), "v"(vtr), "s"(dst0) : "memory");
    {
      unsigned ln = lane; asm volatile("" : "+v"(ln));
      const unsigned f = ln >> 3, sp = ln == 24 ? 2u : ln - 29, isw = (f >= 4) | (ln == 24);
      const unsigned ps = sp * 4 + ks - 2, ex = !h && ps - CE < 4;
      const unsigned c = (isw ? (ps < CE ? ps : ex ? CE + ((ps - CE) >> 1) : ps - 2 + 4 * h) : 16 * (ln & 7) + c0) & 127;
      const unsigned i = (c - 16) & 127, j = i / 7, ip = i - j * 7, col = (j + JROT * ip) & 15;
      const unsigned d = (rank - 1 - ip) & 7, dp = (DBG_SELF == 1 || ((DBG_SELF >> 8 >> rank) & 1)) ? rank : d;
      unsigned lo = 0, hi = 0;
#pragma unroll
      for (unsigned i2 = 0; i2 < NR; ++i2) { const u64 a_ = (u64)p.ws_all[i2]; lo = dp == i2 ? (unsigned)a_ : lo; hi = dp == i2 ? (unsigned)(a_ >> 32) : hi; }
      const unsigned dbl = c - CE < 2, hc = dbl ? 0u : uh, ml = (w & 3) ^ (2 * hc), u = c - 112;
      const unsigned cc = c < 112 ? c : 112 + (ml < 2 ? 8u : 0u) + (u >> 2) * 2 + (u & 1);
      const unsigned sl = c < 112 ? ml : ((u >> 1) & 1) * 2 + (w & 1);
      const unsigned pos = c < CE ? c : dbl ? 2 * c - CE + uh : cc + 2 - 4 * uh;
      const unsigned so = WS_STG + stg_base(hc, um) + pos * 32768u + sl * 1024u + (ub & 7) * 4096 - w * 4096;
      const unsigned xo = d * (4u << 20) + col * 256 + xw;
      const unsigned half = ex ? ps & 1 : h;
      const unsigned wr = wo0 + ((c < 16 ? rank * 1024 + c * 64 : ((rank + 1 + ip) & 7) * 1024 + col * 64) + half * 32) * ROWB;
      const unsigned wr2 = DBG_WHI ? wr | (4096u << 15) : wr | (c >= 16 ? (NTSRC >> (ip + 1)) & 1 : NTSRC & 1);
      tab = isw ? wr2 : f == 0 ? lo : f == 1 ? hi : f == 2 ? xo : so;
      asm volatile("" : "+v"(tab));
    }
    WARM(WARM2);
    asm volatile("s_waitcnt vmcnt(0)" : "+v"(XR[0]), "+v"(XR[1]) :: "memory");
    TS(3);
    PARKX(XR[0], XR[1]);
    BAR();
    vp = call;
    asm volatile("s_mov_b64 exec, 1\n\ts_cmp_lg_u32 %4, %5\n\ts_cbranch_scc1 1f\n\tbuffer_store_dword %0, off, %1, %2 sc0 sc1\n1:\n\tbuffer_store_dword %0, off, %1, %3 sc1\n\ts_mov_b64 exec, -1"
                 :: "v"(vp), "s"(srdF), "s"(WS_FLG + (par * 2 + um) * 32768 + c0 * 128 + (ub & 7) * 16), "s"(cofs), "s"(w), "n"(SW) : "memory", "scc");
    asm volatile(
      "s_cmp_lg_u32 %[ev], -1\n\t"
      "s_cbranch_scc1 1f\n\t"
      "ds_read_b128 %[t], %[tr]\n\t"
      "v_readlane_b32 s90, %[tab], 1\n\t"
      "v_readlane_b32 s91, %[tab], 9\n\t"
      "v_readlane_b32 s89, %[tab], 25\n\t"
      "v_readlane_b32 s92, %[tab], 18\n\t"
      "s_mov_b32 %[q], 2\n\t"
      "s_add_u32 %[ev], %[ev], %[per]\n\t"
      "s_add_u32 s89, s89, %[pb]\n\t"
      "s_add_u32 s90, s90, s89\n\t"
      "s_addc_u32 s91, s91, 0\n\t"
      "s_add_u32 s93, s92, 0x8000\n\t"
      "s_waitcnt lgkmcnt(0)\n\t"
      "global_store_dwordx4 %[va], %[t], s[90:91] sc0 sc1\n\t"
      "buffer_load_dwordx4 %[x0], %[vx], %[srdX], s92 offen nt\n\t"
      "buffer_load_dwordx4 %[x1], %[vx], %[srdX], s93 offen nt\n"
      "1:"
      : [ev] "+&s"(evE), [q] "+&s"(q0), [t] "=&v"(X0[0]), [x0] "+&v"(XR[0]), [x1] "+&v"(XR[1])
      : [tr] "v"(vtr), [tab] "v"(tab), [per] "n"(PERIOD / 2), [pb] "s"(parofs), [va] "v"(va), [vx] "v"(vx), [srdX] "s"(srdX)
      : "memory", "scc", "s89", "s90", "s91", "s92", "s93");
    asm volatile("" : "+s"(evE), "+s"(q0));
    if (!TEARLY) WT_LD(T, w0);
    WT_LD(U, w1);
    WARM(WARM1);
    if (!TEARLY) WAITV(8);
    WT_CVT(T);
    rdo += (unsigned)(size_t)(LAS char*)lds; wro += (unsigned)(size_t)(LAS char*)lds;
    B_WR1(0); B_WR1(1); B_WR1(2); B_WR1(3);
    wro += 32768;
    asm volatile("s_waitcnt vmcnt(0)" : "+v"(XR[0]), "+v"(XR[1]) :: "memory");
    WT_CVT(U);
    B_WR1(0); B_WR1(1); B_WR1(2); B_WR1(3);
    wro += 32768;
  }
  const __amdgpu_buffer_rsrc_t srdA = __builtin_amdgcn_make_buffer_rsrc((void*)(p.ws + WS_STG + parofs + stg_base(h, mt) - 2 * 32768), (short)0, (NSL + 1) * 131072u, 0x00020000);
  const unsigned fb = WS_FLG + (par * 2 + mt) * 32768 - SW * 4096;
  unsigned rdy = 0, fe = fe0;
  const unsigned fl0 = WS_FLG + (par * 2 + um) * 32768 + c0 * 128 + (ub & 7) * 16 - SW * 4096;
  const unsigned lastS = NSL - 4;
  TS(4);
  asm volatile(
      "s_cmp_lg_u32 %[w], %[sw]\n\t"
      "s_cbranch_scc1 1f\n\t"
      "s_mov_b32 s94, %[n0]\n\t"
      POLL_SPIN
      "1:\n\t"
      "s_waitcnt lgkmcnt(0)\n\t"
      "s_barrier"
      : [rdy] "+&s"(rdy), [vp] "+&v"(vp)
      : [w] "s"(w), [sw] "n"(SW), [call] "s"(call), [va] "v"(va), [fb] "s"(fb), [srdF] "s"(srdF), [n0] "n"(6)
      : "memory", "scc", "vcc", "s88", "s89", "s90", "s91", "s92", "s93", "s94");
  TS(7);
  asm volatile("" : "+s"(rdy));
  if (DBG_NOWAIT) rdy = 0x80;
  const __amdgpu_buffer_rsrc_t srdC = __builtin_amdgcn_make_buffer_rsrc((void*)p.C, (short)0, C_BYTES, 0x00020000);
  const unsigned crow = (mt * 512 + w * 64) * ROWB + nt * 512;
  const unsigned c_me = crow + h * 32 * ROWB, c_pt = crow + (h ^ 1) * 32 * ROWB;
  const unsigned xf_me = WS_XFL + (((nt * 2 + mt) * 2 + h) * 8 + w) * 64;
  unsigned slow = 0;

  asm volatile(
      "v_add_u32 v250, 32768, v249\n"
      "s_nop 0\n"
      "v_readlane_b32 s44, v254, 24\n"
      "s_mov_b32 s41, 0\n"
      "s_mov_b32 s42, 65536\n"
      "s_mov_b32 s43, 98304\n"
      "s_mov_b32 s48, 1\n"
      "s_mov_b32 s49, 1\n"
      "s_xor_b32 s51, s40, 512\n"
      "s_lshl_b32 s60, s28, 2\n"
      "s_lshr_b32 s60, 0x10543210, s60\n"
      "s_and_b32 s60, s60, 7\n"
      "s_add_u32 s56, s60, 22\n"
      "buffer_load_dwordx4 v[160:163], v248, s[20:23], s42 offen offset:0 sc0 sc1\n"
      "buffer_load_dwordx4 v[176:179], v248, s[20:23], s43 offen offset:0 sc0 sc1\n"
      "buffer_load_dwordx4 v[164:167], v248, s[20:23], s42 offen offset:1024 sc0 sc1\n"
      "buffer_load_dwordx4 v[180:183], v248, s[20:23], s43 offen offset:1024 sc0 sc1\n"
      "buffer_load_dwordx4 v[168:171], v248, s[20:23], s42 offen offset:2048 sc0 sc1\n"
      "buffer_load_dwordx4 v[184:187], v248, s[20:23], s43 offen offset:2048 sc0 sc1\n"
      "buffer_load_dwordx4 v[172:175], v248, s[20:23], s42 offen offset:3072 sc0 sc1\n"
      "buffer_load_dwordx4 v[188:191], v248, s[20:23], s43 offen offset:3072 sc0 sc1\n"
      "ds_read_b128 v[192:195], v251 offset:16384\n"
      "ds_read_b128 v[196:199], v251 offset:24576\n"
      "s_waitcnt lgkmcnt(0)\n"
      "ds_read_b128 v[200:203], v251 offset:17408\n"
      "ds_read_b128 v[204:207], v251 offset:25600\n"
      "s_waitcnt vmcnt(6)\n"
      "v_mfma_f32_16x16x32_f16 v[0:3], v[160:163], v[192:195], 0\n"
      "s_setprio 1\n"
      "v_mfma_f32_16x16x32_f16 v[0:3], v[176:179], v[196:199], v[0:3]\n"
      "s_waitcnt vmcnt(4)\n"
      "v_mfma_f32_16x16x32_f16 v[32:35], v[164:167], v[192:195], 0\n"
      "v_mfma_f32_16x16x32_f16 v[32:35], v[180:183], v[196:199], v[32:35]\n"
      "s_waitcnt vmcnt(2)\n"
      "v_mfma_f32_16x16x32_f16 v[64:67], v[168:171], v[192:195], 0\n"
      "v_mfma_f32_16x16x32_f16 v[64:67], v[184:187], v[196:199], v[64:67]\n"
      "s_waitcnt vmcnt(0)\n"
      "v_mfma_f32_16x16x32_f16 v[96:99], v[172:175], v[192:195], 0\n"
      "v_mfma_f32_16x16x32_f16 v[96:99], v[188:191], v[196:199], v[96:99]\n"
      "s_cmp_eq_u32 s31, 0\n"
      "s_cbranch_scc1 .Le01_%=\n"
      ".Le0b2_%=:\n"
      "s_cmp_eq_u32 s28, 4\n"
      "s_cbranch_scc1 .Lso3_%=\n"
      ".Lsob4_%=:\n"
      "s_and_b32 s45, s44, -2\n"
      "s_add_u32 s46, s45, 65536\n"
      "buffer_load_dwordx4 v[208:211], v249, s[8:11], s45 offen\n"
      "buffer_load_dwordx4 v[212:215], v250, s[8:11], s45 offen\n"
      "buffer_load_dwordx4 v[216:219], v249, s[8:11], s46 offen\n"
      "buffer_load_dwordx4 v[220:223], v250, s[8:11], s46 offen\n"
      "s_waitcnt lgkmcnt(0)\n"
      "ds_read_b128 v[192:195], v251 offset:18432\n"
      "ds_read_b128 v[196:199], v251 offset:26624\n"
      "v_mfma_f32_16x16x32_f16 v[4:7], v[160:163], v[200:203], 0\n"
      "v_mfma_f32_16x16x32_f16 v[4:7], v[176:179], v[204:207], v[4:7]\n"
      "v_mfma_f32_16x16x32_f16 v[36:39], v[164:167], v[200:203], 0\n"
      "v_mfma_f32_16x16x32_f16 v[36:39], v[180:183], v[204:207], v[36:39]\n"
      "v_mfma_f32_16x16x32_f16 v[68:71], v[168:171], v[200:203], 0\n"
      "v_mfma_f32_16x16x32_f16 v[68:71], v[184:187], v[204:207], v[68:71]\n"
      "v_mfma_f32_16x16x32_f16 v[100:103], v[172:175], v[200:203], 0\n"
      "v_mfma_f32_16x16x32_f16 v[100:103], v[188:191], v[204:207], v[100:103]\n"
      "s_setprio 0\n"
      "s_add_u32 s42, s42, 65536\n"
      "s_add_u32 s43, s43, 65536\n"
      "buffer_load_dwordx4 v[128:131], v248, s[20:23], s42 offen offset:0 sc0 sc1\n"
      "buffer_load_dwordx4 v[144:147], v248, s[20:23], s43 offen offset:0 sc0 sc1\n"
      "s_waitcnt lgkmcnt(0)\n"
      "ds_read_b128 v[200:203], v251 offset:19456\n"
      "ds_read_b128 v[204:207], v251 offset:27648\n"
      "v_mfma_f32_16x16x32_f16 v[8:11], v[160:163], v[192:195], 0\n"
      "s_setprio 1\n"
      "v_mfma_f32_16x16x32_f16 v[8:11], v[176:179], v[196:199], v[8:11]\n"
      "v_mfma_f32_16x16x32_f16 v[40:43], v[164:167], v[192:195], 0\n"
      "v_mfma_f32_16x16x32_f16 v[40:43], v[180:183], v[196:199], v[40:43]\n"
      "v_mfma_f32_16x16x32_f16 v[72:75], v[168:171], v[192:195], 0\n"
      "v_mfma_f32_16x16x32_f16 v[72:75], v[184:187], v[196:199], v[72:75]\n"
      "v_mfma_f32_16x16x32_f16 v[104:107], v[172:175], v[192:195], 0\n"
      "v_mfma_f32_16x16x32_f16 v[104:107], v[188:191], v[196:199], v[104:107]\n"
      "buffer_load_dwordx4 v[132:135], v248, s[20:23], s42 offen offset:1024 sc0 sc1\n"
      "buffer_load_dwordx4 v[148:151], v248, s[20:23], s43 offen offset:1024 sc0 sc1\n"
      "s_waitcnt lgkmcnt(0)\n"
      "ds_read_b128 v[192:195], v251 offset:20480\n"
      "ds_read_b128 v[196:199], v251 offset:28672\n"
      "v_mfma_f32_16x16x32_f16 v[12:15], v[160:163], v[200:203], 0\n"
      "v_mfma_f32_16x16x32_f16 v[12:15], v[176:179], v[204:207], v[12:15]\n"
      "v_mfma_f32_16x16x32_f16 v[44:47], v[164:167], v[200:203], 0\n"
      "v_mfma_f32_16x16x32_f16 v[44:47], v[180:183], v[204:207], v[44:47]\n"
      "v_mfma_f32_16x16x32_f16 v[76:79], v[168:171], v[200:203], 0\n"
      "v_mfma_f32_16x16x32_f16 v[76:79], v[184:187], v[204:207], v[76:79]\n"
      "v_mfma_f32_16x16x32_f16 v[108:111], v[172:175], v[200:203], 0\n"
      "v_mfma_f32_16x16x32_f16 v[108:111], v[188:191], v[204:207], v[108:111]\n"
      "s_setprio 0\n"
      "buffer_load_dwordx4 v[136:139], v248, s[20:23], s42 offen offset:2048 sc0 sc1\n"
      "buffer_load_dwordx4 v[152:155], v248, s[20:23], s43 offen offset:2048 sc0 sc1\n"
      "s_waitcnt lgkmcnt(0)\n"
      "ds_read_b128 v[200:203], v251 offset:21504\n"
      "ds_read_b128 v[204:207], v251 offset:29696\n"
      "v_mfma_f32_16x16x32_f16 v[16:19], v[160:163], v[192:195], 0\n"
      "s_setprio 1\n"
      "v_mfma_f32_16x16x32_f16 v[16:19], v[176:179], v[196:199], v[16:19]\n"
      "v_mfma_f32_16x16x32_f16 v[48:51], v[164:167], v[192:195], 0\n"
      "v_mfma_f32_16x16x32_f16 v[48:51], v[180:183], v[196:199], v[48:51]\n"
      "v_mfma_f32_16x16x32_f16 v[80:83], v[168:171], v[192:195], 0\n"
      "v_mfma_f32_16x16x32_f16 v[80:83], v[184:187], v[196:199], v[80:83]\n"
      "v_mfma_f32_16x16x32_f16 v[112:115], v[172:175], v[192:195], 0\n"
      "v_mfma_f32_16x16x32_f16 v[112:115], v[188:191], v[196:199], v[112:115]\n"
      "buffer_load_dwordx4 v[140:143], v248, s[20:23], s42 offen offset:3072 sc0 sc1\n"
      "buffer_load_dwordx4 v[156:159], v248, s[20:23], s43 offen offset:3072 sc0 sc1\n"
      "s_waitcnt lgkmcnt(0)\n"
      "ds_read_b128 v[192:195], v251 offset:22528\n"
      "ds_read_b128 v[196:199], v251 offset:30720\n"
      "v_mfma_f32_16x16x32_f16 v[20:23], v[160:163], v[200:203], 0\n"
      "v_mfma_f32_16x16x32_f16 v[20:23], v[176:179], v[204:207], v[20:23]\n"
      "v_mfma_f32_16x16x32_f16 v[52:55], v[164:167], v[200:203], 0\n"
      "v_mfma_f32_16x16x32_f16 v[52:55], v[180:183], v[204:207], v[52:55]\n"
      "v_mfma_f32_16x16x32_f16 v[84:87], v[168:171], v[200:203], 0\n"
      "v_mfma_f32_16x16x32_f16 v[84:87], v[184:187], v[204:207], v[84:87]\n"
      "v_mfma_f32_16x16x32_f16 v[116:119], v[172:175], v[200:203], 0\n"
      "v_mfma_f32_16x16x32_f16 v[116:119], v[188:191], v[204:207], v[116:119]\n"
      "s_setprio 0\n"
      "s_waitcnt vmcnt(8)\n"
      "v_cvt_pk_f16_f32 v240, v208, v212\n"
      "v_cvt_pk_f16_f32 v241, v216, v220\n"
      "ds_write_b64 v252, v[240:241] offset:0\n"
      "v_cvt_pk_f16_f32 v242, v209, v213\n"
      "v_cvt_pk_f16_f32 v243, v217, v221\n"
      "ds_write_b64 v252, v[242:243] offset:1024\n"
      "v_cvt_pk_f16_f32 v244, v210, v214\n"
      "v_cvt_pk_f16_f32 v245, v218, v222\n"
      "ds_write_b64 v252, v[244:245] offset:2048\n"
      "v_cvt_pk_f16_f32 v246, v211, v215\n"
      "v_cvt_pk_f16_f32 v247, v219, v223\n"
      "ds_write_b64 v252, v[246:247] offset:3072\n"
      "s_and_b32 s45, s44, -2\n"
      "s_xor_b32 s45, s45, 524288\n"
      "s_add_u32 s46, s45, 65536\n"
      "buffer_load_dwordx4 v[208:211], v249, s[8:11], s45 offen\n"
      "buffer_load_dwordx4 v[212:215], v250, s[8:11], s45 offen\n"
      "buffer_load_dwordx4 v[216:219], v249, s[8:11], s46 offen\n"
      "buffer_load_dwordx4 v[220:223], v250, s[8:11], s46 offen\n"
      "s_waitcnt lgkmcnt(4)\n"
      "ds_read_b128 v[200:203], v251 offset:23552\n"
      "ds_read_b128 v[204:207], v251 offset:31744\n"
      "v_mfma_f32_16x16x32_f16 v[24:27], v[160:163], v[192:195], 0\n"
      "s_setprio 1\n"
      "v_mfma_f32_16x16x32_f16 v[24:27], v[176:179], v[196:199], v[24:27]\n"
      "v_mfma_f32_16x16x32_f16 v[56:59], v[164:167], v[192:195], 0\n"
      "v_mfma_f32_16x16x32_f16 v[56:59], v[180:183], v[196:199], v[56:59]\n"
      "v_mfma_f32_16x16x32_f16 v[88:91], v[168:171], v[192:195], 0\n"
      "v_mfma_f32_16x16x32_f16 v[88:91], v[184:187], v[196:199], v[88:91]\n"
      "v_mfma_f32_16x16x32_f16 v[120:123], v[172:175], v[192:195], 0\n"
      "v_mfma_f32_16x16x32_f16 v[120:123], v[188:191], v[196:199], v[120:123]\n"
      "s_barrier\n"
      "s_waitcnt lgkmcnt(0)\n"
      "v_add_u32 v251, 0x8000, v251\n"
      "v_and_b32 v251, 0x1ffff, v251\n"
      "ds_read_b128 v[192:195], v251 offset:0\n"
      "ds_read_b128 v[196:199], v251 offset:8192\n"
      "v_mfma_f32_16x16x32_f16 v[28:31], v[160:163], v[200:203], 0\n"
      "v_mfma_f32_16x16x32_f16 v[28:31], v[176:179], v[204:207], v[28:31]\n"
      "v_mfma_f32_16x16x32_f16 v[60:63], v[164:167], v[200:203], 0\n"
      "v_mfma_f32_16x16x32_f16 v[60:63], v[180:183], v[204:207], v[60:63]\n"
      "v_mfma_f32_16x16x32_f16 v[92:95], v[168:171], v[200:203], 0\n"
      "v_mfma_f32_16x16x32_f16 v[92:95], v[184:187], v[204:207], v[92:95]\n"
      "v_mfma_f32_16x16x32_f16 v[124:127], v[172:175], v[200:203], 0\n"
      "v_mfma_f32_16x16x32_f16 v[124:127], v[188:191], v[204:207], v[124:127]\n"
      "s_setprio 0\n"
      "s_mov_b32 s41, 1\n"
      "s_mov_b32 s54, 32\n"
      ".Ltop_%=:\n"
      "v_readlane_b32 s44, v254, s54\n"
      "s_cmp_eq_u32 s31, s41\n"
      "s_cbranch_scc1 .Lev_%=\n"
      "s_waitcnt lgkmcnt(0)\n"
      "ds_read_b128 v[200:203], v251 offset:1024\n"
      "ds_read_b128 v[204:207], v251 offset:9216\n"
      "s_waitcnt vmcnt(10)\n"
      "v_mfma_f32_16x16x32_f16 v[0:3], v[128:131], v[192:195], v[0:3]\n"
      "s_setprio 1\n"
      "v_mfma_f32_16x16x32_f16 v[0:3], v[144:147], v[196:199], v[0:3]\n"
      "s_waitcnt vmcnt(8)\n"
      "v_mfma_f32_16x16x32_f16 v[32:35], v[132:135], v[192:195], v[32:35]\n"
      "v_mfma_f32_16x16x32_f16 v[32:35], v[148:151], v[196:199], v[32:35]\n"
      "s_waitcnt vmcnt(6)\n"
      "v_mfma_f32_16x16x32_f16 v[64:67], v[136:139], v[192:195], v[64:67]\n"
      "v_mfma_f32_16x16x32_f16 v[64:67], v[152:155], v[196:199], v[64:67]\n"
      "s_waitcnt vmcnt(4)\n"
      "v_mfma_f32_16x16x32_f16 v[96:99], v[140:143], v[192:195], v[96:99]\n"
      "v_mfma_f32_16x16x32_f16 v[96:99], v[156:159], v[196:199], v[96:99]\n"
      "s_add_u32 s42, s42, 65536\n"
      "s_add_u32 s43, s43, 65536\n"
      "buffer_load_dwordx4 v[160:163], v248, s[20:23], s42 offen offset:0 sc0 sc1\n"
      "buffer_load_dwordx4 v[176:179], v248, s[20:23], s43 offen offset:0 sc0 sc1\n"
      "buffer_load_dwordx4 v[164:167], v248, s[20:23], s42 offen offset:1024 sc0 sc1\n"
      "buffer_load_dwordx4 v[180:183], v248, s[20:23], s43 offen offset:1024 sc0 sc1\n"
      "s_waitcnt lgkmcnt(0)\n"
      "ds_read_b128 v[192:195], v251 offset:2048\n"
      "ds_read_b128 v[196:199], v251 offset:10240\n"
      "v_mfma_f32_16x16x32_f16 v[4:7], v[128:131], v[200:203], v[4:7]\n"
      "v_mfma_f32_16x16x32_f16 v[4:7], v[144:147], v[204:207], v[4:7]\n"
      "v_mfma_f32_16x16x32_f16 v[36:39], v[132:135], v[200:203], v[36:39]\n"
      "v_mfma_f32_16x16x32_f16 v[36:39], v[148:151], v[204:207], v[36:39]\n"
      "v_mfma_f32_16x16x32_f16 v[68:71], v[136:139], v[200:203], v[68:71]\n"
      "v_mfma_f32_16x16x32_f16 v[68:71], v[152:155], v[204:207], v[68:71]\n"
      "v_mfma_f32_16x16x32_f16 v[100:103], v[140:143], v[200:203], v[100:103]\n"
      "v_mfma_f32_16x16x32_f16 v[100:103], v[156:159], v[204:207], v[100:103]\n"
      "s_setprio 0\n"
      "buffer_load_dwordx4 v[168:171], v248, s[20:23], s42 offen offset:2048 sc0 sc1\n"
      "buffer_load_dwordx4 v[184:187], v248, s[20:23], s43 offen offset:2048 sc0 sc1\n"
      "s_waitcnt lgkmcnt(0)\n"
      "ds_read_b128 v[200:203], v251 offset:3072\n"
      "ds_read_b128 v[204:207], v251 offset:11264\n"
      "v_mfma_f32_16x16x32_f16 v[8:11], v[128:131], v[192:195], v[8:11]\n"
      "s_setprio 1\n"
      "v_mfma_f32_16x16x32_f16 v[8:11], v[144:147], v[196:199], v[8:11]\n"
      "v_mfma_f32_16x16x32_f16 v[40:43], v[132:135], v[192:195], v[40:43]\n"
      "v_mfma_f32_16x16x32_f16 v[40:43], v[148:151], v[196:199], v[40:43]\n"
      "v_mfma_f32_16x16x32_f16 v[72:75], v[136:139], v[192:195], v[72:75]\n"
      "v_mfma_f32_16x16x32_f16 v[72:75], v[152:155], v[196:199], v[72:75]\n"
      "v_mfma_f32_16x16x32_f16 v[104:107], v[140:143], v[192:195], v[104:107]\n"
      "v_mfma_f32_16x16x32_f16 v[104:107], v[156:159], v[196:199], v[104:107]\n"
      "buffer_load_dwordx4 v[172:175], v248, s[20:23], s42 offen offset:3072 sc0 sc1\n"
      "buffer_load_dwordx4 v[188:191], v248, s[20:23], s43 offen offset:3072 sc0 sc1\n"
      "s_waitcnt vmcnt(8)\n"
      "v_cvt_pk_f16_f32 v240, v208, v212\n"
      "v_cvt_pk_f16_f32 v241, v216, v220\n"
      "ds_write_b64 v252, v[240:241] offset:8\n"
      "v_cvt_pk_f16_f32 v242, v209, v213\n"
      "v_cvt_pk_f16_f32 v243, v217, v221\n"
      "ds_write_b64 v252, v[242:243] offset:1032\n"
      "v_cvt_pk_f16_f32 v244, v210, v214\n"
      "v_cvt_pk_f16_f32 v245, v218, v222\n"
      "ds_write_b64 v252, v[244:245] offset:2056\n"
      "v_cvt_pk_f16_f32 v246, v211, v215\n"
      "v_cvt_pk_f16_f32 v247, v219, v223\n"
      "ds_write_b64 v252, v[246:247] offset:3080\n"
      "v_add_u32 v252, 0x8000, v252\n"
      "v_and_b32 v252, 0x1ffff, v252\n"
      "s_waitcnt lgkmcnt(4)\n"
      "ds_read_b128 v[192:195], v251 offset:4096\n"
      "ds_read_b128 v[196:199], v251 offset:12288\n"
      "v_mfma_f32_16x16x32_f16 v[12:15], v[128:131], v[200:203], v[12:15]\n"
      "v_mfma_f32_16x16x32_f16 v[12:15], v[144:147], v[204:207], v[12:15]\n"
      "v_mfma_f32_16x16x32_f16 v[44:47], v[132:135], v[200:203], v[44:47]\n"
      "v_mfma_f32_16x16x32_f16 v[44:47], v[148:151], v[204:207], v[44:47]\n"
      "v_mfma_f32_16x16x32_f16 v[76:79], v[136:139], v[200:203], v[76:79]\n"
      "v_mfma_f32_16x16x32_f16 v[76:79], v[152:155], v[204:207], v[76:79]\n"
      "v_mfma_f32_16x16x32_f16 v[108:111], v[140:143], v[200:203], v[108:111]\n"
      "v_mfma_f32_16x16x32_f16 v[108:111], v[156:159], v[204:207], v[108:111]\n"
      "s_setprio 0\n"
      "s_cmp_eq_u32 s28, 4\n"
      "s_cbranch_scc1 .Lso6_%=\n"
      ".Lsob7_%=:\n"
      "s_and_b32 s45, s44, -2\n"
      "s_add_u32 s46, s45, 65536\n"
      "buffer_load_dwordx4 v[208:211], v249, s[8:11], s45 offen\n"
      "buffer_load_dwordx4 v[212:215], v250, s[8:11], s45 offen\n"
      "buffer_load_dwordx4 v[216:219], v249, s[8:11], s46 offen\n"
      "buffer_load_dwordx4 v[220:223], v250, s[8:11], s46 offen\n"
      "s_waitcnt lgkmcnt(0)\n"
      "ds_read_b128 v[200:203], v251 offset:5120\n"
      "ds_read_b128 v[204:207], v251 offset:13312\n"
      "v_mfma_f32_16x16x32_f16 v[16:19], v[128:131], v[192:195], v[16:19]\n"
      "s_setprio 1\n"
      "v_mfma_f32_16x16x32_f16 v[16:19], v[144:147], v[196:199], v[16:19]\n"
      "v_mfma_f32_16x16x32_f16 v[48:51], v[132:135], v[192:195], v[48:51]\n"
      "v_mfma_f32_16x16x32_f16 v[48:51], v[148:151], v[196:199], v[48:51]\n"
      "v_mfma_f32_16x16x32_f16 v[80:83], v[136:139], v[192:195], v[80:83]\n"
      "v_mfma_f32_16x16x32_f16 v[80:83], v[152:155], v[196:199], v[80:83]\n"
      "v_mfma_f32_16x16x32_f16 v[112:115], v[140:143], v[192:195], v[112:115]\n"
      "v_mfma_f32_16x16x32_f16 v[112:115], v[156:159], v[196:199], v[112:115]\n"
      "s_waitcnt lgkmcnt(0)\n"
      "ds_read_b128 v[192:195], v251 offset:6144\n"
      "ds_read_b128 v[196:199], v251 offset:14336\n"
      "v_mfma_f32_16x16x32_f16 v[20:23], v[128:131], v[200:203], v[20:23]\n"
      "v_mfma_f32_16x16x32_f16 v[20:23], v[144:147], v[204:207], v[20:23]\n"
      "v_mfma_f32_16x16x32_f16 v[52:55], v[132:135], v[200:203], v[52:55]\n"
      "v_mfma_f32_16x16x32_f16 v[52:55], v[148:151], v[204:207], v[52:55]\n"
      "v_mfma_f32_16x16x32_f16 v[84:87], v[136:139], v[200:203], v[84:87]\n"
      "v_mfma_f32_16x16x32_f16 v[84:87], v[152:155], v[204:207], v[84:87]\n"
      "v_mfma_f32_16x16x32_f16 v[116:119], v[140:143], v[200:203], v[116:119]\n"
      "v_mfma_f32_16x16x32_f16 v[116:119], v[156:159], v[204:207], v[116:119]\n"
      "s_setprio 0\n"
      "s_waitcnt lgkmcnt(0)\n"
      "ds_read_b128 v[200:203], v251 offset:7168\n"
      "ds_read_b128 v[204:207], v251 offset:15360\n"
      "v_mfma_f32_16x16x32_f16 v[24:27], v[128:131], v[192:195], v[24:27]\n"
      "s_setprio 1\n"
      "v_mfma_f32_16x16x32_f16 v[24:27], v[144:147], v[196:199], v[24:27]\n"
      "v_mfma_f32_16x16x32_f16 v[56:59], v[132:135], v[192:195], v[56:59]\n"
      "v_mfma_f32_16x16x32_f16 v[56:59], v[148:151], v[196:199], v[56:59]\n"
      "v_mfma_f32_16x16x32_f16 v[88:91], v[136:139], v[192:195], v[88:91]\n"
      "v_mfma_f32_16x16x32_f16 v[88:91], v[152:155], v[196:199], v[88:91]\n"
      "v_mfma_f32_16x16x32_f16 v[120:123], v[140:143], v[192:195], v[120:123]\n"
      "v_mfma_f32_16x16x32_f16 v[120:123], v[156:159], v[196:199], v[120:123]\n"
      "s_waitcnt lgkmcnt(0)\n"
      "ds_read_b128 v[192:195], v251 offset:16384\n"
      "ds_read_b128 v[196:199], v251 offset:24576\n"
      "v_mfma_f32_16x16x32_f16 v[28:31], v[128:131], v[200:203], v[28:31]\n"
      "v_mfma_f32_16x16x32_f16 v[28:31], v[144:147], v[204:207], v[28:31]\n"
      "v_mfma_f32_16x16x32_f16 v[60:63], v[132:135], v[200:203], v[60:63]\n"
      "v_mfma_f32_16x16x32_f16 v[60:63], v[148:151], v[204:207], v[60:63]\n"
      "v_mfma_f32_16x16x32_f16 v[92:95], v[136:139], v[200:203], v[92:95]\n"
      "v_mfma_f32_16x16x32_f16 v[92:95], v[152:155], v[204:207], v[92:95]\n"
      "v_mfma_f32_16x16x32_f16 v[124:127], v[140:143], v[200:203], v[124:127]\n"
      "v_mfma_f32_16x16x32_f16 v[124:127], v[156:159], v[204:207], v[124:127]\n"
      "s_setprio 0\n"
      "s_waitcnt lgkmcnt(0)\n"
      "ds_read_b128 v[200:203], v251 offset:17408\n"
      "ds_read_b128 v[204:207], v251 offset:25600\n"
      "s_waitcnt vmcnt(10)\n"
      "v_mfma_f32_16x16x32_f16 v[0:3], v[160:163], v[192:195], v[0:3]\n"
      "s_setprio 1\n"
      "v_mfma_f32_16x16x32_f16 v[0:3], v[176:179], v[196:199], v[0:3]\n"
      "s_waitcnt vmcnt(8)\n"
      "v_mfma_f32_16x16x32_f16 v[32:35], v[164:167], v[192:195], v[32:35]\n"
      "v_mfma_f32_16x16x32_f16 v[32:35], v[180:183], v[196:199], v[32:35]\n"
      "s_waitcnt vmcnt(6)\n"
      "v_mfma_f32_16x16x32_f16 v[64:67], v[168:171], v[192:195], v[64:67]\n"
      "v_mfma_f32_16x16x32_f16 v[64:67], v[184:187], v[196:199], v[64:67]\n"
      "s_waitcnt vmcnt(4)\n"
      "v_mfma_f32_16x16x32_f16 v[96:99], v[172:175], v[192:195], v[96:99]\n"
      "v_mfma_f32_16x16x32_f16 v[96:99], v[188:191], v[196:199], v[96:99]\n"
      "s_waitcnt lgkmcnt(0)\n"
      "ds_read_b128 v[192:195], v251 offset:18432\n"
      "ds_read_b128 v[196:199], v251 offset:26624\n"
      "v_mfma_f32_16x16x32_f16 v[4:7], v[160:163], v[200:203], v[4:7]\n"
      "v_mfma_f32_16x16x32_f16 v[4:7], v[176:179], v[204:207], v[4:7]\n"
      "v_mfma_f32_16x16x32_f16 v[36:39], v[164:167], v[200:203], v[36:39]\n"
      "v_mfma_f32_16x16x32_f16 v[36:39], v[180:183], v[204:207], v[36:39]\n"
      "v_mfma_f32_16x16x32_f16 v[68:71], v[168:171], v[200:203], v[68:71]\n"
      "v_mfma_f32_16x16x32_f16 v[68:71], v[184:187], v[204:207], v[68:71]\n"
      "v_mfma_f32_16x16x32_f16 v[100:103], v[172:175], v[200:203], v[100:103]\n"
      "v_mfma_f32_16x16x32_f16 v[100:103], v[188:191], v[204:207], v[100:103]\n"
      "s_setprio 0\n"
      "s_cmp_eq_u32 s28, 4\n"
      "s_cbranch_scc1 .Lsvc9_%=\n"
      ".Lsvcb10_%=:\n"
      "s_barrier\n"
      "s_sub_u32 s60, s41, 0\n"
      "s_cmp_lt_u32 s60, 10\n"
      "s_cbranch_scc1 .Lth15_%=\n"
      ".Lthb16_%=:\n"
      "s_add_u32 s42, s42, 65536\n"
      "s_add_u32 s43, s43, 65536\n"
      "buffer_load_dwordx4 v[128:131], v248, s[20:23], s42 offen offset:0 sc0 sc1\n"
      "buffer_load_dwordx4 v[144:147], v248, s[20:23], s43 offen offset:0 sc0 sc1\n"
      "s_waitcnt lgkmcnt(0)\n"
      "ds_read_b128 v[200:203], v251 offset:19456\n"
      "ds_read_b128 v[204:207], v251 offset:27648\n"
      "v_mfma_f32_16x16x32_f16 v[8:11], v[160:163], v[192:195], v[8:11]\n"
      "s_setprio 1\n"
      "v_mfma_f32_16x16x32_f16 v[8:11], v[176:179], v[196:199], v[8:11]\n"
      "v_mfma_f32_16x16x32_f16 v[40:43], v[164:167], v[192:195], v[40:43]\n"
      "v_mfma_f32_16x16x32_f16 v[40:43], v[180:183], v[196:199], v[40:43]\n"
      "v_mfma_f32_16x16x32_f16 v[72:75], v[168:171], v[192:195], v[72:75]\n"
      "v_mfma_f32_16x16x32_f16 v[72:75], v[184:187], v[196:199], v[72:75]\n"
      "v_mfma_f32_16x16x32_f16 v[104:107], v[172:175], v[192:195], v[104:107]\n"
      "v_mfma_f32_16x16x32_f16 v[104:107], v[188:191], v[196:199], v[104:107]\n"
      "buffer_load_dwordx4 v[132:135], v248, s[20:23], s42 offen offset:1024 sc0 sc1\n"
      "buffer_load_dwordx4 v[148:151], v248, s[20:23], s43 offen offset:1024 sc0 sc1\n"
      "s_waitcnt lgkmcnt(0)\n"
      "ds_read_b128 v[192:195], v251 offset:20480\n"
      "ds_read_b128 v[196:199], v251 offset:28672\n"
      "v_mfma_f32_16x16x32_f16 v[12:15], v[160:163], v[200:203], v[12:15]\n"
      "v_mfma_f32_16x16x32_f16 v[12:15], v[176:179], v[204:207], v[12:15]\n"
      "v_mfma_f32_16x16x32_f16 v[44:47], v[164:167], v[200:203], v[44:47]\n"
      "v_mfma_f32_16x16x32_f16 v[44:47], v[180:183], v[204:207], v[44:47]\n"
      "v_mfma_f32_16x16x32_f16 v[76:79], v[168:171], v[200:203], v[76:79]\n"
      "v_mfma_f32_16x16x32_f16 v[76:79], v[184:187], v[204:207], v[76:79]\n"
      "v_mfma_f32_16x16x32_f16 v[108:111], v[172:175], v[200:203], v[108:111]\n"
      "v_mfma_f32_16x16x32_f16 v[108:111], v[188:191], v[204:207], v[108:111]\n"
      "s_setprio 0\n"
      "buffer_load_dwordx4 v[136:139], v248, s[20:23], s42 offen offset:2048 sc0 sc1\n"
      "buffer_load_dwordx4 v[152:155], v248, s[20:23], s43 offen offset:2048 sc0 sc1\n"
      "s_waitcnt lgkmcnt(0)\n"
      "ds_read_b128 v[200:203], v251 offset:21504\n"
      "ds_read_b128 v[204:207], v251 offset:29696\n"
      "v_mfma_f32_16x16x32_f16 v[16:19], v[160:163], v[192:195], v[16:19]\n"
      "s_setprio 1\n"
      "v_mfma_f32_16x16x32_f16 v[16:19], v[176:179], v[196:199], v[16:19]\n"
      "v_mfma_f32_16x16x32_f16 v[48:51], v[164:167], v[192:195], v[48:51]\n"
      "v_mfma_f32_16x16x32_f16 v[48:51], v[180:183], v[196:199], v[48:51]\n"
      "v_mfma_f32_16x16x32_f16 v[80:83], v[168:171], v[192:195], v[80:83]\n"
      "v_mfma_f32_16x16x32_f16 v[80:83], v[184:187], v[196:199], v[80:83]\n"
      "v_mfma_f32_16x16x32_f16 v[112:115], v[172:175], v[192:195], v[112:115]\n"
      "v_mfma_f32_16x16x32_f16 v[112:115], v[188:191], v[196:199], v[112:115]\n"
      "buffer_load_dwordx4 v[140:143], v248, s[20:23], s42 offen offset:3072 sc0 sc1\n"
      "buffer_load_dwordx4 v[156:159], v248, s[20:23], s43 offen offset:3072 sc0 sc1\n"
      "s_waitcnt lgkmcnt(0)\n"
      "ds_read_b128 v[192:195], v251 offset:22528\n"
      "ds_read_b128 v[196:199], v251 offset:30720\n"
      "v_mfma_f32_16x16x32_f16 v[20:23], v[160:163], v[200:203], v[20:23]\n"
      "v_mfma_f32_16x16x32_f16 v[20:23], v[176:179], v[204:207], v[20:23]\n"
      "v_mfma_f32_16x16x32_f16 v[52:55], v[164:167], v[200:203], v[52:55]\n"
      "v_mfma_f32_16x16x32_f16 v[52:55], v[180:183], v[204:207], v[52:55]\n"
      "v_mfma_f32_16x16x32_f16 v[84:87], v[168:171], v[200:203], v[84:87]\n"
      "v_mfma_f32_16x16x32_f16 v[84:87], v[184:187], v[204:207], v[84:87]\n"
      "v_mfma_f32_16x16x32_f16 v[116:119], v[172:175], v[200:203], v[116:119]\n"
      "v_mfma_f32_16x16x32_f16 v[116:119], v[188:191], v[204:207], v[116:119]\n"
      "s_setprio 0\n"
      "s_waitcnt vmcnt(8)\n"
      "v_cvt_pk_f16_f32 v240, v208, v212\n"
      "v_cvt_pk_f16_f32 v241, v216, v220\n"
      "ds_write_b64 v252, v[240:241] offset:0\n"
      "v_cvt_pk_f16_f32 v242, v209, v213\n"
      "v_cvt_pk_f16_f32 v243, v217, v221\n"
      "ds_write_b64 v252, v[242:243] offset:1024\n"
      "v_cvt_pk_f16_f32 v244, v210, v214\n"
      "v_cvt_pk_f16_f32 v245, v218, v222\n"
      "ds_write_b64 v252, v[244:245] offset:2048\n"
      "v_cvt_pk_f16_f32 v246, v211, v215\n"
      "v_cvt_pk_f16_f32 v247, v219, v223\n"
      "ds_write_b64 v252, v[246:247] offset:3072\n"
      "s_and_b32 s45, s44, -2\n"
      "s_xor_b32 s45, s45, 524288\n"
      "s_add_u32 s46, s45, 65536\n"
      "buffer_load_dwordx4 v[208:211], v249, s[8:11], s45 offen\n"
      "buffer_load_dwordx4 v[212:215], v250, s[8:11], s45 offen\n"
      "buffer_load_dwordx4 v[216:219], v249, s[8:11], s46 offen\n"
      "buffer_load_dwordx4 v[220:223], v250, s[8:11], s46 offen\n"
      "s_waitcnt lgkmcnt(4)\n"
      "ds_read_b128 v[200:203], v251 offset:23552\n"
      "ds_read_b128 v[204:207], v251 offset:31744\n"
      "v_mfma_f32_16x16x32_f16 v[24:27], v[160:163], v[192:195], v[24:27]\n"
      "s_setprio 1\n"
      "v_mfma_f32_16x16x32_f16 v[24:27], v[176:179], v[196:199], v[24:27]\n"
      "v_mfma_f32_16x16x32_f16 v[56:59], v[164:167], v[192:195], v[56:59]\n"
      "v_mfma_f32_16x16x32_f16 v[56:59], v[180:183], v[196:199], v[56:59]\n"
      "v_mfma_f32_16x16x32_f16 v[88:91], v[168:171], v[192:195], v[88:91]\n"
      "v_mfma_f32_16x16x32_f16 v[88:91], v[184:187], v[196:199], v[88:91]\n"
      "v_mfma_f32_16x16x32_f16 v[120:123], v[172:175], v[192:195], v[120:123]\n"
      "v_mfma_f32_16x16x32_f16 v[120:123], v[188:191], v[196:199], v[120:123]\n"
      "s_waitcnt lgkmcnt(0)\n"
      "v_add_u32 v251, 0x8000, v251\n"
      "v_and_b32 v251, 0x1ffff, v251\n"
      "ds_read_b128 v[192:195], v251 offset:0\n"
      "ds_read_b128 v[196:199], v251 offset:8192\n"
      "v_mfma_f32_16x16x32_f16 v[28:31], v[160:163], v[200:203], v[28:31]\n"
      "v_mfma_f32_16x16x32_f16 v[28:31], v[176:179], v[204:207], v[28:31]\n"
      "v_mfma_f32_16x16x32_f16 v[60:63], v[164:167], v[200:203], v[60:63]\n"
      "v_mfma_f32_16x16x32_f16 v[60:63], v[180:183], v[204:207], v[60:63]\n"
      "v_mfma_f32_16x16x32_f16 v[92:95], v[168:171], v[200:203], v[92:95]\n"
      "v_mfma_f32_16x16x32_f16 v[92:95], v[184:187], v[204:207], v[92:95]\n"
      "v_mfma_f32_16x16x32_f16 v[124:127], v[172:175], v[200:203], v[124:127]\n"
      "v_mfma_f32_16x16x32_f16 v[124:127], v[188:191], v[204:207], v[124:127]\n"
      "s_setprio 0\n"
      ".Lnext_%=:\n"
      "s_add_u32 s54, s54, 1\n"
      "s_add_u32 s41, s41, 1\n"
      "s_cmp_le_u32 s41, s37\n"
      "s_cbranch_scc1 .Ltop_%=\n"
      "ds_read_b128 v[160:163], v251 offset:0\n"
      "ds_read_b128 v[176:179], v251 offset:8192\n"
      "ds_read_b128 v[164:167], v251 offset:1024\n"
      "ds_read_b128 v[180:183], v251 offset:9216\n"
      "ds_read_b128 v[168:171], v251 offset:2048\n"
      "ds_read_b128 v[184:187], v251 offset:10240\n"
      "ds_read_b128 v[172:175], v251 offset:3072\n"
      "ds_read_b128 v[188:191], v251 offset:11264\n"
      "v_readlane_b32 s44, v254, s54\n"
      "s_cmp_lt_u32 s28, 4\n"
      "s_setprio 2\n"
      "s_cbranch_scc1 .Lpr45_%=\n"
      "s_setprio 3\n"
      ".Lpr45_%=:\n"
      "s_and_b32 s45, s44, -2\n"
      "s_add_u32 s46, s45, 65536\n"
      "buffer_load_dwordx4 v[224:227], v249, s[8:11], s45 offen\n"
      "buffer_load_dwordx4 v[228:231], v250, s[8:11], s45 offen\n"
      "buffer_load_dwordx4 v[232:235], v249, s[8:11], s46 offen\n"
      "buffer_load_dwordx4 v[236:239], v250, s[8:11], s46 offen\n"
      "s_add_u32 s42, s42, 65536\n"
      "s_add_u32 s43, s43, 65536\n"
      "s_waitcnt vmcnt(14)\n"
      "s_waitcnt lgkmcnt(6)\n"
      "v_mfma_f32_16x16x32_f16 v[64:67], v[128:131], v[160:163], v[64:67]\n"
      "v_mfma_f32_16x16x32_f16 v[64:67], v[144:147], v[176:179], v[64:67]\n"
      "s_waitcnt lgkmcnt(4)\n"
      "v_mfma_f32_16x16x32_f16 v[68:71], v[128:131], v[164:167], v[68:71]\n"
      "v_mfma_f32_16x16x32_f16 v[68:71], v[144:147], v[180:183], v[68:71]\n"
      "s_waitcnt lgkmcnt(2)\n"
      "v_mfma_f32_16x16x32_f16 v[72:75], v[128:131], v[168:171], v[72:75]\n"
      "v_mfma_f32_16x16x32_f16 v[72:75], v[144:147], v[184:187], v[72:75]\n"
      "s_waitcnt lgkmcnt(0)\n"
      "v_mfma_f32_16x16x32_f16 v[76:79], v[128:131], v[172:175], v[76:79]\n"
      "v_mfma_f32_16x16x32_f16 v[76:79], v[144:147], v[188:191], v[76:79]\n"
      "s_waitcnt vmcnt(4)\n"
      "v_cvt_pk_f16_f32 v240, v208, v212\n"
      "v_cvt_pk_f16_f32 v241, v216, v220\n"
      "ds_write_b64 v252, v[240:241] offset:8\n"
      "v_cvt_pk_f16_f32 v242, v209, v213\n"
      "v_cvt_pk_f16_f32 v243, v217, v221\n"
      "ds_write_b64 v252, v[242:243] offset:1032\n"
      "v_cvt_pk_f16_f32 v244, v210, v214\n"
      "v_cvt_pk_f16_f32 v245, v218, v222\n"
      "ds_write_b64 v252, v[244:245] offset:2056\n"
      "v_cvt_pk_f16_f32 v246, v211, v215\n"
      "v_cvt_pk_f16_f32 v247, v219, v223\n"
      "ds_write_b64 v252, v[246:247] offset:3080\n"
      "v_add_u32 v252, 0x8000, v252\n"
      "v_and_b32 v252, 0x1ffff, v252\n"
      "s_and_b32 s45, s44, -2\n"
      "s_xor_b32 s45, s45, 524288\n"
      "s_add_u32 s46, s45, 65536\n"
      "buffer_load_dwordx4 v[208:211], v249, s[8:11], s45 offen\n"
      "buffer_load_dwordx4 v[212:215], v250, s[8:11], s45 offen\n"
      "buffer_load_dwordx4 v[216:219], v249, s[8:11], s46 offen\n"
      "buffer_load_dwordx4 v[220:223], v250, s[8:11], s46 offen\n"
      "v_mfma_f32_16x16x32_f16 v[96:99], v[132:135], v[160:163], v[96:99]\n"
      "v_mfma_f32_16x16x32_f16 v[96:99], v[148:151], v[176:179], v[96:99]\n"
      "ds_read_b128 v[160:163], v251 offset:4096\n"
      "ds_read_b128 v[176:179], v251 offset:12288\n"
      "v_mfma_f32_16x16x32_f16 v[100:103], v[132:135], v[164:167], v[100:103]\n"
      "v_mfma_f32_16x16x32_f16 v[100:103], v[148:151], v[180:183], v[100:103]\n"
      "ds_read_b128 v[164:167], v251 offset:5120\n"
      "ds_read_b128 v[180:183], v251 offset:13312\n"
      "v_mfma_f32_16x16x32_f16 v[104:107], v[132:135], v[168:171], v[104:107]\n"
      "v_mfma_f32_16x16x32_f16 v[104:107], v[148:151], v[184:187], v[104:107]\n"
      "ds_read_b128 v[168:171], v251 offset:6144\n"
      "ds_read_b128 v[184:187], v251 offset:14336\n"
      "v_mfma_f32_16x16x32_f16 v[108:111], v[132:135], v[172:175], v[108:111]\n"
      "v_mfma_f32_16x16x32_f16 v[108:111], v[148:151], v[188:191], v[108:111]\n"
      "ds_read_b128 v[172:175], v251 offset:7168\n"
      "ds_read_b128 v[188:191], v251 offset:15360\n"
      "s_waitcnt lgkmcnt(6)\n"
      "v_mfma_f32_16x16x32_f16 v[80:83], v[128:131], v[160:163], v[80:83]\n"
      "v_mfma_f32_16x16x32_f16 v[80:83], v[144:147], v[176:179], v[80:83]\n"
      "s_waitcnt lgkmcnt(4)\n"
      "v_mfma_f32_16x16x32_f16 v[84:87], v[128:131], v[164:167], v[84:87]\n"
      "v_mfma_f32_16x16x32_f16 v[84:87], v[144:147], v[180:183], v[84:87]\n"
      "s_waitcnt lgkmcnt(2)\n"
      "v_mfma_f32_16x16x32_f16 v[88:91], v[128:131], v[168:171], v[88:91]\n"
      "v_mfma_f32_16x16x32_f16 v[88:91], v[144:147], v[184:187], v[88:91]\n"
      "s_waitcnt lgkmcnt(0)\n"
      "v_mfma_f32_16x16x32_f16 v[92:95], v[128:131], v[172:175], v[92:95]\n"
      "v_mfma_f32_16x16x32_f16 v[92:95], v[144:147], v[188:191], v[92:95]\n"
      "buffer_load_dwordx4 v[128:131], v248, s[20:23], s42 offen offset:0 sc0 sc1\n"
      "buffer_load_dwordx4 v[144:147], v248, s[20:23], s43 offen offset:0 sc0 sc1\n"
      "v_mfma_f32_16x16x32_f16 v[112:115], v[132:135], v[160:163], v[112:115]\n"
      "v_mfma_f32_16x16x32_f16 v[112:115], v[148:151], v[176:179], v[112:115]\n"
      "ds_read_b128 v[160:163], v251 offset:16384\n"
      "ds_read_b128 v[176:179], v251 offset:24576\n"
      "v_mfma_f32_16x16x32_f16 v[116:119], v[132:135], v[164:167], v[116:119]\n"
      "v_mfma_f32_16x16x32_f16 v[116:119], v[148:151], v[180:183], v[116:119]\n"
      "ds_read_b128 v[164:167], v251 offset:17408\n"
      "ds_read_b128 v[180:183], v251 offset:25600\n"
      "v_mfma_f32_16x16x32_f16 v[120:123], v[132:135], v[168:171], v[120:123]\n"
      "v_mfma_f32_16x16x32_f16 v[120:123], v[148:151], v[184:187], v[120:123]\n"
      "ds_read_b128 v[168:171], v251 offset:18432\n"
      "ds_read_b128 v[184:187], v251 offset:26624\n"
      "v_mfma_f32_16x16x32_f16 v[124:127], v[132:135], v[172:175], v[124:127]\n"
      "v_mfma_f32_16x16x32_f16 v[124:127], v[148:151], v[188:191], v[124:127]\n"
      "ds_read_b128 v[172:175], v251 offset:19456\n"
      "ds_read_b128 v[188:191], v251 offset:27648\n"
      "buffer_load_dwordx4 v[132:135], v248, s[20:23], s42 offen offset:1024 sc0 sc1\n"
      "buffer_load_dwordx4 v[148:151], v248, s[20:23], s43 offen offset:1024 sc0 sc1\n"
      "s_waitcnt lgkmcnt(6)\n"
      "v_mfma_f32_16x16x32_f16 v[64:67], v[136:139], v[160:163], v[64:67]\n"
      "v_mfma_f32_16x16x32_f16 v[64:67], v[152:155], v[176:179], v[64:67]\n"
      "s_waitcnt lgkmcnt(4)\n"
      "v_mfma_f32_16x16x32_f16 v[68:71], v[136:139], v[164:167], v[68:71]\n"
      "v_mfma_f32_16x16x32_f16 v[68:71], v[152:155], v[180:183], v[68:71]\n"
      "s_waitcnt lgkmcnt(2)\n"
      "v_mfma_f32_16x16x32_f16 v[72:75], v[136:139], v[168:171], v[72:75]\n"
      "v_mfma_f32_16x16x32_f16 v[72:75], v[152:155], v[184:187], v[72:75]\n"
      "s_waitcnt lgkmcnt(0)\n"
      "v_mfma_f32_16x16x32_f16 v[76:79], v[136:139], v[172:175], v[76:79]\n"
      "v_mfma_f32_16x16x32_f16 v[76:79], v[152:155], v[188:191], v[76:79]\n"
      "v_mfma_f32_16x16x32_f16 v[96:99], v[140:143], v[160:163], v[96:99]\n"
      "v_mfma_f32_16x16x32_f16 v[96:99], v[156:159], v[176:179], v[96:99]\n"
      "ds_read_b128 v[160:163], v251 offset:20480\n"
      "ds_read_b128 v[176:179], v251 offset:28672\n"
      "v_mfma_f32_16x16x32_f16 v[100:103], v[140:143], v[164:167], v[100:103]\n"
      "v_mfma_f32_16x16x32_f16 v[100:103], v[156:159], v[180:183], v[100:103]\n"
      "ds_read_b128 v[164:167], v251 offset:21504\n"
      "ds_read_b128 v[180:183], v251 offset:29696\n"
      "v_mfma_f32_16x16x32_f16 v[104:107], v[140:143], v[168:171], v[104:107]\n"
      "v_mfma_f32_16x16x32_f16 v[104:107], v[156:159], v[184:187], v[104:107]\n"
      "ds_read_b128 v[168:171], v251 offset:22528\n"
      "ds_read_b128 v[184:187], v251 offset:30720\n"
      "v_mfma_f32_16x16x32_f16 v[108:111], v[140:143], v[172:175], v[108:111]\n"
      "v_mfma_f32_16x16x32_f16 v[108:111], v[156:159], v[188:191], v[108:111]\n"
      "ds_read_b128 v[172:175], v251 offset:23552\n"
      "ds_read_b128 v[188:191], v251 offset:31744\n"
      "s_waitcnt lgkmcnt(6)\n"
      "v_mfma_f32_16x16x32_f16 v[80:83], v[136:139], v[160:163], v[80:83]\n"
      "v_mfma_f32_16x16x32_f16 v[80:83], v[152:155], v[176:179], v[80:83]\n"
      "s_waitcnt lgkmcnt(4)\n"
      "v_mfma_f32_16x16x32_f16 v[84:87], v[136:139], v[164:167], v[84:87]\n"
      "v_mfma_f32_16x16x32_f16 v[84:87], v[152:155], v[180:183], v[84:87]\n"
      "s_waitcnt lgkmcnt(2)\n"
      "v_mfma_f32_16x16x32_f16 v[88:91], v[136:139], v[168:171], v[88:91]\n"
      "v_mfma_f32_16x16x32_f16 v[88:91], v[152:155], v[184:187], v[88:91]\n"
      "s_waitcnt lgkmcnt(0)\n"
      "v_mfma_f32_16x16x32_f16 v[92:95], v[136:139], v[172:175], v[92:95]\n"
      "v_mfma_f32_16x16x32_f16 v[92:95], v[152:155], v[188:191], v[92:95]\n"
      "buffer_load_dwordx4 v[136:139], v248, s[20:23], s42 offen offset:2048 sc0 sc1\n"
      "buffer_load_dwordx4 v[152:155], v248, s[20:23], s43 offen offset:2048 sc0 sc1\n"
      "s_barrier\n"
      "v_add_u32 v251, 0x8000, v251\n"
      "v_and_b32 v251, 0x1ffff, v251\n"
      "v_mfma_f32_16x16x32_f16 v[112:115], v[140:143], v[160:163], v[112:115]\n"
      "v_mfma_f32_16x16x32_f16 v[112:115], v[156:159], v[176:179], v[112:115]\n"
      "ds_read_b128 v[160:163], v251 offset:0\n"
      "ds_read_b128 v[176:179], v251 offset:8192\n"
      "v_mfma_f32_16x16x32_f16 v[116:119], v[140:143], v[164:167], v[116:119]\n"
      "v_mfma_f32_16x16x32_f16 v[116:119], v[156:159], v[180:183], v[116:119]\n"
      "ds_read_b128 v[164:167], v251 offset:1024\n"
      "ds_read_b128 v[180:183], v251 offset:9216\n"
      "v_mfma_f32_16x16x32_f16 v[120:123], v[140:143], v[168:171], v[120:123]\n"
      "v_mfma_f32_16x16x32_f16 v[120:123], v[156:159], v[184:187], v[120:123]\n"
      "ds_read_b128 v[168:171], v251 offset:2048\n"
      "ds_read_b128 v[184:187], v251 offset:10240\n"
      "v_mfma_f32_16x16x32_f16 v[124:127], v[140:143], v[172:175], v[124:127]\n"
      "v_mfma_f32_16x16x32_f16 v[124:127], v[156:159], v[188:191], v[124:127]\n"
      "ds_read_b128 v[172:175], v251 offset:3072\n"
      "ds_read_b128 v[188:191], v251 offset:11264\n"
      "buffer_load_dwordx4 v[140:143], v248, s[20:23], s42 offen offset:3072 sc0 sc1\n"
      "buffer_load_dwordx4 v[156:159], v248, s[20:23], s43 offen offset:3072 sc0 sc1\n"
      "s_add_u32 s42, s42, 65536\n"
      "s_add_u32 s43, s43, 65536\n"
      "s_waitcnt vmcnt(6)\n"
      "s_waitcnt lgkmcnt(6)\n"
      "v_mfma_f32_16x16x32_f16 v[64:67], v[128:131], v[160:163], v[64:67]\n"
      "v_mfma_f32_16x16x32_f16 v[64:67], v[144:147], v[176:179], v[64:67]\n"
      "s_waitcnt lgkmcnt(4)\n"
      "v_mfma_f32_16x16x32_f16 v[68:71], v[128:131], v[164:167], v[68:71]\n"
      "v_mfma_f32_16x16x32_f16 v[68:71], v[144:147], v[180:183], v[68:71]\n"
      "s_waitcnt lgkmcnt(2)\n"
      "v_mfma_f32_16x16x32_f16 v[72:75], v[128:131], v[168:171], v[72:75]\n"
      "v_mfma_f32_16x16x32_f16 v[72:75], v[144:147], v[184:187], v[72:75]\n"
      "s_waitcnt lgkmcnt(0)\n"
      "v_mfma_f32_16x16x32_f16 v[76:79], v[128:131], v[172:175], v[76:79]\n"
      "v_mfma_f32_16x16x32_f16 v[76:79], v[144:147], v[188:191], v[76:79]\n"
      "v_cvt_pk_f16_f32 v240, v224, v228\n"
      "v_cvt_pk_f16_f32 v241, v232, v236\n"
      "ds_write_b64 v252, v[240:241] offset:0\n"
      "v_cvt_pk_f16_f32 v242, v225, v229\n"
      "v_cvt_pk_f16_f32 v243, v233, v237\n"
      "ds_write_b64 v252, v[242:243] offset:1024\n"
      "v_cvt_pk_f16_f32 v244, v226, v230\n"
      "v_cvt_pk_f16_f32 v245, v234, v238\n"
      "ds_write_b64 v252, v[244:245] offset:2048\n"
      "v_cvt_pk_f16_f32 v246, v227, v231\n"
      "v_cvt_pk_f16_f32 v247, v235, v239\n"
      "ds_write_b64 v252, v[246:247] offset:3072\n"
      "s_waitcnt vmcnt(4)\n"
      "v_mfma_f32_16x16x32_f16 v[96:99], v[132:135], v[160:163], v[96:99]\n"
      "v_mfma_f32_16x16x32_f16 v[96:99], v[148:151], v[176:179], v[96:99]\n"
      "ds_read_b128 v[160:163], v251 offset:4096\n"
      "ds_read_b128 v[176:179], v251 offset:12288\n"
      "v_mfma_f32_16x16x32_f16 v[100:103], v[132:135], v[164:167], v[100:103]\n"
      "v_mfma_f32_16x16x32_f16 v[100:103], v[148:151], v[180:183], v[100:103]\n"
      "ds_read_b128 v[164:167], v251 offset:5120\n"
      "ds_read_b128 v[180:183], v251 offset:13312\n"
      "v_mfma_f32_16x16x32_f16 v[104:107], v[132:135], v[168:171], v[104:107]\n"
      "v_mfma_f32_16x16x32_f16 v[104:107], v[148:151], v[184:187], v[104:107]\n"
      "ds_read_b128 v[168:171], v251 offset:6144\n"
      "ds_read_b128 v[184:187], v251 offset:14336\n"
      "v_mfma_f32_16x16x32_f16 v[108:111], v[132:135], v[172:175], v[108:111]\n"
      "v_mfma_f32_16x16x32_f16 v[108:111], v[148:151], v[188:191], v[108:111]\n"
      "ds_read_b128 v[172:175], v251 offset:7168\n"
      "ds_read_b128 v[188:191], v251 offset:15360\n"
      "s_waitcnt lgkmcnt(6)\n"
      "v_mfma_f32_16x16x32_f16 v[80:83], v[128:131], v[160:163], v[80:83]\n"
      "v_mfma_f32_16x16x32_f16 v[80:83], v[144:147], v[176:179], v[80:83]\n"
      "s_waitcnt lgkmcnt(4)\n"
      "v_mfma_f32_16x16x32_f16 v[84:87], v[128:131], v[164:167], v[84:87]\n"
      "v_mfma_f32_16x16x32_f16 v[84:87], v[144:147], v[180:183], v[84:87]\n"
      "s_waitcnt lgkmcnt(2)\n"
      "v_mfma_f32_16x16x32_f16 v[88:91], v[128:131], v[168:171], v[88:91]\n"
      "v_mfma_f32_16x16x32_f16 v[88:91], v[144:147], v[184:187], v[88:91]\n"
      "s_waitcnt lgkmcnt(0)\n"
      "v_mfma_f32_16x16x32_f16 v[92:95], v[128:131], v[172:175], v[92:95]\n"
      "v_mfma_f32_16x16x32_f16 v[92:95], v[144:147], v[188:191], v[92:95]\n"
      "buffer_load_dwordx4 v[128:131], v248, s[20:23], s42 offen offset:0 sc0 sc1\n"
      "buffer_load_dwordx4 v[144:147], v248, s[20:23], s43 offen offset:0 sc0 sc1\n"
      "v_cvt_pk_f16_f32 v240, v208, v212\n"
      "v_cvt_pk_f16_f32 v241, v216, v220\n"
      "ds_write_b64 v252, v[240:241] offset:8\n"
      "v_cvt_pk_f16_f32 v242, v209, v213\n"
      "v_cvt_pk_f16_f32 v243, v217, v221\n"
      "ds_write_b64 v252, v[242:243] offset:1032\n"
      "v_cvt_pk_f16_f32 v244, v210, v214\n"
      "v_cvt_pk_f16_f32 v245, v218, v222\n"
      "ds_write_b64 v252, v[244:245] offset:2056\n"
      "v_cvt_pk_f16_f32 v246, v211, v215\n"
      "v_cvt_pk_f16_f32 v247, v219, v223\n"
      "ds_write_b64 v252, v[246:247] offset:3080\n"
      "v_mfma_f32_16x16x32_f16 v[112:115], v[132:135], v[160:163], v[112:115]\n"
      "v_mfma_f32_16x16x32_f16 v[112:115], v[148:151], v[176:179], v[112:115]\n"
      "ds_read_b128 v[160:163], v251 offset:16384\n"
      "ds_read_b128 v[176:179], v251 offset:24576\n"
      "v_mfma_f32_16x16x32_f16 v[116:119], v[132:135], v[164:167], v[116:119]\n"
      "v_mfma_f32_16x16x32_f16 v[116:119], v[148:151], v[180:183], v[116:119]\n"
      "ds_read_b128 v[164:167], v251 offset:17408\n"
      "ds_read_b128 v[180:183], v251 offset:25600\n"
      "v_mfma_f32_16x16x32_f16 v[120:123], v[132:135], v[168:171], v[120:123]\n"
      "v_mfma_f32_16x16x32_f16 v[120:123], v[148:151], v[184:187], v[120:123]\n"
      "ds_read_b128 v[168:171], v251 offset:18432\n"
      "ds_read_b128 v[184:187], v251 offset:26624\n"
      "v_mfma_f32_16x16x32_f16 v[124:127], v[132:135], v[172:175], v[124:127]\n"
      "v_mfma_f32_16x16x32_f16 v[124:127], v[148:151], v[188:191], v[124:127]\n"
      "ds_read_b128 v[172:175], v251 offset:19456\n"
      "ds_read_b128 v[188:191], v251 offset:27648\n"
      "buffer_load_dwordx4 v[132:135], v248, s[20:23], s42 offen offset:1024 sc0 sc1\n"
      "buffer_load_dwordx4 v[148:151], v248, s[20:23], s43 offen offset:1024 sc0 sc1\n"
      "s_waitcnt vmcnt(6)\n"
      "s_waitcnt lgkmcnt(6)\n"
      "v_mfma_f32_16x16x32_f16 v[64:67], v[136:139], v[160:163], v[64:67]\n"
      "v_mfma_f32_16x16x32_f16 v[64:67], v[152:155], v[176:179], v[64:67]\n"
      "s_waitcnt lgkmcnt(4)\n"
      "v_mfma_f32_16x16x32_f16 v[68:71], v[136:139], v[164:167], v[68:71]\n"
      "v_mfma_f32_16x16x32_f16 v[68:71], v[152:155], v[180:183], v[68:71]\n"
      "s_waitcnt lgkmcnt(2)\n"
      "v_mfma_f32_16x16x32_f16 v[72:75], v[136:139], v[168:171], v[72:75]\n"
      "v_mfma_f32_16x16x32_f16 v[72:75], v[152:155], v[184:187], v[72:75]\n"
      "s_waitcnt lgkmcnt(0)\n"
      "v_mfma_f32_16x16x32_f16 v[76:79], v[136:139], v[172:175], v[76:79]\n"
      "v_mfma_f32_16x16x32_f16 v[76:79], v[152:155], v[188:191], v[76:79]\n"
      "s_waitcnt vmcnt(4)\n"
      "v_mfma_f32_16x16x32_f16 v[96:99], v[140:143], v[160:163], v[96:99]\n"
      "v_mfma_f32_16x16x32_f16 v[96:99], v[156:159], v[176:179], v[96:99]\n"
      "ds_read_b128 v[160:163], v251 offset:20480\n"
      "ds_read_b128 v[176:179], v251 offset:28672\n"
      "v_mfma_f32_16x16x32_f16 v[100:103], v[140:143], v[164:167], v[100:103]\n"
      "v_mfma_f32_16x16x32_f16 v[100:103], v[156:159], v[180:183], v[100:103]\n"
      "ds_read_b128 v[164:167], v251 offset:21504\n"
      "ds_read_b128 v[180:183], v251 offset:29696\n"
      "v_mfma_f32_16x16x32_f16 v[104:107], v[140:143], v[168:171], v[104:107]\n"
      "v_mfma_f32_16x16x32_f16 v[104:107], v[156:159], v[184:187], v[104:107]\n"
      "ds_read_b128 v[168:171], v251 offset:22528\n"
      "ds_read_b128 v[184:187], v251 offset:30720\n"
      "v_mfma_f32_16x16x32_f16 v[108:111], v[140:143], v[172:175], v[108:111]\n"
      "v_mfma_f32_16x16x32_f16 v[108:111], v[156:159], v[188:191], v[108:111]\n"
      "ds_read_b128 v[172:175], v251 offset:23552\n"
      "ds_read_b128 v[188:191], v251 offset:31744\n"
      "s_waitcnt lgkmcnt(6)\n"
      "v_mfma_f32_16x16x32_f16 v[80:83], v[136:139], v[160:163], v[80:83]\n"
      "v_mfma_f32_16x16x32_f16 v[80:83], v[152:155], v[176:179], v[80:83]\n"
      "s_waitcnt lgkmcnt(4)\n"
      "v_mfma_f32_16x16x32_f16 v[84:87], v[136:139], v[164:167], v[84:87]\n"
      "v_mfma_f32_16x16x32_f16 v[84:87], v[152:155], v[180:183], v[84:87]\n"
      "s_waitcnt lgkmcnt(2)\n"
      "v_mfma_f32_16x16x32_f16 v[88:91], v[136:139], v[168:171], v[88:91]\n"
      "v_mfma_f32_16x16x32_f16 v[88:91], v[152:155], v[184:187], v[88:91]\n"
      "s_waitcnt lgkmcnt(0)\n"
      "v_mfma_f32_16x16x32_f16 v[92:95], v[136:139], v[172:175], v[92:95]\n"
      "v_mfma_f32_16x16x32_f16 v[92:95], v[152:155], v[188:191], v[92:95]\n"
      "buffer_load_dwordx4 v[136:139], v248, s[20:23], s42 offen offset:2048 sc0 sc1\n"
      "buffer_load_dwordx4 v[152:155], v248, s[20:23], s43 offen offset:2048 sc0 sc1\n"
      "s_barrier\n"
      "v_add_u32 v251, 0x8000, v251\n"
      "v_and_b32 v251, 0x1ffff, v251\n"
      "v_mfma_f32_16x16x32_f16 v[112:115], v[140:143], v[160:163], v[112:115]\n"
      "v_mfma_f32_16x16x32_f16 v[112:115], v[156:159], v[176:179], v[112:115]\n"
      "ds_read_b128 v[160:163], v251 offset:0\n"
      "ds_read_b128 v[176:179], v251 offset:8192\n"
      "v_mfma_f32_16x16x32_f16 v[116:119], v[140:143], v[164:167], v[116:119]\n"
      "v_mfma_f32_16x16x32_f16 v[116:119], v[156:159], v[180:183], v[116:119]\n"
      "ds_read_b128 v[164:167], v251 offset:1024\n"
      "ds_read_b128 v[180:183], v251 offset:9216\n"
      "v_mfma_f32_16x16x32_f16 v[120:123], v[140:143], v[168:171], v[120:123]\n"
      "v_mfma_f32_16x16x32_f16 v[120:123], v[156:159], v[184:187], v[120:123]\n"
      "ds_read_b128 v[168:171], v251 offset:2048\n"
      "ds_read_b128 v[184:187], v251 offset:10240\n"
      "v_mfma_f32_16x16x32_f16 v[124:127], v[140:143], v[172:175], v[124:127]\n"
      "v_mfma_f32_16x16x32_f16 v[124:127], v[156:159], v[188:191], v[124:127]\n"
      "ds_read_b128 v[172:175], v251 offset:3072\n"
      "ds_read_b128 v[188:191], v251 offset:11264\n"
      "buffer_load_dwordx4 v[140:143], v248, s[20:23], s42 offen offset:3072 sc0 sc1\n"
      "buffer_load_dwordx4 v[156:159], v248, s[20:23], s43 offen offset:3072 sc0 sc1\n"
      "s_add_u32 s42, s42, 65536\n"
      "s_add_u32 s43, s43, 65536\n"
      "s_waitcnt vmcnt(6)\n"
      "s_waitcnt lgkmcnt(6)\n"
      "v_mfma_f32_16x16x32_f16 v[64:67], v[128:131], v[160:163], v[64:67]\n"
      "v_mfma_f32_16x16x32_f16 v[64:67], v[144:147], v[176:179], v[64:67]\n"
      "s_waitcnt lgkmcnt(4)\n"
      "v_mfma_f32_16x16x32_f16 v[68:71], v[128:131], v[164:167], v[68:71]\n"
      "v_mfma_f32_16x16x32_f16 v[68:71], v[144:147], v[180:183], v[68:71]\n"
      "s_waitcnt lgkmcnt(2)\n"
      "v_mfma_f32_16x16x32_f16 v[72:75], v[128:131], v[168:171], v[72:75]\n"
      "v_mfma_f32_16x16x32_f16 v[72:75], v[144:147], v[184:187], v[72:75]\n"
      "s_waitcnt lgkmcnt(0)\n"
      "v_mfma_f32_16x16x32_f16 v[76:79], v[128:131], v[172:175], v[76:79]\n"
      "v_mfma_f32_16x16x32_f16 v[76:79], v[144:147], v[188:191], v[76:79]\n"
      "s_waitcnt vmcnt(4)\n"
      "v_mfma_f32_16x16x32_f16 v[96:99], v[132:135], v[160:163], v[96:99]\n"
      "v_mfma_f32_16x16x32_f16 v[96:99], v[148:151], v[176:179], v[96:99]\n"
      "ds_read_b128 v[160:163], v251 offset:4096\n"
      "ds_read_b128 v[176:179], v251 offset:12288\n"
      "v_mfma_f32_16x16x32_f16 v[100:103], v[132:135], v[164:167], v[100:103]\n"
      "v_mfma_f32_16x16x32_f16 v[100:103], v[148:151], v[180:183], v[100:103]\n"
      "ds_read_b128 v[164:167], v251 offset:5120\n"
      "ds_read_b128 v[180:183], v251 offset:13312\n"
      "v_mfma_f32_16x16x32_f16 v[104:107], v[132:135], v[168:171], v[104:107]\n"
      "v_mfma_f32_16x16x32_f16 v[104:107], v[148:151], v[184:187], v[104:107]\n"
      "ds_read_b128 v[168:171], v251 offset:6144\n"
      "ds_read_b128 v[184:187], v251 offset:14336\n"
      "v_mfma_f32_16x16x32_f16 v[108:111], v[132:135], v[172:175], v[108:111]\n"
      "v_mfma_f32_16x16x32_f16 v[108:111], v[148:151], v[188:191], v[108:111]\n"
      "ds_read_b128 v[172:175], v251 offset:7168\n"
      "ds_read_b128 v[188:191], v251 offset:15360\n"
      "s_waitcnt lgkmcnt(6)\n"
      "v_mfma_f32_16x16x32_f16 v[80:83], v[128:131], v[160:163], v[80:83]\n"
      "v_mfma_f32_16x16x32_f16 v[80:83], v[144:147], v[176:179], v[80:83]\n"
      "s_waitcnt lgkmcnt(4)\n"
      "v_mfma_f32_16x16x32_f16 v[84:87], v[128:131], v[164:167], v[84:87]\n"
      "v_mfma_f32_16x16x32_f16 v[84:87], v[144:147], v[180:183], v[84:87]\n"
      "s_waitcnt lgkmcnt(2)\n"
      "v_mfma_f32_16x16x32_f16 v[88:91], v[128:131], v[168:171], v[88:91]\n"
      "v_mfma_f32_16x16x32_f16 v[88:91], v[144:147], v[184:187], v[88:91]\n"
      "s_waitcnt lgkmcnt(0)\n"
      "v_mfma_f32_16x16x32_f16 v[92:95], v[128:131], v[172:175], v[92:95]\n"
      "v_mfma_f32_16x16x32_f16 v[92:95], v[144:147], v[188:191], v[92:95]\n"
      "buffer_load_dwordx4 v[128:131], v248, s[20:23], s42 offen offset:0 sc0 sc1\n"
      "buffer_load_dwordx4 v[144:147], v248, s[20:23], s43 offen offset:0 sc0 sc1\n"
      "v_mfma_f32_16x16x32_f16 v[112:115], v[132:135], v[160:163], v[112:115]\n"
      "v_mfma_f32_16x16x32_f16 v[112:115], v[148:151], v[176:179], v[112:115]\n"
      "ds_read_b128 v[160:163], v251 offset:16384\n"
      "ds_read_b128 v[176:179], v251 offset:24576\n"
      "v_mfma_f32_16x16x32_f16 v[116:119], v[132:135], v[164:167], v[116:119]\n"
      "v_mfma_f32_16x16x32_f16 v[116:119], v[148:151], v[180:183], v[116:119]\n"
      "ds_read_b128 v[164:167], v251 offset:17408\n"
      "ds_read_b128 v[180:183], v251 offset:25600\n"
      "v_mfma_f32_16x16x32_f16 v[120:123], v[132:135], v[168:171], v[120:123]\n"
      "v_mfma_f32_16x16x32_f16 v[120:123], v[148:151], v[184:187], v[120:123]\n"
      "ds_read_b128 v[168:171], v251 offset:18432\n"
      "ds_read_b128 v[184:187], v251 offset:26624\n"
      "v_mfma_f32_16x16x32_f16 v[124:127], v[132:135], v[172:175], v[124:127]\n"
      "v_mfma_f32_16x16x32_f16 v[124:127], v[148:151], v[188:191], v[124:127]\n"
      "ds_read_b128 v[172:175], v251 offset:19456\n"
      "ds_read_b128 v[188:191], v251 offset:27648\n"
      "buffer_load_dwordx4 v[132:135], v248, s[20:23], s42 offen offset:1024 sc0 sc1\n"
      "buffer_load_dwordx4 v[148:151], v248, s[20:23], s43 offen offset:1024 sc0 sc1\n"
      "s_waitcnt vmcnt(6)\n"
      "s_waitcnt lgkmcnt(6)\n"
      "v_mfma_f32_16x16x32_f16 v[64:67], v[136:139], v[160:163], v[64:67]\n"
      "v_mfma_f32_16x16x32_f16 v[64:67], v[152:155], v[176:179], v[64:67]\n"
      "s_waitcnt lgkmcnt(4)\n"
      "v_mfma_f32_16x16x32_f16 v[68:71], v[136:139], v[164:167], v[68:71]\n"
      "v_mfma_f32_16x16x32_f16 v[68:71], v[152:155], v[180:183], v[68:71]\n"
      "s_waitcnt lgkmcnt(2)\n"
      "v_mfma_f32_16x16x32_f16 v[72:75], v[136:139], v[168:171], v[72:75]\n"
      "v_mfma_f32_16x16x32_f16 v[72:75], v[152:155], v[184:187], v[72:75]\n"
      "s_waitcnt lgkmcnt(0)\n"
      "v_mfma_f32_16x16x32_f16 v[76:79], v[136:139], v[172:175], v[76:79]\n"
      "v_mfma_f32_16x16x32_f16 v[76:79], v[152:155], v[188:191], v[76:79]\n"
      "s_waitcnt vmcnt(4)\n"
      "v_mfma_f32_16x16x32_f16 v[96:99], v[140:143], v[160:163], v[96:99]\n"
      "v_mfma_f32_16x16x32_f16 v[96:99], v[156:159], v[176:179], v[96:99]\n"
      "ds_read_b128 v[160:163], v251 offset:20480\n"
      "ds_read_b128 v[176:179], v251 offset:28672\n"
      "v_mfma_f32_16x16x32_f16 v[100:103], v[140:143], v[164:167], v[100:103]\n"
      "v_mfma_f32_16x16x32_f16 v[100:103], v[156:159], v[180:183], v[100:103]\n"
      "ds_read_b128 v[164:167], v251 offset:21504\n"
      "ds_read_b128 v[180:183], v251 offset:29696\n"
      "v_mfma_f32_16x16x32_f16 v[104:107], v[140:143], v[168:171], v[104:107]\n"
      "v_mfma_f32_16x16x32_f16 v[104:107], v[156:159], v[184:187], v[104:107]\n"
      "ds_read_b128 v[168:171], v251 offset:22528\n"
      "ds_read_b128 v[184:187], v251 offset:30720\n"
      "v_mfma_f32_16x16x32_f16 v[108:111], v[140:143], v[172:175], v[108:111]\n"
      "v_mfma_f32_16x16x32_f16 v[108:111], v[156:159], v[188:191], v[108:111]\n"
      "ds_read_b128 v[172:175], v251 offset:23552\n"
      "ds_read_b128 v[188:191], v251 offset:31744\n"
      "s_waitcnt lgkmcnt(6)\n"
      "v_mfma_f32_16x16x32_f16 v[80:83], v[136:139], v[160:163], v[80:83]\n"
      "v_mfma_f32_16x16x32_f16 v[80:83], v[152:155], v[176:179], v[80:83]\n"
      "s_waitcnt lgkmcnt(4)\n"
      "v_mfma_f32_16x16x32_f16 v[84:87], v[136:139], v[164:167], v[84:87]\n"
      "v_mfma_f32_16x16x32_f16 v[84:87], v[152:155], v[180:183], v[84:87]\n"
      "s_waitcnt lgkmcnt(2)\n"
      "v_mfma_f32_16x16x32_f16 v[88:91], v[136:139], v[168:171], v[88:91]\n"
      "v_mfma_f32_16x16x32_f16 v[88:91], v[152:155], v[184:187], v[88:91]\n"
      "s_waitcnt lgkmcnt(0)\n"
      "v_mfma_f32_16x16x32_f16 v[92:95], v[136:139], v[172:175], v[92:95]\n"
      "v_mfma_f32_16x16x32_f16 v[92:95], v[152:155], v[188:191], v[92:95]\n"
      "buffer_load_dwordx4 v[136:139], v248, s[20:23], s42 offen offset:2048 sc0 sc1\n"
      "buffer_load_dwordx4 v[152:155], v248, s[20:23], s43 offen offset:2048 sc0 sc1\n"
      "v_add_u32 v251, 0x8000, v251\n"
      "v_and_b32 v251, 0x1ffff, v251\n"
      "v_mfma_f32_16x16x32_f16 v[112:115], v[140:143], v[160:163], v[112:115]\n"
      "v_mfma_f32_16x16x32_f16 v[112:115], v[156:159], v[176:179], v[112:115]\n"
      "ds_read_b128 v[160:163], v251 offset:0\n"
      "ds_read_b128 v[176:179], v251 offset:8192\n"
      "v_mfma_f32_16x16x32_f16 v[116:119], v[140:143], v[164:167], v[116:119]\n"
      "v_mfma_f32_16x16x32_f16 v[116:119], v[156:159], v[180:183], v[116:119]\n"
      "ds_read_b128 v[164:167], v251 offset:1024\n"
      "ds_read_b128 v[180:183], v251 offset:9216\n"
      "v_mfma_f32_16x16x32_f16 v[120:123], v[140:143], v[168:171], v[120:123]\n"
      "v_mfma_f32_16x16x32_f16 v[120:123], v[156:159], v[184:187], v[120:123]\n"
      "ds_read_b128 v[168:171], v251 offset:2048\n"
      "ds_read_b128 v[184:187], v251 offset:10240\n"
      "v_mfma_f32_16x16x32_f16 v[124:127], v[140:143], v[172:175], v[124:127]\n"
      "v_mfma_f32_16x16x32_f16 v[124:127], v[156:159], v[188:191], v[124:127]\n"
      "ds_read_b128 v[172:175], v251 offset:3072\n"
      "ds_read_b128 v[188:191], v251 offset:11264\n"
      "buffer_load_dwordx4 v[140:143], v248, s[20:23], s42 offen offset:3072 sc0 sc1\n"
      "buffer_load_dwordx4 v[156:159], v248, s[20:23], s43 offen offset:3072 sc0 sc1\n"
      "s_add_u32 s52, s42, 131072\n"
      "s_add_u32 s53, s43, 131072\n"
      "v_and_b32 v236, 0x200, v248\n"
      "v_and_b32 v237, 0x1ff, v248\n"
      "v_lshl_add_u32 v236, v236, 6, v237\n"
      "s_add_u32 s42, s42, 65536\n"
      "s_add_u32 s43, s43, 65536\n"
      "s_waitcnt vmcnt(6)\n"
      "s_waitcnt lgkmcnt(6)\n"
      "v_mfma_f32_16x16x32_f16 v[64:67], v[128:131], v[160:163], v[64:67]\n"
      "v_mfma_f32_16x16x32_f16 v[64:67], v[144:147], v[176:179], v[64:67]\n"
      "s_waitcnt lgkmcnt(4)\n"
      "v_mfma_f32_16x16x32_f16 v[68:71], v[128:131], v[164:167], v[68:71]\n"
      "v_mfma_f32_16x16x32_f16 v[68:71], v[144:147], v[180:183], v[68:71]\n"
      "s_waitcnt lgkmcnt(2)\n"
      "v_mfma_f32_16x16x32_f16 v[72:75], v[128:131], v[168:171], v[72:75]\n"
      "v_mfma_f32_16x16x32_f16 v[72:75], v[144:147], v[184:187], v[72:75]\n"
      "s_waitcnt lgkmcnt(0)\n"
      "v_mfma_f32_16x16x32_f16 v[76:79], v[128:131], v[172:175], v[76:79]\n"
      "v_mfma_f32_16x16x32_f16 v[76:79], v[144:147], v[188:191], v[76:79]\n"
      "buffer_load_dwordx4 v[192:195], v248, s[20:23], s52 offen offset:0 sc0 sc1\n"
      "buffer_load_dwordx4 v[208:211], v248, s[20:23], s53 offen offset:0 sc0 sc1\n"
      "s_waitcnt vmcnt(6)\n"
      "v_mfma_f32_16x16x32_f16 v[96:99], v[132:135], v[160:163], v[96:99]\n"
      "v_mfma_f32_16x16x32_f16 v[96:99], v[148:151], v[176:179], v[96:99]\n"
      "ds_read_b128 v[160:163], v251 offset:4096\n"
      "ds_read_b128 v[176:179], v251 offset:12288\n"
      "v_mfma_f32_16x16x32_f16 v[100:103], v[132:135], v[164:167], v[100:103]\n"
      "v_mfma_f32_16x16x32_f16 v[100:103], v[148:151], v[180:183], v[100:103]\n"
      "ds_read_b128 v[164:167], v251 offset:5120\n"
      "ds_read_b128 v[180:183], v251 offset:13312\n"
      "v_mfma_f32_16x16x32_f16 v[104:107], v[132:135], v[168:171], v[104:107]\n"
      "v_mfma_f32_16x16x32_f16 v[104:107], v[148:151], v[184:187], v[104:107]\n"
      "ds_read_b128 v[168:171], v251 offset:6144\n"
      "ds_read_b128 v[184:187], v251 offset:14336\n"
      "v_mfma_f32_16x16x32_f16 v[108:111], v[132:135], v[172:175], v[108:111]\n"
      "v_mfma_f32_16x16x32_f16 v[108:111], v[148:151], v[188:191], v[108:111]\n"
      "ds_read_b128 v[172:175], v251 offset:7168\n"
      "ds_read_b128 v[188:191], v251 offset:15360\n"
      "buffer_load_dwordx4 v[196:199], v248, s[20:23], s52 offen offset:1024 sc0 sc1\n"
      "buffer_load_dwordx4 v[212:215], v248, s[20:23], s53 offen offset:1024 sc0 sc1\n"
      "s_waitcnt lgkmcnt(6)\n"
      "v_mfma_f32_16x16x32_f16 v[80:83], v[128:131], v[160:163], v[80:83]\n"
      "v_mfma_f32_16x16x32_f16 v[80:83], v[144:147], v[176:179], v[80:83]\n"
      "s_waitcnt lgkmcnt(4)\n"
      "v_mfma_f32_16x16x32_f16 v[84:87], v[128:131], v[164:167], v[84:87]\n"
      "v_mfma_f32_16x16x32_f16 v[84:87], v[144:147], v[180:183], v[84:87]\n"
      "s_waitcnt lgkmcnt(2)\n"
      "v_mfma_f32_16x16x32_f16 v[88:91], v[128:131], v[168:171], v[88:91]\n"
      "v_mfma_f32_16x16x32_f16 v[88:91], v[144:147], v[184:187], v[88:91]\n"
      "s_waitcnt lgkmcnt(0)\n"
      "v_mfma_f32_16x16x32_f16 v[92:95], v[128:131], v[172:175], v[92:95]\n"
      "v_mfma_f32_16x16x32_f16 v[92:95], v[144:147], v[188:191], v[92:95]\n"
      "buffer_load_dwordx4 v[128:131], v248, s[20:23], s42 offen offset:0 sc0 sc1\n"
      "buffer_load_dwordx4 v[144:147], v248, s[20:23], s43 offen offset:0 sc0 sc1\n"
      "v_mfma_f32_16x16x32_f16 v[112:115], v[132:135], v[160:163], v[112:115]\n"
      "v_mfma_f32_16x16x32_f16 v[112:115], v[148:151], v[176:179], v[112:115]\n"
      "ds_read_b128 v[160:163], v251 offset:16384\n"
      "ds_read_b128 v[176:179], v251 offset:24576\n"
      "v_mfma_f32_16x16x32_f16 v[116:119], v[132:135], v[164:167], v[116:119]\n"
      "v_mfma_f32_16x16x32_f16 v[116:119], v[148:151], v[180:183], v[116:119]\n"
      "ds_read_b128 v[164:167], v251 offset:17408\n"
      "ds_read_b128 v[180:183], v251 offset:25600\n"
      "v_mfma_f32_16x16x32_f16 v[120:123], v[132:135], v[168:171], v[120:123]\n"
      "v_mfma_f32_16x16x32_f16 v[120:123], v[148:151], v[184:187], v[120:123]\n"
      "ds_read_b128 v[168:171], v251 offset:18432\n"
      "ds_read_b128 v[184:187], v251 offset:26624\n"
      "v_mfma_f32_16x16x32_f16 v[124:127], v[132:135], v[172:175], v[124:127]\n"
      "v_mfma_f32_16x16x32_f16 v[124:127], v[148:151], v[188:191], v[124:127]\n"
      "ds_read_b128 v[172:175], v251 offset:19456\n"
      "ds_read_b128 v[188:191], v251 offset:27648\n"
      "buffer_load_dwordx4 v[132:135], v248, s[20:23], s42 offen offset:1024 sc0 sc1\n"
      "buffer_load_dwordx4 v[148:151], v248, s[20:23], s43 offen offset:1024 sc0 sc1\n"
      "s_waitcnt vmcnt(10)\n"
      "s_waitcnt lgkmcnt(6)\n"
      "v_mfma_f32_16x16x32_f16 v[64:67], v[136:139], v[160:163], v[64:67]\n"
      "v_mfma_f32_16x16x32_f16 v[64:67], v[152:155], v[176:179], v[64:67]\n"
      "s_waitcnt lgkmcnt(4)\n"
      "v_mfma_f32_16x16x32_f16 v[68:71], v[136:139], v[164:167], v[68:71]\n"
      "v_mfma_f32_16x16x32_f16 v[68:71], v[152:155], v[180:183], v[68:71]\n"
      "s_waitcnt lgkmcnt(2)\n"
      "v_mfma_f32_16x16x32_f16 v[72:75], v[136:139], v[168:171], v[72:75]\n"
      "v_mfma_f32_16x16x32_f16 v[72:75], v[152:155], v[184:187], v[72:75]\n"
      "s_waitcnt lgkmcnt(0)\n"
      "v_mfma_f32_16x16x32_f16 v[76:79], v[136:139], v[172:175], v[76:79]\n"
      "v_mfma_f32_16x16x32_f16 v[76:79], v[152:155], v[188:191], v[76:79]\n"
      "buffer_load_dwordx4 v[200:203], v248, s[20:23], s52 offen offset:2048 sc0 sc1\n"
      "buffer_load_dwordx4 v[216:219], v248, s[20:23], s53 offen offset:2048 sc0 sc1\n"
      "s_waitcnt vmcnt(10)\n"
      "v_mfma_f32_16x16x32_f16 v[96:99], v[140:143], v[160:163], v[96:99]\n"
      "v_mfma_f32_16x16x32_f16 v[96:99], v[156:159], v[176:179], v[96:99]\n"
      "ds_read_b128 v[160:163], v251 offset:20480\n"
      "ds_read_b128 v[176:179], v251 offset:28672\n"
      "v_mfma_f32_16x16x32_f16 v[100:103], v[140:143], v[164:167], v[100:103]\n"
      "v_mfma_f32_16x16x32_f16 v[100:103], v[156:159], v[180:183], v[100:103]\n"
      "ds_read_b128 v[164:167], v251 offset:21504\n"
      "ds_read_b128 v[180:183], v251 offset:29696\n"
      "v_mfma_f32_16x16x32_f16 v[104:107], v[140:143], v[168:171], v[104:107]\n"
      "v_mfma_f32_16x16x32_f16 v[104:107], v[156:159], v[184:187], v[104:107]\n"
      "ds_read_b128 v[168:171], v251 offset:22528\n"
      "ds_read_b128 v[184:187], v251 offset:30720\n"
      "v_mfma_f32_16x16x32_f16 v[108:111], v[140:143], v[172:175], v[108:111]\n"
      "v_mfma_f32_16x16x32_f16 v[108:111], v[156:159], v[188:191], v[108:111]\n"
      "ds_read_b128 v[172:175], v251 offset:23552\n"
      "ds_read_b128 v[188:191], v251 offset:31744\n"
      "buffer_load_dwordx4 v[204:207], v248, s[20:23], s52 offen offset:3072 sc0 sc1\n"
      "buffer_load_dwordx4 v[220:223], v248, s[20:23], s53 offen offset:3072 sc0 sc1\n"
      "s_waitcnt lgkmcnt(6)\n"
      "v_mfma_f32_16x16x32_f16 v[80:83], v[136:139], v[160:163], v[80:83]\n"
      "v_mfma_f32_16x16x32_f16 v[80:83], v[152:155], v[176:179], v[80:83]\n"
      "s_waitcnt lgkmcnt(4)\n"
      "v_mfma_f32_16x16x32_f16 v[84:87], v[136:139], v[164:167], v[84:87]\n"
      "v_mfma_f32_16x16x32_f16 v[84:87], v[152:155], v[180:183], v[84:87]\n"
      "s_waitcnt lgkmcnt(2)\n"
      "v_mfma_f32_16x16x32_f16 v[88:91], v[136:139], v[168:171], v[88:91]\n"
      "v_mfma_f32_16x16x32_f16 v[88:91], v[152:155], v[184:187], v[88:91]\n"
      "s_waitcnt lgkmcnt(0)\n"
      "v_mfma_f32_16x16x32_f16 v[92:95], v[136:139], v[172:175], v[92:95]\n"
      "v_mfma_f32_16x16x32_f16 v[92:95], v[152:155], v[188:191], v[92:95]\n"
      "buffer_load_dwordx4 v[136:139], v248, s[20:23], s42 offen offset:2048 sc0 sc1\n"
      "buffer_load_dwordx4 v[152:155], v248, s[20:23], s43 offen offset:2048 sc0 sc1\n"
      "v_add_u32 v251, 0x8000, v251\n"
      "v_and_b32 v251, 0x1ffff, v251\n"
      "v_mfma_f32_16x16x32_f16 v[112:115], v[140:143], v[160:163], v[112:115]\n"
      "v_mfma_f32_16x16x32_f16 v[112:115], v[156:159], v[176:179], v[112:115]\n"
      "ds_read_b128 v[160:163], v251 offset:0\n"
      "ds_read_b128 v[176:179], v251 offset:8192\n"
      "v_mfma_f32_16x16x32_f16 v[116:119], v[140:143], v[164:167], v[116:119]\n"
      "v_mfma_f32_16x16x32_f16 v[116:119], v[156:159], v[180:183], v[116:119]\n"
      "ds_read_b128 v[164:167], v251 offset:1024\n"
      "ds_read_b128 v[180:183], v251 offset:9216\n"
      "v_mfma_f32_16x16x32_f16 v[120:123], v[140:143], v[168:171], v[120:123]\n"
      "v_mfma_f32_16x16x32_f16 v[120:123], v[156:159], v[184:187], v[120:123]\n"
      "ds_read_b128 v[168:171], v251 offset:2048\n"
      "ds_read_b128 v[184:187], v251 offset:10240\n"
      "v_mfma_f32_16x16x32_f16 v[124:127], v[140:143], v[172:175], v[124:127]\n"
      "v_mfma_f32_16x16x32_f16 v[124:127], v[156:159], v[188:191], v[124:127]\n"
      "ds_read_b128 v[172:175], v251 offset:3072\n"
      "ds_read_b128 v[188:191], v251 offset:11264\n"
      "buffer_load_dwordx4 v[140:143], v248, s[20:23], s42 offen offset:3072 sc0 sc1\n"
      "buffer_load_dwordx4 v[156:159], v248, s[20:23], s43 offen offset:3072 sc0 sc1\n"
      "s_cmp_lt_u32 s28, 4\n"
      "s_setprio 0\n"
      "s_cbranch_scc1 .Lpr46_%=\n"
      "s_setprio 1\n"
      ".Lpr46_%=:\n"
      "s_add_u32 s60, s39, 0\n"
      "buffer_store_dwordx4 v[64:67], v236, s[24:27], s60 offen sc1\n"
      "s_add_u32 s60, s39, 65536\n"
      "buffer_store_dwordx4 v[68:71], v236, s[24:27], s60 offen sc1\n"
      "s_add_u32 s60, s39, 131072\n"
      "buffer_store_dwordx4 v[72:75], v236, s[24:27], s60 offen sc1\n"
      "s_add_u32 s60, s39, 196608\n"
      "buffer_store_dwordx4 v[76:79], v236, s[24:27], s60 offen sc1\n"
      "s_add_u32 s42, s42, 131072\n"
      "s_add_u32 s43, s43, 131072\n"
      "s_waitcnt vmcnt(14)\n"
      "s_waitcnt lgkmcnt(6)\n"
      "v_mfma_f32_16x16x32_f16 v[0:3], v[128:131], v[160:163], v[0:3]\n"
      "v_mfma_f32_16x16x32_f16 v[0:3], v[144:147], v[176:179], v[0:3]\n"
      "s_waitcnt lgkmcnt(4)\n"
      "v_mfma_f32_16x16x32_f16 v[4:7], v[128:131], v[164:167], v[4:7]\n"
      "v_mfma_f32_16x16x32_f16 v[4:7], v[144:147], v[180:183], v[4:7]\n"
      "s_waitcnt lgkmcnt(2)\n"
      "v_mfma_f32_16x16x32_f16 v[8:11], v[128:131], v[168:171], v[8:11]\n"
      "v_mfma_f32_16x16x32_f16 v[8:11], v[144:147], v[184:187], v[8:11]\n"
      "s_waitcnt lgkmcnt(0)\n"
      "v_mfma_f32_16x16x32_f16 v[12:15], v[128:131], v[172:175], v[12:15]\n"
      "v_mfma_f32_16x16x32_f16 v[12:15], v[144:147], v[188:191], v[12:15]\n"
      "s_add_u32 s60, s39, 262144\n"
      "buffer_store_dwordx4 v[80:83], v236, s[24:27], s60 offen sc1\n"
      "s_add_u32 s60, s39, 327680\n"
      "buffer_store_dwordx4 v[84:87], v236, s[24:27], s60 offen sc1\n"
      "s_add_u32 s60, s39, 393216\n"
      "buffer_store_dwordx4 v[88:91], v236, s[24:27], s60 offen sc1\n"
      "s_add_u32 s60, s39, 458752\n"
      "buffer_store_dwordx4 v[92:95], v236, s[24:27], s60 offen sc1\n"
      "s_waitcnt vmcnt(16)\n"
      "v_mfma_f32_16x16x32_f16 v[32:35], v[132:135], v[160:163], v[32:35]\n"
      "v_mfma_f32_16x16x32_f16 v[32:35], v[148:151], v[176:179], v[32:35]\n"
      "ds_read_b128 v[160:163], v251 offset:4096\n"
      "ds_read_b128 v[176:179], v251 offset:12288\n"
      "v_mfma_f32_16x16x32_f16 v[36:39], v[132:135], v[164:167], v[36:39]\n"
      "v_mfma_f32_16x16x32_f16 v[36:39], v[148:151], v[180:183], v[36:39]\n"
      "ds_read_b128 v[164:167], v251 offset:5120\n"
      "ds_read_b128 v[180:183], v251 offset:13312\n"
      "v_mfma_f32_16x16x32_f16 v[40:43], v[132:135], v[168:171], v[40:43]\n"
      "v_mfma_f32_16x16x32_f16 v[40:43], v[148:151], v[184:187], v[40:43]\n"
      "ds_read_b128 v[168:171], v251 offset:6144\n"
      "ds_read_b128 v[184:187], v251 offset:14336\n"
      "v_mfma_f32_16x16x32_f16 v[44:47], v[132:135], v[172:175], v[44:47]\n"
      "v_mfma_f32_16x16x32_f16 v[44:47], v[148:151], v[188:191], v[44:47]\n"
      "ds_read_b128 v[172:175], v251 offset:7168\n"
      "ds_read_b128 v[188:191], v251 offset:15360\n"
      "s_add_u32 s60, s39, 524288\n"
      "buffer_store_dwordx4 v[96:99], v236, s[24:27], s60 offen sc1\n"
      "s_add_u32 s60, s39, 589824\n"
      "buffer_store_dwordx4 v[100:103], v236, s[24:27], s60 offen sc1\n"
      "s_add_u32 s60, s39, 655360\n"
      "buffer_store_dwordx4 v[104:107], v236, s[24:27], s60 offen sc1\n"
      "s_add_u32 s60, s39, 720896\n"
      "buffer_store_dwordx4 v[108:111], v236, s[24:27], s60 offen sc1\n"
      "s_waitcnt lgkmcnt(6)\n"
      "v_mfma_f32_16x16x32_f16 v[16:19], v[128:131], v[160:163], v[16:19]\n"
      "v_mfma_f32_16x16x32_f16 v[16:19], v[144:147], v[176:179], v[16:19]\n"
      "s_waitcnt lgkmcnt(4)\n"
      "v_mfma_f32_16x16x32_f16 v[20:23], v[128:131], v[164:167], v[20:23]\n"
      "v_mfma_f32_16x16x32_f16 v[20:23], v[144:147], v[180:183], v[20:23]\n"
      "s_waitcnt lgkmcnt(2)\n"
      "v_mfma_f32_16x16x32_f16 v[24:27], v[128:131], v[168:171], v[24:27]\n"
      "v_mfma_f32_16x16x32_f16 v[24:27], v[144:147], v[184:187], v[24:27]\n"
      "s_waitcnt lgkmcnt(0)\n"
      "v_mfma_f32_16x16x32_f16 v[28:31], v[128:131], v[172:175], v[28:31]\n"
      "v_mfma_f32_16x16x32_f16 v[28:31], v[144:147], v[188:191], v[28:31]\n"
      "buffer_load_dwordx4 v[128:131], v248, s[20:23], s42 offen offset:0 sc0 sc1\n"
      "buffer_load_dwordx4 v[144:147], v248, s[20:23], s43 offen offset:0 sc0 sc1\n"
      "s_add_u32 s60, s39, 786432\n"
      "buffer_store_dwordx4 v[112:115], v236, s[24:27], s60 offen sc1\n"
      "s_add_u32 s60, s39, 851968\n"
      "buffer_store_dwordx4 v[116:119], v236, s[24:27], s60 offen sc1\n"
      "s_add_u32 s60, s39, 917504\n"
      "buffer_store_dwordx4 v[120:123], v236, s[24:27], s60 offen sc1\n"
      "s_add_u32 s60, s39, 983040\n"
      "buffer_store_dwordx4 v[124:127], v236, s[24:27], s60 offen sc1\n"
      "v_mfma_f32_16x16x32_f16 v[48:51], v[132:135], v[160:163], v[48:51]\n"
      "v_mfma_f32_16x16x32_f16 v[48:51], v[148:151], v[176:179], v[48:51]\n"
      "ds_read_b128 v[160:163], v251 offset:16384\n"
      "ds_read_b128 v[176:179], v251 offset:24576\n"
      "v_mfma_f32_16x16x32_f16 v[52:55], v[132:135], v[164:167], v[52:55]\n"
      "v_mfma_f32_16x16x32_f16 v[52:55], v[148:151], v[180:183], v[52:55]\n"
      "ds_read_b128 v[164:167], v251 offset:17408\n"
      "ds_read_b128 v[180:183], v251 offset:25600\n"
      "v_mfma_f32_16x16x32_f16 v[56:59], v[132:135], v[168:171], v[56:59]\n"
      "v_mfma_f32_16x16x32_f16 v[56:59], v[148:151], v[184:187], v[56:59]\n"
      "ds_read_b128 v[168:171], v251 offset:18432\n"
      "ds_read_b128 v[184:187], v251 offset:26624\n"
      "v_mfma_f32_16x16x32_f16 v[60:63], v[132:135], v[172:175], v[60:63]\n"
      "v_mfma_f32_16x16x32_f16 v[60:63], v[148:151], v[188:191], v[60:63]\n"
      "ds_read_b128 v[172:175], v251 offset:19456\n"
      "ds_read_b128 v[188:191], v251 offset:27648\n"
      "buffer_load_dwordx4 v[132:135], v248, s[20:23], s42 offen offset:1024 sc0 sc1\n"
      "buffer_load_dwordx4 v[148:151], v248, s[20:23], s43 offen offset:1024 sc0 sc1\n"
      "s_waitcnt vmcnt(22)\n"
      "s_waitcnt lgkmcnt(6)\n"
      "v_mfma_f32_16x16x32_f16 v[0:3], v[136:139], v[160:163], v[0:3]\n"
      "v_mfma_f32_16x16x32_f16 v[0:3], v[152:155], v[176:179], v[0:3]\n"
      "s_waitcnt lgkmcnt(4)\n"
      "v_mfma_f32_16x16x32_f16 v[4:7], v[136:139], v[164:167], v[4:7]\n"
      "v_mfma_f32_16x16x32_f16 v[4:7], v[152:155], v[180:183], v[4:7]\n"
      "s_waitcnt lgkmcnt(2)\n"
      "v_mfma_f32_16x16x32_f16 v[8:11], v[136:139], v[168:171], v[8:11]\n"
      "v_mfma_f32_16x16x32_f16 v[8:11], v[152:155], v[184:187], v[8:11]\n"
      "s_waitcnt lgkmcnt(0)\n"
      "v_mfma_f32_16x16x32_f16 v[12:15], v[136:139], v[172:175], v[12:15]\n"
      "v_mfma_f32_16x16x32_f16 v[12:15], v[152:155], v[188:191], v[12:15]\n"
      "s_waitcnt vmcnt(20)\n"
      "v_mfma_f32_16x16x32_f16 v[32:35], v[140:143], v[160:163], v[32:35]\n"
      "v_mfma_f32_16x16x32_f16 v[32:35], v[156:159], v[176:179], v[32:35]\n"
      "ds_read_b128 v[160:163], v251 offset:20480\n"
      "ds_read_b128 v[176:179], v251 offset:28672\n"
      "v_mfma_f32_16x16x32_f16 v[36:39], v[140:143], v[164:167], v[36:39]\n"
      "v_mfma_f32_16x16x32_f16 v[36:39], v[156:159], v[180:183], v[36:39]\n"
      "ds_read_b128 v[164:167], v251 offset:21504\n"
      "ds_read_b128 v[180:183], v251 offset:29696\n"
      "v_mfma_f32_16x16x32_f16 v[40:43], v[140:143], v[168:171], v[40:43]\n"
      "v_mfma_f32_16x16x32_f16 v[40:43], v[156:159], v[184:187], v[40:43]\n"
      "ds_read_b128 v[168:171], v251 offset:22528\n"
      "ds_read_b128 v[184:187], v251 offset:30720\n"
      "v_mfma_f32_16x16x32_f16 v[44:47], v[140:143], v[172:175], v[44:47]\n"
      "v_mfma_f32_16x16x32_f16 v[44:47], v[156:159], v[188:191], v[44:47]\n"
      "ds_read_b128 v[172:175], v251 offset:23552\n"
      "ds_read_b128 v[188:191], v251 offset:31744\n"
      "s_waitcnt lgkmcnt(6)\n"
      "v_mfma_f32_16x16x32_f16 v[16:19], v[136:139], v[160:163], v[16:19]\n"
      "v_mfma_f32_16x16x32_f16 v[16:19], v[152:155], v[176:179], v[16:19]\n"
      "s_waitcnt lgkmcnt(4)\n"
      "v_mfma_f32_16x16x32_f16 v[20:23], v[136:139], v[164:167], v[20:23]\n"
      "v_mfma_f32_16x16x32_f16 v[20:23], v[152:155], v[180:183], v[20:23]\n"
      "s_waitcnt lgkmcnt(2)\n"
      "v_mfma_f32_16x16x32_f16 v[24:27], v[136:139], v[168:171], v[24:27]\n"
      "v_mfma_f32_16x16x32_f16 v[24:27], v[152:155], v[184:187], v[24:27]\n"
      "s_waitcnt lgkmcnt(0)\n"
      "v_mfma_f32_16x16x32_f16 v[28:31], v[136:139], v[172:175], v[28:31]\n"
      "v_mfma_f32_16x16x32_f16 v[28:31], v[152:155], v[188:191], v[28:31]\n"
      "buffer_load_dwordx4 v[136:139], v248, s[20:23], s42 offen offset:2048 sc0 sc1\n"
      "buffer_load_dwordx4 v[152:155], v248, s[20:23], s43 offen offset:2048 sc0 sc1\n"
      "v_add_u32 v251, 0x8000, v251\n"
      "v_and_b32 v251, 0x1ffff, v251\n"
      "v_mfma_f32_16x16x32_f16 v[48:51], v[140:143], v[160:163], v[48:51]\n"
      "v_mfma_f32_16x16x32_f16 v[48:51], v[156:159], v[176:179], v[48:51]\n"
      "ds_read_b128 v[160:163], v251 offset:0\n"
      "ds_read_b128 v[176:179], v251 offset:8192\n"
      "v_mfma_f32_16x16x32_f16 v[52:55], v[140:143], v[164:167], v[52:55]\n"
      "v_mfma_f32_16x16x32_f16 v[52:55], v[156:159], v[180:183], v[52:55]\n"
      "ds_read_b128 v[164:167], v251 offset:1024\n"
      "ds_read_b128 v[180:183], v251 offset:9216\n"
      "v_mfma_f32_16x16x32_f16 v[56:59], v[140:143], v[168:171], v[56:59]\n"
      "v_mfma_f32_16x16x32_f16 v[56:59], v[156:159], v[184:187], v[56:59]\n"
      "ds_read_b128 v[168:171], v251 offset:2048\n"
      "ds_read_b128 v[184:187], v251 offset:10240\n"
      "v_mfma_f32_16x16x32_f16 v[60:63], v[140:143], v[172:175], v[60:63]\n"
      "v_mfma_f32_16x16x32_f16 v[60:63], v[156:159], v[188:191], v[60:63]\n"
      "ds_read_b128 v[172:175], v251 offset:3072\n"
      "ds_read_b128 v[188:191], v251 offset:11264\n"
      "buffer_load_dwordx4 v[140:143], v248, s[20:23], s42 offen offset:3072 sc0 sc1\n"
      "buffer_load_dwordx4 v[156:159], v248, s[20:23], s43 offen offset:3072 sc0 sc1\n"
      "s_add_u32 s42, s42, 65536\n"
      "s_add_u32 s43, s43, 65536\n"
      "s_waitcnt lgkmcnt(6)\n"
      "v_mfma_f32_16x16x32_f16 v[0:3], v[192:195], v[160:163], v[0:3]\n"
      "v_mfma_f32_16x16x32_f16 v[0:3], v[208:211], v[176:179], v[0:3]\n"
      "s_waitcnt lgkmcnt(4)\n"
      "v_mfma_f32_16x16x32_f16 v[4:7], v[192:195], v[164:167], v[4:7]\n"
      "v_mfma_f32_16x16x32_f16 v[4:7], v[208:211], v[180:183], v[4:7]\n"
      "s_waitcnt lgkmcnt(2)\n"
      "v_mfma_f32_16x16x32_f16 v[8:11], v[192:195], v[168:171], v[8:11]\n"
      "v_mfma_f32_16x16x32_f16 v[8:11], v[208:211], v[184:187], v[8:11]\n"
      "s_waitcnt lgkmcnt(0)\n"
      "v_mfma_f32_16x16x32_f16 v[12:15], v[192:195], v[172:175], v[12:15]\n"
      "v_mfma_f32_16x16x32_f16 v[12:15], v[208:211], v[188:191], v[12:15]\n"
      "v_mfma_f32_16x16x32_f16 v[32:35], v[196:199], v[160:163], v[32:35]\n"
      "v_mfma_f32_16x16x32_f16 v[32:35], v[212:215], v[176:179], v[32:35]\n"
      "ds_read_b128 v[160:163], v251 offset:4096\n"
      "ds_read_b128 v[176:179], v251 offset:12288\n"
      "v_mfma_f32_16x16x32_f16 v[36:39], v[196:199], v[164:167], v[36:39]\n"
      "v_mfma_f32_16x16x32_f16 v[36:39], v[212:215], v[180:183], v[36:39]\n"
      "ds_read_b128 v[164:167], v251 offset:5120\n"
      "ds_read_b128 v[180:183], v251 offset:13312\n"
      "v_mfma_f32_16x16x32_f16 v[40:43], v[196:199], v[168:171], v[40:43]\n"
      "v_mfma_f32_16x16x32_f16 v[40:43], v[212:215], v[184:187], v[40:43]\n"
      "ds_read_b128 v[168:171], v251 offset:6144\n"
      "ds_read_b128 v[184:187], v251 offset:14336\n"
      "v_mfma_f32_16x16x32_f16 v[44:47], v[196:199], v[172:175], v[44:47]\n"
      "v_mfma_f32_16x16x32_f16 v[44:47], v[212:215], v[188:191], v[44:47]\n"
      "ds_read_b128 v[172:175], v251 offset:7168\n"
      "ds_read_b128 v[188:191], v251 offset:15360\n"
      "s_waitcnt lgkmcnt(6)\n"
      "v_mfma_f32_16x16x32_f16 v[16:19], v[192:195], v[160:163], v[16:19]\n"
      "v_mfma_f32_16x16x32_f16 v[16:19], v[208:211], v[176:179], v[16:19]\n"
      "s_waitcnt lgkmcnt(4)\n"
      "v_mfma_f32_16x16x32_f16 v[20:23], v[192:195], v[164:167], v[20:23]\n"
      "v_mfma_f32_16x16x32_f16 v[20:23], v[208:211], v[180:183], v[20:23]\n"
      "s_waitcnt lgkmcnt(2)\n"
      "v_mfma_f32_16x16x32_f16 v[24:27], v[192:195], v[168:171], v[24:27]\n"
      "v_mfma_f32_16x16x32_f16 v[24:27], v[208:211], v[184:187], v[24:27]\n"
      "s_waitcnt lgkmcnt(0)\n"
      "v_mfma_f32_16x16x32_f16 v[28:31], v[192:195], v[172:175], v[28:31]\n"
      "v_mfma_f32_16x16x32_f16 v[28:31], v[208:211], v[188:191], v[28:31]\n"
      "buffer_load_dwordx4 v[192:195], v248, s[20:23], s42 offen offset:0 sc0 sc1\n"
      "buffer_load_dwordx4 v[208:211], v248, s[20:23], s43 offen offset:0 sc0 sc1\n"
      "v_mfma_f32_16x16x32_f16 v[48:51], v[196:199], v[160:163], v[48:51]\n"
      "v_mfma_f32_16x16x32_f16 v[48:51], v[212:215], v[176:179], v[48:51]\n"
      "ds_read_b128 v[160:163], v251 offset:16384\n"
      "ds_read_b128 v[176:179], v251 offset:24576\n"
      "v_mfma_f32_16x16x32_f16 v[52:55], v[196:199], v[164:167], v[52:55]\n"
      "v_mfma_f32_16x16x32_f16 v[52:55], v[212:215], v[180:183], v[52:55]\n"
      "ds_read_b128 v[164:167], v251 offset:17408\n"
      "ds_read_b128 v[180:183], v251 offset:25600\n"
      "v_mfma_f32_16x16x32_f16 v[56:59], v[196:199], v[168:171], v[56:59]\n"
      "v_mfma_f32_16x16x32_f16 v[56:59], v[212:215], v[184:187], v[56:59]\n"
      "ds_read_b128 v[168:171], v251 offset:18432\n"
      "ds_read_b128 v[184:187], v251 offset:26624\n"
      "v_mfma_f32_16x16x32_f16 v[60:63], v[196:199], v[172:175], v[60:63]\n"
      "v_mfma_f32_16x16x32_f16 v[60:63], v[212:215], v[188:191], v[60:63]\n"
      "ds_read_b128 v[172:175], v251 offset:19456\n"
      "ds_read_b128 v[188:191], v251 offset:27648\n"
      "buffer_load_dwordx4 v[196:199], v248, s[20:23], s42 offen offset:1024 sc0 sc1\n"
      "buffer_load_dwordx4 v[212:215], v248, s[20:23], s43 offen offset:1024 sc0 sc1\n"
      "s_waitcnt lgkmcnt(6)\n"
      "v_mfma_f32_16x16x32_f16 v[0:3], v[200:203], v[160:163], v[0:3]\n"
      "v_mfma_f32_16x16x32_f16 v[0:3], v[216:219], v[176:179], v[0:3]\n"
      "s_waitcnt lgkmcnt(4)\n"
      "v_mfma_f32_16x16x32_f16 v[4:7], v[200:203], v[164:167], v[4:7]\n"
      "v_mfma_f32_16x16x32_f16 v[4:7], v[216:219], v[180:183], v[4:7]\n"
      "s_waitcnt lgkmcnt(2)\n"
      "v_mfma_f32_16x16x32_f16 v[8:11], v[200:203], v[168:171], v[8:11]\n"
      "v_mfma_f32_16x16x32_f16 v[8:11], v[216:219], v[184:187], v[8:11]\n"
      "s_waitcnt lgkmcnt(0)\n"
      "v_mfma_f32_16x16x32_f16 v[12:15], v[200:203], v[172:175], v[12:15]\n"
      "v_mfma_f32_16x16x32_f16 v[12:15], v[216:219], v[188:191], v[12:15]\n"
      "v_mfma_f32_16x16x32_f16 v[32:35], v[204:207], v[160:163], v[32:35]\n"
      "v_mfma_f32_16x16x32_f16 v[32:35], v[220:223], v[176:179], v[32:35]\n"
      "ds_read_b128 v[160:163], v251 offset:20480\n"
      "ds_read_b128 v[176:179], v251 offset:28672\n"
      "v_mfma_f32_16x16x32_f16 v[36:39], v[204:207], v[164:167], v[36:39]\n"
      "v_mfma_f32_16x16x32_f16 v[36:39], v[220:223], v[180:183], v[36:39]\n"
      "ds_read_b128 v[164:167], v251 offset:21504\n"
      "ds_read_b128 v[180:183], v251 offset:29696\n"
      "v_mfma_f32_16x16x32_f16 v[40:43], v[204:207], v[168:171], v[40:43]\n"
      "v_mfma_f32_16x16x32_f16 v[40:43], v[220:223], v[184:187], v[40:43]\n"
      "ds_read_b128 v[168:171], v251 offset:22528\n"
      "ds_read_b128 v[184:187], v251 offset:30720\n"
      "v_mfma_f32_16x16x32_f16 v[44:47], v[204:207], v[172:175], v[44:47]\n"
      "v_mfma_f32_16x16x32_f16 v[44:47], v[220:223], v[188:191], v[44:47]\n"
      "ds_read_b128 v[172:175], v251 offset:23552\n"
      "ds_read_b128 v[188:191], v251 offset:31744\n"
      "s_waitcnt lgkmcnt(6)\n"
      "v_mfma_f32_16x16x32_f16 v[16:19], v[200:203], v[160:163], v[16:19]\n"
      "v_mfma_f32_16x16x32_f16 v[16:19], v[216:219], v[176:179], v[16:19]\n"
      "s_waitcnt lgkmcnt(4)\n"
      "v_mfma_f32_16x16x32_f16 v[20:23], v[200:203], v[164:167], v[20:23]\n"
      "v_mfma_f32_16x16x32_f16 v[20:23], v[216:219], v[180:183], v[20:23]\n"
      "s_waitcnt lgkmcnt(2)\n"
      "v_mfma_f32_16x16x32_f16 v[24:27], v[200:203], v[168:171], v[24:27]\n"
      "v_mfma_f32_16x16x32_f16 v[24:27], v[216:219], v[184:187], v[24:27]\n"
      "s_waitcnt lgkmcnt(0)\n"
      "v_mfma_f32_16x16x32_f16 v[28:31], v[200:203], v[172:175], v[28:31]\n"
      "v_mfma_f32_16x16x32_f16 v[28:31], v[216:219], v[188:191], v[28:31]\n"
      "buffer_load_dwordx4 v[200:203], v248, s[20:23], s42 offen offset:2048 sc0 sc1\n"
      "buffer_load_dwordx4 v[216:219], v248, s[20:23], s43 offen offset:2048 sc0 sc1\n"
      "v_add_u32 v251, 0x8000, v251\n"
      "v_and_b32 v251, 0x1ffff, v251\n"
      "v_mfma_f32_16x16x32_f16 v[48:51], v[204:207], v[160:163], v[48:51]\n"
      "v_mfma_f32_16x16x32_f16 v[48:51], v[220:223], v[176:179], v[48:51]\n"
      "ds_read_b128 v[160:163], v251 offset:0\n"
      "ds_read_b128 v[176:179], v251 offset:8192\n"
      "v_mfma_f32_16x16x32_f16 v[52:55], v[204:207], v[164:167], v[52:55]\n"
      "v_mfma_f32_16x16x32_f16 v[52:55], v[220:223], v[180:183], v[52:55]\n"
      "ds_read_b128 v[164:167], v251 offset:1024\n"
      "ds_read_b128 v[180:183], v251 offset:9216\n"
      "v_mfma_f32_16x16x32_f16 v[56:59], v[204:207], v[168:171], v[56:59]\n"
      "v_mfma_f32_16x16x32_f16 v[56:59], v[220:223], v[184:187], v[56:59]\n"
      "ds_read_b128 v[168:171], v251 offset:2048\n"
      "ds_read_b128 v[184:187], v251 offset:10240\n"
      "v_mfma_f32_16x16x32_f16 v[60:63], v[204:207], v[172:175], v[60:63]\n"
      "v_mfma_f32_16x16x32_f16 v[60:63], v[220:223], v[188:191], v[60:63]\n"
      "ds_read_b128 v[172:175], v251 offset:3072\n"
      "ds_read_b128 v[188:191], v251 offset:11264\n"
      "buffer_load_dwordx4 v[204:207], v248, s[20:23], s42 offen offset:3072 sc0 sc1\n"
      "buffer_load_dwordx4 v[220:223], v248, s[20:23], s43 offen offset:3072 sc0 sc1\n"
      "s_waitcnt vmcnt(18)\n"
      "s_waitcnt lgkmcnt(6)\n"
      "v_mfma_f32_16x16x32_f16 v[0:3], v[128:131], v[160:163], v[0:3]\n"
      "v_mfma_f32_16x16x32_f16 v[0:3], v[144:147], v[176:179], v[0:3]\n"
      "s_waitcnt lgkmcnt(4)\n"
      "v_mfma_f32_16x16x32_f16 v[4:7], v[128:131], v[164:167], v[4:7]\n"
      "v_mfma_f32_16x16x32_f16 v[4:7], v[144:147], v[180:183], v[4:7]\n"
      "s_waitcnt lgkmcnt(2)\n"
      "v_mfma_f32_16x16x32_f16 v[8:11], v[128:131], v[168:171], v[8:11]\n"
      "v_mfma_f32_16x16x32_f16 v[8:11], v[144:147], v[184:187], v[8:11]\n"
      "s_waitcnt lgkmcnt(0)\n"
      "v_mfma_f32_16x16x32_f16 v[12:15], v[128:131], v[172:175], v[12:15]\n"
      "v_mfma_f32_16x16x32_f16 v[12:15], v[144:147], v[188:191], v[12:15]\n"
      "s_waitcnt vmcnt(14)\n"
      "v_mov_b32 v253, s29\n"
      "s_mov_b64 exec, 1\n"
      "buffer_store_dword v253, off, s[16:19], s40 sc1\n"
      "s_mov_b64 exec, -1\n"
      "s_waitcnt vmcnt(13)\n"
      "v_mfma_f32_16x16x32_f16 v[32:35], v[132:135], v[160:163], v[32:35]\n"
      "v_mfma_f32_16x16x32_f16 v[32:35], v[148:151], v[176:179], v[32:35]\n"
      "ds_read_b128 v[160:163], v251 offset:4096\n"
      "ds_read_b128 v[176:179], v251 offset:12288\n"
      "v_mfma_f32_16x16x32_f16 v[36:39], v[132:135], v[164:167], v[36:39]\n"
      "v_mfma_f32_16x16x32_f16 v[36:39], v[148:151], v[180:183], v[36:39]\n"
      "ds_read_b128 v[164:167], v251 offset:5120\n"
      "ds_read_b128 v[180:183], v251 offset:13312\n"
      "v_mfma_f32_16x16x32_f16 v[40:43], v[132:135], v[168:171], v[40:43]\n"
      "v_mfma_f32_16x16x32_f16 v[40:43], v[148:151], v[184:187], v[40:43]\n"
      "ds_read_b128 v[168:171], v251 offset:6144\n"
      "ds_read_b128 v[184:187], v251 offset:14336\n"
      "v_mfma_f32_16x16x32_f16 v[44:47], v[132:135], v[172:175], v[44:47]\n"
      "v_mfma_f32_16x16x32_f16 v[44:47], v[148:151], v[188:191], v[44:47]\n"
      "ds_read_b128 v[172:175], v251 offset:7168\n"
      "ds_read_b128 v[188:191], v251 offset:15360\n"
      "s_waitcnt lgkmcnt(6)\n"
      "v_mfma_f32_16x16x32_f16 v[16:19], v[128:131], v[160:163], v[16:19]\n"
      "v_mfma_f32_16x16x32_f16 v[16:19], v[144:147], v[176:179], v[16:19]\n"
      "s_waitcnt lgkmcnt(4)\n"
      "v_mfma_f32_16x16x32_f16 v[20:23], v[128:131], v[164:167], v[20:23]\n"
      "v_mfma_f32_16x16x32_f16 v[20:23], v[144:147], v[180:183], v[20:23]\n"
      "s_waitcnt lgkmcnt(2)\n"
      "v_mfma_f32_16x16x32_f16 v[24:27], v[128:131], v[168:171], v[24:27]\n"
      "v_mfma_f32_16x16x32_f16 v[24:27], v[144:147], v[184:187], v[24:27]\n"
      "s_waitcnt lgkmcnt(0)\n"
      "v_mfma_f32_16x16x32_f16 v[28:31], v[128:131], v[172:175], v[28:31]\n"
      "v_mfma_f32_16x16x32_f16 v[28:31], v[144:147], v[188:191], v[28:31]\n"
      "v_mfma_f32_16x16x32_f16 v[48:51], v[132:135], v[160:163], v[48:51]\n"
      "v_mfma_f32_16x16x32_f16 v[48:51], v[148:151], v[176:179], v[48:51]\n"
      "ds_read_b128 v[160:163], v251 offset:16384\n"
      "ds_read_b128 v[176:179], v251 offset:24576\n"
      "v_mfma_f32_16x16x32_f16 v[52:55], v[132:135], v[164:167], v[52:55]\n"
      "v_mfma_f32_16x16x32_f16 v[52:55], v[148:151], v[180:183], v[52:55]\n"
      "ds_read_b128 v[164:167], v251 offset:17408\n"
      "ds_read_b128 v[180:183], v251 offset:25600\n"
      "v_mfma_f32_16x16x32_f16 v[56:59], v[132:135], v[168:171], v[56:59]\n"
      "v_mfma_f32_16x16x32_f16 v[56:59], v[148:151], v[184:187], v[56:59]\n"
      "ds_read_b128 v[168:171], v251 offset:18432\n"
      "ds_read_b128 v[184:187], v251 offset:26624\n"
      "v_mfma_f32_16x16x32_f16 v[60:63], v[132:135], v[172:175], v[60:63]\n"
      "v_mfma_f32_16x16x32_f16 v[60:63], v[148:151], v[188:191], v[60:63]\n"
      "ds_read_b128 v[172:175], v251 offset:19456\n"
      "ds_read_b128 v[188:191], v251 offset:27648\n"
      "s_waitcnt vmcnt(11)\n"
      "s_waitcnt lgkmcnt(6)\n"
      "v_mfma_f32_16x16x32_f16 v[0:3], v[136:139], v[160:163], v[0:3]\n"
      "v_mfma_f32_16x16x32_f16 v[0:3], v[152:155], v[176:179], v[0:3]\n"
      "s_waitcnt lgkmcnt(4)\n"
      "v_mfma_f32_16x16x32_f16 v[4:7], v[136:139], v[164:167], v[4:7]\n"
      "v_mfma_f32_16x16x32_f16 v[4:7], v[152:155], v[180:183], v[4:7]\n"
      "s_waitcnt lgkmcnt(2)\n"
      "v_mfma_f32_16x16x32_f16 v[8:11], v[136:139], v[168:171], v[8:11]\n"
      "v_mfma_f32_16x16x32_f16 v[8:11], v[152:155], v[184:187], v[8:11]\n"
      "s_waitcnt lgkmcnt(0)\n"
      "v_mfma_f32_16x16x32_f16 v[12:15], v[136:139], v[172:175], v[12:15]\n"
      "v_mfma_f32_16x16x32_f16 v[12:15], v[152:155], v[188:191], v[12:15]\n"
      "buffer_load_dword v253, off, s[16:19], s51 sc1\n"
      "s_waitcnt vmcnt(10)\n"
      "v_mfma_f32_16x16x32_f16 v[32:35], v[140:143], v[160:163], v[32:35]\n"
      "v_mfma_f32_16x16x32_f16 v[32:35], v[156:159], v[176:179], v[32:35]\n"
      "ds_read_b128 v[160:163], v251 offset:20480\n"
      "ds_read_b128 v[176:179], v251 offset:28672\n"
      "v_mfma_f32_16x16x32_f16 v[36:39], v[140:143], v[164:167], v[36:39]\n"
      "v_mfma_f32_16x16x32_f16 v[36:39], v[156:159], v[180:183], v[36:39]\n"
      "ds_read_b128 v[164:167], v251 offset:21504\n"
      "ds_read_b128 v[180:183], v251 offset:29696\n"
      "v_mfma_f32_16x16x32_f16 v[40:43], v[140:143], v[168:171], v[40:43]\n"
      "v_mfma_f32_16x16x32_f16 v[40:43], v[156:159], v[184:187], v[40:43]\n"
      "ds_read_b128 v[168:171], v251 offset:22528\n"
      "ds_read_b128 v[184:187], v251 offset:30720\n"
      "v_mfma_f32_16x16x32_f16 v[44:47], v[140:143], v[172:175], v[44:47]\n"
      "v_mfma_f32_16x16x32_f16 v[44:47], v[156:159], v[188:191], v[44:47]\n"
      "ds_read_b128 v[172:175], v251 offset:23552\n"
      "ds_read_b128 v[188:191], v251 offset:31744\n"
      "s_waitcnt lgkmcnt(6)\n"
      "v_mfma_f32_16x16x32_f16 v[16:19], v[136:139], v[160:163], v[16:19]\n"
      "v_mfma_f32_16x16x32_f16 v[16:19], v[152:155], v[176:179], v[16:19]\n"
      "s_waitcnt lgkmcnt(4)\n"
      "v_mfma_f32_16x16x32_f16 v[20:23], v[136:139], v[164:167], v[20:23]\n"
      "v_mfma_f32_16x16x32_f16 v[20:23], v[152:155], v[180:183], v[20:23]\n"
      "s_waitcnt lgkmcnt(2)\n"
      "v_mfma_f32_16x16x32_f16 v[24:27], v[136:139], v[168:171], v[24:27]\n"
      "v_mfma_f32_16x16x32_f16 v[24:27], v[152:155], v[184:187], v[24:27]\n"
      "s_waitcnt lgkmcnt(0)\n"
      "v_mfma_f32_16x16x32_f16 v[28:31], v[136:139], v[172:175], v[28:31]\n"
      "v_mfma_f32_16x16x32_f16 v[28:31], v[152:155], v[188:191], v[28:31]\n"
      "v_add_u32 v251, 0x8000, v251\n"
      "v_and_b32 v251, 0x1ffff, v251\n"
      "v_mfma_f32_16x16x32_f16 v[48:51], v[140:143], v[160:163], v[48:51]\n"
      "v_mfma_f32_16x16x32_f16 v[48:51], v[156:159], v[176:179], v[48:51]\n"
      "ds_read_b128 v[160:163], v251 offset:0\n"
      "ds_read_b128 v[176:179], v251 offset:8192\n"
      "v_mfma_f32_16x16x32_f16 v[52:55], v[140:143], v[164:167], v[52:55]\n"
      "v_mfma_f32_16x16x32_f16 v[52:55], v[156:159], v[180:183], v[52:55]\n"
      "ds_read_b128 v[164:167], v251 offset:1024\n"
      "ds_read_b128 v[180:183], v251 offset:9216\n"
      "v_mfma_f32_16x16x32_f16 v[56:59], v[140:143], v[168:171], v[56:59]\n"
      "v_mfma_f32_16x16x32_f16 v[56:59], v[156:159], v[184:187], v[56:59]\n"
      "ds_read_b128 v[168:171], v251 offset:2048\n"
      "ds_read_b128 v[184:187], v251 offset:10240\n"
      "v_mfma_f32_16x16x32_f16 v[60:63], v[140:143], v[172:175], v[60:63]\n"
      "v_mfma_f32_16x16x32_f16 v[60:63], v[156:159], v[188:191], v[60:63]\n"
      "ds_read_b128 v[172:175], v251 offset:3072\n"
      "ds_read_b128 v[188:191], v251 offset:11264\n"
      "s_waitcnt vmcnt(0)\n"
      "v_readfirstlane_b32 s60, v253\n"
      "s_nop 0\n"
      "s_cmp_lt_u32 s60, s29\n"
      "s_cbranch_scc1 .Lxw47_%=\n"
      ".Lxwb48_%=:\n"
      "s_mov_b32 s61, 0x20000\n"
      "buffer_load_dword v237, off, s[16:19], s61 sc1\n"
      "s_add_u32 s60, s38, 0\n"
      "buffer_load_dwordx4 v[64:67], v236, s[24:27], s60 offen sc1\n"
      "s_add_u32 s60, s38, 65536\n"
      "buffer_load_dwordx4 v[68:71], v236, s[24:27], s60 offen sc1\n"
      "s_add_u32 s60, s38, 131072\n"
      "buffer_load_dwordx4 v[72:75], v236, s[24:27], s60 offen sc1\n"
      "s_add_u32 s60, s38, 196608\n"
      "buffer_load_dwordx4 v[76:79], v236, s[24:27], s60 offen sc1\n"
      "s_waitcnt lgkmcnt(6)\n"
      "v_mfma_f32_16x16x32_f16 v[0:3], v[192:195], v[160:163], v[0:3]\n"
      "v_mfma_f32_16x16x32_f16 v[0:3], v[208:211], v[176:179], v[0:3]\n"
      "s_waitcnt lgkmcnt(4)\n"
      "v_mfma_f32_16x16x32_f16 v[4:7], v[192:195], v[164:167], v[4:7]\n"
      "v_mfma_f32_16x16x32_f16 v[4:7], v[208:211], v[180:183], v[4:7]\n"
      "s_waitcnt lgkmcnt(2)\n"
      "v_mfma_f32_16x16x32_f16 v[8:11], v[192:195], v[168:171], v[8:11]\n"
      "v_mfma_f32_16x16x32_f16 v[8:11], v[208:211], v[184:187], v[8:11]\n"
      "s_waitcnt lgkmcnt(0)\n"
      "v_mfma_f32_16x16x32_f16 v[12:15], v[192:195], v[172:175], v[12:15]\n"
      "v_mfma_f32_16x16x32_f16 v[12:15], v[208:211], v[188:191], v[12:15]\n"
      "s_add_u32 s60, s38, 262144\n"
      "buffer_load_dwordx4 v[80:83], v236, s[24:27], s60 offen sc1\n"
      "s_add_u32 s60, s38, 327680\n"
      "buffer_load_dwordx4 v[84:87], v236, s[24:27], s60 offen sc1\n"
      "s_add_u32 s60, s38, 393216\n"
      "buffer_load_dwordx4 v[88:91], v236, s[24:27], s60 offen sc1\n"
      "s_add_u32 s60, s38, 458752\n"
      "buffer_load_dwordx4 v[92:95], v236, s[24:27], s60 offen sc1\n"
      "v_mfma_f32_16x16x32_f16 v[32:35], v[196:199], v[160:163], v[32:35]\n"
      "v_mfma_f32_16x16x32_f16 v[32:35], v[212:215], v[176:179], v[32:35]\n"
      "ds_read_b128 v[160:163], v251 offset:4096\n"
      "ds_read_b128 v[176:179], v251 offset:12288\n"
      "v_mfma_f32_16x16x32_f16 v[36:39], v[196:199], v[164:167], v[36:39]\n"
      "v_mfma_f32_16x16x32_f16 v[36:39], v[212:215], v[180:183], v[36:39]\n"
      "ds_read_b128 v[164:167], v251 offset:5120\n"
      "ds_read_b128 v[180:183], v251 offset:13312\n"
      "v_mfma_f32_16x16x32_f16 v[40:43], v[196:199], v[168:171], v[40:43]\n"
      "v_mfma_f32_16x16x32_f16 v[40:43], v[212:215], v[184:187], v[40:43]\n"
      "ds_read_b128 v[168:171], v251 offset:6144\n"
      "ds_read_b128 v[184:187], v251 offset:14336\n"
      "v_mfma_f32_16x16x32_f16 v[44:47], v[196:199], v[172:175], v[44:47]\n"
      "v_mfma_f32_16x16x32_f16 v[44:47], v[212:215], v[188:191], v[44:47]\n"
      "ds_read_b128 v[172:175], v251 offset:7168\n"
      "ds_read_b128 v[188:191], v251 offset:15360\n"
      "s_add_u32 s60, s38, 524288\n"
      "buffer_load_dwordx4 v[96:99], v236, s[24:27], s60 offen sc1\n"
      "s_add_u32 s60, s38, 589824\n"
      "buffer_load_dwordx4 v[100:103], v236, s[24:27], s60 offen sc1\n"
      "s_add_u32 s60, s38, 655360\n"
      "buffer_load_dwordx4 v[104:107], v236, s[24:27], s60 offen sc1\n"
      "s_add_u32 s60, s38, 720896\n"
      "buffer_load_dwordx4 v[108:111], v236, s[24:27], s60 offen sc1\n"
      "s_waitcnt lgkmcnt(6)\n"
      "v_mfma_f32_16x16x32_f16 v[16:19], v[192:195], v[160:163], v[16:19]\n"
      "v_mfma_f32_16x16x32_f16 v[16:19], v[208:211], v[176:179], v[16:19]\n"
      "s_waitcnt lgkmcnt(4)\n"
      "v_mfma_f32_16x16x32_f16 v[20:23], v[192:195], v[164:167], v[20:23]\n"
      "v_mfma_f32_16x16x32_f16 v[20:23], v[208:211], v[180:183], v[20:23]\n"
      "s_waitcnt lgkmcnt(2)\n"
      "v_mfma_f32_16x16x32_f16 v[24:27], v[192:195], v[168:171], v[24:27]\n"
      "v_mfma_f32_16x16x32_f16 v[24:27], v[208:211], v[184:187], v[24:27]\n"
      "s_waitcnt lgkmcnt(0)\n"
      "v_mfma_f32_16x16x32_f16 v[28:31], v[192:195], v[172:175], v[28:31]\n"
      "v_mfma_f32_16x16x32_f16 v[28:31], v[208:211], v[188:191], v[28:31]\n"
      "s_add_u32 s60, s38, 786432\n"
      "buffer_load_dwordx4 v[112:115], v236, s[24:27], s60 offen sc1\n"
      "s_add_u32 s60, s38, 851968\n"
      "buffer_load_dwordx4 v[116:119], v236, s[24:27], s60 offen sc1\n"
      "s_add_u32 s60, s38, 917504\n"
      "buffer_load_dwordx4 v[120:123], v236, s[24:27], s60 offen sc1\n"
      "s_add_u32 s60, s38, 983040\n"
      "buffer_load_dwordx4 v[124:127], v236, s[24:27], s60 offen sc1\n"
      "v_mfma_f32_16x16x32_f16 v[48:51], v[196:199], v[160:163], v[48:51]\n"
      "v_mfma_f32_16x16x32_f16 v[48:51], v[212:215], v[176:179], v[48:51]\n"
      "ds_read_b128 v[160:163], v251 offset:16384\n"
      "ds_read_b128 v[176:179], v251 offset:24576\n"
      "v_mfma_f32_16x16x32_f16 v[52:55], v[196:199], v[164:167], v[52:55]\n"
      "v_mfma_f32_16x16x32_f16 v[52:55], v[212:215], v[180:183], v[52:55]\n"
      "ds_read_b128 v[164:167], v251 offset:17408\n"
      "ds_read_b128 v[180:183], v251 offset:25600\n"
      "v_mfma_f32_16x16x32_f16 v[56:59], v[196:199], v[168:171], v[56:59]\n"
      "v_mfma_f32_16x16x32_f16 v[56:59], v[212:215], v[184:187], v[56:59]\n"
      "ds_read_b128 v[168:171], v251 offset:18432\n"
      "ds_read_b128 v[184:187], v251 offset:26624\n"
      "v_mfma_f32_16x16x32_f16 v[60:63], v[196:199], v[172:175], v[60:63]\n"
      "v_mfma_f32_16x16x32_f16 v[60:63], v[212:215], v[188:191], v[60:63]\n"
      "ds_read_b128 v[172:175], v251 offset:19456\n"
      "ds_read_b128 v[188:191], v251 offset:27648\n"
      "s_waitcnt lgkmcnt(6)\n"
      "v_mfma_f32_16x16x32_f16 v[0:3], v[200:203], v[160:163], v[0:3]\n"
      "v_mfma_f32_16x16x32_f16 v[0:3], v[216:219], v[176:179], v[0:3]\n"
      "s_waitcnt lgkmcnt(4)\n"
      "v_mfma_f32_16x16x32_f16 v[4:7], v[200:203], v[164:167], v[4:7]\n"
      "v_mfma_f32_16x16x32_f16 v[4:7], v[216:219], v[180:183], v[4:7]\n"
      "s_waitcnt lgkmcnt(2)\n"
      "v_mfma_f32_16x16x32_f16 v[8:11], v[200:203], v[168:171], v[8:11]\n"
      "v_mfma_f32_16x16x32_f16 v[8:11], v[216:219], v[184:187], v[8:11]\n"
      "s_waitcnt lgkmcnt(0)\n"
      "v_mfma_f32_16x16x32_f16 v[12:15], v[200:203], v[172:175], v[12:15]\n"
      "v_mfma_f32_16x16x32_f16 v[12:15], v[216:219], v[188:191], v[12:15]\n"
      "v_mfma_f32_16x16x32_f16 v[32:35], v[204:207], v[160:163], v[32:35]\n"
      "v_mfma_f32_16x16x32_f16 v[32:35], v[220:223], v[176:179], v[32:35]\n"
      "ds_read_b128 v[160:163], v251 offset:20480\n"
      "ds_read_b128 v[176:179], v251 offset:28672\n"
      "v_mfma_f32_16x16x32_f16 v[36:39], v[204:207], v[164:167], v[36:39]\n"
      "v_mfma_f32_16x16x32_f16 v[36:39], v[220:223], v[180:183], v[36:39]\n"
      "ds_read_b128 v[164:167], v251 offset:21504\n"
      "ds_read_b128 v[180:183], v251 offset:29696\n"
      "v_mfma_f32_16x16x32_f16 v[40:43], v[204:207], v[168:171], v[40:43]\n"
      "v_mfma_f32_16x16x32_f16 v[40:43], v[220:223], v[184:187], v[40:43]\n"
      "ds_read_b128 v[168:171], v251 offset:22528\n"
      "ds_read_b128 v[184:187], v251 offset:30720\n"
      "v_mfma_f32_16x16x32_f16 v[44:47], v[204:207], v[172:175], v[44:47]\n"
      "v_mfma_f32_16x16x32_f16 v[44:47], v[220:223], v[188:191], v[44:47]\n"
      "ds_read_b128 v[172:175], v251 offset:23552\n"
      "ds_read_b128 v[188:191], v251 offset:31744\n"
      "s_waitcnt lgkmcnt(6)\n"
      "v_mfma_f32_16x16x32_f16 v[16:19], v[200:203], v[160:163], v[16:19]\n"
      "v_mfma_f32_16x16x32_f16 v[16:19], v[216:219], v[176:179], v[16:19]\n"
      "s_waitcnt lgkmcnt(4)\n"
      "v_mfma_f32_16x16x32_f16 v[20:23], v[200:203], v[164:167], v[20:23]\n"
      "v_mfma_f32_16x16x32_f16 v[20:23], v[216:219], v[180:183], v[20:23]\n"
      "s_waitcnt lgkmcnt(2)\n"
      "v_mfma_f32_16x16x32_f16 v[24:27], v[200:203], v[168:171], v[24:27]\n"
      "v_mfma_f32_16x16x32_f16 v[24:27], v[216:219], v[184:187], v[24:27]\n"
      "s_waitcnt lgkmcnt(0)\n"
      "v_mfma_f32_16x16x32_f16 v[28:31], v[200:203], v[172:175], v[28:31]\n"
      "v_mfma_f32_16x16x32_f16 v[28:31], v[216:219], v[188:191], v[28:31]\n"
      "v_mfma_f32_16x16x32_f16 v[48:51], v[204:207], v[160:163], v[48:51]\n"
      "v_mfma_f32_16x16x32_f16 v[48:51], v[220:223], v[176:179], v[48:51]\n"
      "v_mfma_f32_16x16x32_f16 v[52:55], v[204:207], v[164:167], v[52:55]\n"
      "v_mfma_f32_16x16x32_f16 v[52:55], v[220:223], v[180:183], v[52:55]\n"
      "v_mfma_f32_16x16x32_f16 v[56:59], v[204:207], v[168:171], v[56:59]\n"
      "v_mfma_f32_16x16x32_f16 v[56:59], v[220:223], v[184:187], v[56:59]\n"
      "v_mfma_f32_16x16x32_f16 v[60:63], v[204:207], v[172:175], v[60:63]\n"
      "v_mfma_f32_16x16x32_f16 v[60:63], v[220:223], v[188:191], v[60:63]\n"
      "s_nop 7\n"
      "s_waitcnt vmcnt(0)\n"
      "v_readfirstlane_b32 s60, v237\n"
      "v_and_b32 v236, 0x300, v248\n"
      "v_and_b32 v237, 0xff, v248\n"
      "v_lshl_add_u32 v236, v236, 9, v237\n"
      "s_cmp_eq_u32 s60, s29\n"
      "s_cbranch_scc1 .Lnan52_%=\n"
      "s_add_u32 s60, s38, 0\n"
      "v_add_f32 v160, v0, v64\n"
      "v_add_f32 v161, v4, v68\n"
      "v_add_f32 v162, v8, v72\n"
      "v_add_f32 v163, v12, v76\n"
      "buffer_store_dwordx4 v[160:163], v236, s[24:27], s60 offen offset:0 sc1\n"
      "v_add_f32 v164, v16, v80\n"
      "v_add_f32 v165, v20, v84\n"
      "v_add_f32 v166, v24, v88\n"
      "v_add_f32 v167, v28, v92\n"
      "buffer_store_dwordx4 v[164:167], v236, s[24:27], s60 offen offset:256 sc1\n"
      "s_add_u32 s60, s38, 32768\n"
      "v_add_f32 v168, v1, v65\n"
      "v_add_f32 v169, v5, v69\n"
      "v_add_f32 v170, v9, v73\n"
      "v_add_f32 v171, v13, v77\n"
      "buffer_store_dwordx4 v[168:171], v236, s[24:27], s60 offen offset:0 sc1\n"
      "v_add_f32 v172, v17, v81\n"
      "v_add_f32 v173, v21, v85\n"
      "v_add_f32 v174, v25, v89\n"
      "v_add_f32 v175, v29, v93\n"
      "buffer_store_dwordx4 v[172:175], v236, s[24:27], s60 offen offset:256 sc1\n"
      "s_add_u32 s60, s38, 65536\n"
      "v_add_f32 v176, v2, v66\n"
      "v_add_f32 v177, v6, v70\n"
      "v_add_f32 v178, v10, v74\n"
      "v_add_f32 v179, v14, v78\n"
      "buffer_store_dwordx4 v[176:179], v236, s[24:27], s60 offen offset:0 sc1\n"
      "v_add_f32 v180, v18, v82\n"
      "v_add_f32 v181, v22, v86\n"
      "v_add_f32 v182, v26, v90\n"
      "v_add_f32 v183, v30, v94\n"
      "buffer_store_dwordx4 v[180:183], v236, s[24:27], s60 offen offset:256 sc1\n"
      "s_add_u32 s60, s38, 98304\n"
      "v_add_f32 v184, v3, v67\n"
      "v_add_f32 v185, v7, v71\n"
      "v_add_f32 v186, v11, v75\n"
      "v_add_f32 v187, v15, v79\n"
      "buffer_store_dwordx4 v[184:187], v236, s[24:27], s60 offen offset:0 sc1\n"
      "v_add_f32 v188, v19, v83\n"
      "v_add_f32 v189, v23, v87\n"
      "v_add_f32 v190, v27, v91\n"
      "v_add_f32 v191, v31, v95\n"
      "buffer_store_dwordx4 v[188:191], v236, s[24:27], s60 offen offset:256 sc1\n"
      "s_add_u32 s60, s38, 524288\n"
      "v_add_f32 v160, v32, v96\n"
      "v_add_f32 v161, v36, v100\n"
      "v_add_f32 v162, v40, v104\n"
      "v_add_f32 v163, v44, v108\n"
      "buffer_store_dwordx4 v[160:163], v236, s[24:27], s60 offen offset:0 sc1\n"
      "v_add_f32 v164, v48, v112\n"
      "v_add_f32 v165, v52, v116\n"
      "v_add_f32 v166, v56, v120\n"
      "v_add_f32 v167, v60, v124\n"
      "buffer_store_dwordx4 v[164:167], v236, s[24:27], s60 offen offset:256 sc1\n"
      "s_add_u32 s60, s38, 557056\n"
      "v_add_f32 v168, v33, v97\n"
      "v_add_f32 v169, v37, v101\n"
      "v_add_f32 v170, v41, v105\n"
      "v_add_f32 v171, v45, v109\n"
      "buffer_store_dwordx4 v[168:171], v236, s[24:27], s60 offen offset:0 sc1\n"
      "v_add_f32 v172, v49, v113\n"
      "v_add_f32 v173, v53, v117\n"
      "v_add_f32 v174, v57, v121\n"
      "v_add_f32 v175, v61, v125\n"
      "buffer_store_dwordx4 v[172:175], v236, s[24:27], s60 offen offset:256 sc1\n"
      "s_add_u32 s60, s38, 589824\n"
      "v_add_f32 v176, v34, v98\n"
      "v_add_f32 v177, v38, v102\n"
      "v_add_f32 v178, v42, v106\n"
      "v_add_f32 v179, v46, v110\n"
      "buffer_store_dwordx4 v[176:179], v236, s[24:27], s60 offen offset:0 sc1\n"
      "v_add_f32 v180, v50, v114\n"
      "v_add_f32 v181, v54, v118\n"
      "v_add_f32 v182, v58, v122\n"
      "v_add_f32 v183, v62, v126\n"
      "buffer_store_dwordx4 v[180:183], v236, s[24:27], s60 offen offset:256 sc1\n"
      "s_add_u32 s60, s38, 622592\n"
      "v_add_f32 v184, v35, v99\n"
      "v_add_f32 v185, v39, v103\n"
      "v_add_f32 v186, v43, v107\n"
      "v_add_f32 v187, v47, v111\n"
      "buffer_store_dwordx4 v[184:187], v236, s[24:27], s60 offen offset:0 sc1\n"
      "v_add_f32 v188, v51, v115\n"
      "v_add_f32 v189, v55, v119\n"
      "v_add_f32 v190, v59, v123\n"
      "v_add_f32 v191, v63, v127\n"
      "buffer_store_dwordx4 v[188:191], v236, s[24:27], s60 offen offset:256 sc1\n"
      "s_branch .Lend_%=\n"
      ".Lev_%=:\n"
      "s_cmp_ge_u32 s47, 8\n"
      "s_cbranch_scc1 .Ldbl_%=\n"
      "s_waitcnt lgkmcnt(0)\n"
      "ds_read_b128 v[200:203], v251 offset:1024\n"
      "ds_read_b128 v[204:207], v251 offset:9216\n"
      "s_waitcnt vmcnt(10)\n"
      "v_mfma_f32_16x16x32_f16 v[0:3], v[128:131], v[192:195], v[0:3]\n"
      "s_setprio 1\n"
      "v_mfma_f32_16x16x32_f16 v[0:3], v[144:147], v[196:199], v[0:3]\n"
      "s_waitcnt vmcnt(8)\n"
      "v_mfma_f32_16x16x32_f16 v[32:35], v[132:135], v[192:195], v[32:35]\n"
      "v_mfma_f32_16x16x32_f16 v[32:35], v[148:151], v[196:199], v[32:35]\n"
      "s_waitcnt vmcnt(6)\n"
      "v_mfma_f32_16x16x32_f16 v[64:67], v[136:139], v[192:195], v[64:67]\n"
      "v_mfma_f32_16x16x32_f16 v[64:67], v[152:155], v[196:199], v[64:67]\n"
      "s_waitcnt vmcnt(4)\n"
      "v_mfma_f32_16x16x32_f16 v[96:99], v[140:143], v[192:195], v[96:99]\n"
      "v_mfma_f32_16x16x32_f16 v[96:99], v[156:159], v[196:199], v[96:99]\n"
      "s_add_u32 s42, s42, 65536\n"
      "s_add_u32 s43, s43, 65536\n"
      "buffer_load_dwordx4 v[160:163], v248, s[20:23], s42 offen offset:0 sc0 sc1\n"
      "buffer_load_dwordx4 v[176:179], v248, s[20:23], s43 offen offset:0 sc0 sc1\n"
      "buffer_load_dwordx4 v[164:167], v248, s[20:23], s42 offen offset:1024 sc0 sc1\n"
      "buffer_load_dwordx4 v[180:183], v248, s[20:23], s43 offen offset:1024 sc0 sc1\n"
      "s_waitcnt lgkmcnt(0)\n"
      "ds_read_b128 v[192:195], v251 offset:2048\n"
      "ds_read_b128 v[196:199], v251 offset:10240\n"
      "v_mfma_f32_16x16x32_f16 v[4:7], v[128:131], v[200:203], v[4:7]\n"
      "v_mfma_f32_16x16x32_f16 v[4:7], v[144:147], v[204:207], v[4:7]\n"
      "v_mfma_f32_16x16x32_f16 v[36:39], v[132:135], v[200:203], v[36:39]\n"
      "v_mfma_f32_16x16x32_f16 v[36:39], v[148:151], v[204:207], v[36:39]\n"
      "v_mfma_f32_16x16x32_f16 v[68:71], v[136:139], v[200:203], v[68:71]\n"
      "v_mfma_f32_16x16x32_f16 v[68:71], v[152:155], v[204:207], v[68:71]\n"
      "v_mfma_f32_16x16x32_f16 v[100:103], v[140:143], v[200:203], v[100:103]\n"
      "v_mfma_f32_16x16x32_f16 v[100:103], v[156:159], v[204:207], v[100:103]\n"
      "s_setprio 0\n"
      "buffer_load_dwordx4 v[168:171], v248, s[20:23], s42 offen offset:2048 sc0 sc1\n"
      "buffer_load_dwordx4 v[184:187], v248, s[20:23], s43 offen offset:2048 sc0 sc1\n"
      "s_waitcnt lgkmcnt(0)\n"
      "ds_read_b128 v[200:203], v251 offset:3072\n"
      "ds_read_b128 v[204:207], v251 offset:11264\n"
      "v_mfma_f32_16x16x32_f16 v[8:11], v[128:131], v[192:195], v[8:11]\n"
      "s_setprio 1\n"
      "v_mfma_f32_16x16x32_f16 v[8:11], v[144:147], v[196:199], v[8:11]\n"
      "v_mfma_f32_16x16x32_f16 v[40:43], v[132:135], v[192:195], v[40:43]\n"
      "v_mfma_f32_16x16x32_f16 v[40:43], v[148:151], v[196:199], v[40:43]\n"
      "v_mfma_f32_16x16x32_f16 v[72:75], v[136:139], v[192:195], v[72:75]\n"
      "v_mfma_f32_16x16x32_f16 v[72:75], v[152:155], v[196:199], v[72:75]\n"
      "v_mfma_f32_16x16x32_f16 v[104:107], v[140:143], v[192:195], v[104:107]\n"
      "v_mfma_f32_16x16x32_f16 v[104:107], v[156:159], v[196:199], v[104:107]\n"
      "buffer_load_dwordx4 v[172:175], v248, s[20:23], s42 offen offset:3072 sc0 sc1\n"
      "buffer_load_dwordx4 v[188:191], v248, s[20:23], s43 offen offset:3072 sc0 sc1\n"
      "s_waitcnt vmcnt(8)\n"
      "v_cvt_pk_f16_f32 v240, v208, v212\n"
      "v_cvt_pk_f16_f32 v241, v216, v220\n"
      "ds_write_b64 v252, v[240:241] offset:8\n"
      "v_cvt_pk_f16_f32 v242, v209, v213\n"
      "v_cvt_pk_f16_f32 v243, v217, v221\n"
      "ds_write_b64 v252, v[242:243] offset:1032\n"
      "v_cvt_pk_f16_f32 v244, v210, v214\n"
      "v_cvt_pk_f16_f32 v245, v218, v222\n"
      "ds_write_b64 v252, v[244:245] offset:2056\n"
      "v_cvt_pk_f16_f32 v246, v211, v215\n"
      "v_cvt_pk_f16_f32 v247, v219, v223\n"
      "ds_write_b64 v252, v[246:247] offset:3080\n"
      "v_add_u32 v252, 0x8000, v252\n"
      "v_and_b32 v252, 0x1ffff, v252\n"
      "v_cvt_pk_f16_f32 v232, v224, v225\n"
      "v_cvt_pk_f16_f32 v233, v226, v227\n"
      "v_cvt_pk_f16_f32 v234, v228, v229\n"
      "v_cvt_pk_f16_f32 v235, v230, v231\n"
      "ds_write2_b64 v237, v[232:233], v[234:235] offset1:16\n"
      "ds_read_b128 v[232:235], v238\n"
      "v_readlane_b32 s62, v254, s47\n"
      "s_add_u32 s60, s47, 8\n"
      "v_readlane_b32 s63, v254, s60\n"
      "s_add_u32 s60, s47, 24\n"
      "v_readlane_b32 s61, v254, s60\n"
      "s_add_u32 s60, s47, 17\n"
      "v_readlane_b32 s64, v254, s60\n"
      "s_waitcnt lgkmcnt(6)\n"
      "ds_read_b128 v[192:195], v251 offset:4096\n"
      "ds_read_b128 v[196:199], v251 offset:12288\n"
      "v_mfma_f32_16x16x32_f16 v[12:15], v[128:131], v[200:203], v[12:15]\n"
      "v_mfma_f32_16x16x32_f16 v[12:15], v[144:147], v[204:207], v[12:15]\n"
      "v_mfma_f32_16x16x32_f16 v[44:47], v[132:135], v[200:203], v[44:47]\n"
      "v_mfma_f32_16x16x32_f16 v[44:47], v[148:151], v[204:207], v[44:47]\n"
      "v_mfma_f32_16x16x32_f16 v[76:79], v[136:139], v[200:203], v[76:79]\n"
      "v_mfma_f32_16x16x32_f16 v[76:79], v[152:155], v[204:207], v[76:79]\n"
      "v_mfma_f32_16x16x32_f16 v[108:111], v[140:143], v[200:203], v[108:111]\n"
      "v_mfma_f32_16x16x32_f16 v[108:111], v[156:159], v[204:207], v[108:111]\n"
      "s_setprio 0\n"
      "s_add_u32 s61, s61, s30\n"
      "s_add_u32 s62, s62, s61\n"
      "s_addc_u32 s63, s63, 0\n"
      "s_cmp_lt_u32 s47, 7\n"
      "s_cselect_b32 s64, s64, 0x80000000\n"
      "s_add_u32 s65, s64, 0x8000\n"
      "s_waitcnt lgkmcnt(2)\n"
      "global_store_dwordx4 v248, v[232:235], s[62:63] sc0 sc1\n"
      "buffer_load_dwordx4 v[224:227], v236, s[12:15], s64 offen nt\n"
      "buffer_load_dwordx4 v[228:231], v236, s[12:15], s65 offen nt\n"
      "s_cmp_eq_u32 s28, 4\n"
      "s_cbranch_scc1 .Lso19_%=\n"
      ".Lsob20_%=:\n"
      "s_and_b32 s45, s44, -2\n"
      "s_add_u32 s46, s45, 65536\n"
      "buffer_load_dwordx4 v[208:211], v249, s[8:11], s45 offen\n"
      "buffer_load_dwordx4 v[212:215], v250, s[8:11], s45 offen\n"
      "buffer_load_dwordx4 v[216:219], v249, s[8:11], s46 offen\n"
      "buffer_load_dwordx4 v[220:223], v250, s[8:11], s46 offen\n"
      "s_waitcnt lgkmcnt(0)\n"
      "ds_read_b128 v[200:203], v251 offset:5120\n"
      "ds_read_b128 v[204:207], v251 offset:13312\n"
      "v_mfma_f32_16x16x32_f16 v[16:19], v[128:131], v[192:195], v[16:19]\n"
      "s_setprio 1\n"
      "v_mfma_f32_16x16x32_f16 v[16:19], v[144:147], v[196:199], v[16:19]\n"
      "v_mfma_f32_16x16x32_f16 v[48:51], v[132:135], v[192:195], v[48:51]\n"
      "v_mfma_f32_16x16x32_f16 v[48:51], v[148:151], v[196:199], v[48:51]\n"
      "v_mfma_f32_16x16x32_f16 v[80:83], v[136:139], v[192:195], v[80:83]\n"
      "v_mfma_f32_16x16x32_f16 v[80:83], v[152:155], v[196:199], v[80:83]\n"
      "v_mfma_f32_16x16x32_f16 v[112:115], v[140:143], v[192:195], v[112:115]\n"
      "v_mfma_f32_16x16x32_f16 v[112:115], v[156:159], v[196:199], v[112:115]\n"
      "s_waitcnt lgkmcnt(0)\n"
      "ds_read_b128 v[192:195], v251 offset:6144\n"
      "ds_read_b128 v[196:199], v251 offset:14336\n"
      "v_mfma_f32_16x16x32_f16 v[20:23], v[128:131], v[200:203], v[20:23]\n"
      "v_mfma_f32_16x16x32_f16 v[20:23], v[144:147], v[204:207], v[20:23]\n"
      "v_mfma_f32_16x16x32_f16 v[52:55], v[132:135], v[200:203], v[52:55]\n"
      "v_mfma_f32_16x16x32_f16 v[52:55], v[148:151], v[204:207], v[52:55]\n"
      "v_mfma_f32_16x16x32_f16 v[84:87], v[136:139], v[200:203], v[84:87]\n"
      "v_mfma_f32_16x16x32_f16 v[84:87], v[152:155], v[204:207], v[84:87]\n"
      "v_mfma_f32_16x16x32_f16 v[116:119], v[140:143], v[200:203], v[116:119]\n"
      "v_mfma_f32_16x16x32_f16 v[116:119], v[156:159], v[204:207], v[116:119]\n"
      "s_setprio 0\n"
      "s_waitcnt lgkmcnt(0)\n"
      "ds_read_b128 v[200:203], v251 offset:7168\n"
      "ds_read_b128 v[204:207], v251 offset:15360\n"
      "v_mfma_f32_16x16x32_f16 v[24:27], v[128:131], v[192:195], v[24:27]\n"
      "s_setprio 1\n"
      "v_mfma_f32_16x16x32_f16 v[24:27], v[144:147], v[196:199], v[24:27]\n"
      "v_mfma_f32_16x16x32_f16 v[56:59], v[132:135], v[192:195], v[56:59]\n"
      "v_mfma_f32_16x16x32_f16 v[56:59], v[148:151], v[196:199], v[56:59]\n"
      "v_mfma_f32_16x16x32_f16 v[88:91], v[136:139], v[192:195], v[88:91]\n"
      "v_mfma_f32_16x16x32_f16 v[88:91], v[152:155], v[196:199], v[88:91]\n"
      "v_mfma_f32_16x16x32_f16 v[120:123], v[140:143], v[192:195], v[120:123]\n"
      "v_mfma_f32_16x16x32_f16 v[120:123], v[156:159], v[196:199], v[120:123]\n"
      "s_waitcnt lgkmcnt(0)\n"
      "ds_read_b128 v[192:195], v251 offset:16384\n"
      "ds_read_b128 v[196:199], v251 offset:24576\n"
      "v_mfma_f32_16x16x32_f16 v[28:31], v[128:131], v[200:203], v[28:31]\n"
      "v_mfma_f32_16x16x32_f16 v[28:31], v[144:147], v[204:207], v[28:31]\n"
      "v_mfma_f32_16x16x32_f16 v[60:63], v[132:135], v[200:203], v[60:63]\n"
      "v_mfma_f32_16x16x32_f16 v[60:63], v[148:151], v[204:207], v[60:63]\n"
      "v_mfma_f32_16x16x32_f16 v[92:95], v[136:139], v[200:203], v[92:95]\n"
      "v_mfma_f32_16x16x32_f16 v[92:95], v[152:155], v[204:207], v[92:95]\n"
      "v_mfma_f32_16x16x32_f16 v[124:127], v[140:143], v[200:203], v[124:127]\n"
      "v_mfma_f32_16x16x32_f16 v[124:127], v[156:159], v[204:207], v[124:127]\n"
      "s_setprio 0\n"
      "s_waitcnt lgkmcnt(0)\n"
      "ds_read_b128 v[200:203], v251 offset:17408\n"
      "ds_read_b128 v[204:207], v251 offset:25600\n"
      "s_waitcnt vmcnt(13)\n"
      "v_mfma_f32_16x16x32_f16 v[0:3], v[160:163], v[192:195], v[0:3]\n"
      "s_setprio 1\n"
      "v_mfma_f32_16x16x32_f16 v[0:3], v[176:179], v[196:199], v[0:3]\n"
      "s_waitcnt vmcnt(11)\n"
      "v_mfma_f32_16x16x32_f16 v[32:35], v[164:167], v[192:195], v[32:35]\n"
      "v_mfma_f32_16x16x32_f16 v[32:35], v[180:183], v[196:199], v[32:35]\n"
      "s_waitcnt vmcnt(9)\n"
      "v_mfma_f32_16x16x32_f16 v[64:67], v[168:171], v[192:195], v[64:67]\n"
      "v_mfma_f32_16x16x32_f16 v[64:67], v[184:187], v[196:199], v[64:67]\n"
      "s_waitcnt vmcnt(7)\n"
      "v_mfma_f32_16x16x32_f16 v[96:99], v[172:175], v[192:195], v[96:99]\n"
      "v_mfma_f32_16x16x32_f16 v[96:99], v[188:191], v[196:199], v[96:99]\n"
      "s_waitcnt lgkmcnt(0)\n"
      "ds_read_b128 v[192:195], v251 offset:18432\n"
      "ds_read_b128 v[196:199], v251 offset:26624\n"
      "v_mfma_f32_16x16x32_f16 v[4:7], v[160:163], v[200:203], v[4:7]\n"
      "v_mfma_f32_16x16x32_f16 v[4:7], v[176:179], v[204:207], v[4:7]\n"
      "v_mfma_f32_16x16x32_f16 v[36:39], v[164:167], v[200:203], v[36:39]\n"
      "v_mfma_f32_16x16x32_f16 v[36:39], v[180:183], v[204:207], v[36:39]\n"
      "v_mfma_f32_16x16x32_f16 v[68:71], v[168:171], v[200:203], v[68:71]\n"
      "v_mfma_f32_16x16x32_f16 v[68:71], v[184:187], v[204:207], v[68:71]\n"
      "v_mfma_f32_16x16x32_f16 v[100:103], v[172:175], v[200:203], v[100:103]\n"
      "v_mfma_f32_16x16x32_f16 v[100:103], v[188:191], v[204:207], v[100:103]\n"
      "s_setprio 0\n"
      "s_waitcnt vmcnt(6)\n"
      "s_cmp_eq_u32 s28, 4\n"
      "s_cbranch_scc1 .Lsvc22_%=\n"
      ".Lsvcb23_%=:\n"
      "s_barrier\n"
      "s_sub_u32 s60, s41, 0\n"
      "s_cmp_lt_u32 s60, 10\n"
      "s_cbranch_scc1 .Lth28_%=\n"
      ".Lthb29_%=:\n"
      "s_add_u32 s42, s42, 65536\n"
      "s_add_u32 s43, s43, 65536\n"
      "buffer_load_dwordx4 v[128:131], v248, s[20:23], s42 offen offset:0 sc0 sc1\n"
      "buffer_load_dwordx4 v[144:147], v248, s[20:23], s43 offen offset:0 sc0 sc1\n"
      "s_waitcnt lgkmcnt(0)\n"
      "ds_read_b128 v[200:203], v251 offset:19456\n"
      "ds_read_b128 v[204:207], v251 offset:27648\n"
      "v_mfma_f32_16x16x32_f16 v[8:11], v[160:163], v[192:195], v[8:11]\n"
      "s_setprio 1\n"
      "v_mfma_f32_16x16x32_f16 v[8:11], v[176:179], v[196:199], v[8:11]\n"
      "v_mfma_f32_16x16x32_f16 v[40:43], v[164:167], v[192:195], v[40:43]\n"
      "v_mfma_f32_16x16x32_f16 v[40:43], v[180:183], v[196:199], v[40:43]\n"
      "v_mfma_f32_16x16x32_f16 v[72:75], v[168:171], v[192:195], v[72:75]\n"
      "v_mfma_f32_16x16x32_f16 v[72:75], v[184:187], v[196:199], v[72:75]\n"
      "v_mfma_f32_16x16x32_f16 v[104:107], v[172:175], v[192:195], v[104:107]\n"
      "v_mfma_f32_16x16x32_f16 v[104:107], v[188:191], v[196:199], v[104:107]\n"
      "buffer_load_dwordx4 v[132:135], v248, s[20:23], s42 offen offset:1024 sc0 sc1\n"
      "buffer_load_dwordx4 v[148:151], v248, s[20:23], s43 offen offset:1024 sc0 sc1\n"
      "s_waitcnt lgkmcnt(0)\n"
      "ds_read_b128 v[192:195], v251 offset:20480\n"
      "ds_read_b128 v[196:199], v251 offset:28672\n"
      "v_mfma_f32_16x16x32_f16 v[12:15], v[160:163], v[200:203], v[12:15]\n"
      "v_mfma_f32_16x16x32_f16 v[12:15], v[176:179], v[204:207], v[12:15]\n"
      "v_mfma_f32_16x16x32_f16 v[44:47], v[164:167], v[200:203], v[44:47]\n"
      "v_mfma_f32_16x16x32_f16 v[44:47], v[180:183], v[204:207], v[44:47]\n"
      "v_mfma_f32_16x16x32_f16 v[76:79], v[168:171], v[200:203], v[76:79]\n"
      "v_mfma_f32_16x16x32_f16 v[76:79], v[184:187], v[204:207], v[76:79]\n"
      "v_mfma_f32_16x16x32_f16 v[108:111], v[172:175], v[200:203], v[108:111]\n"
      "v_mfma_f32_16x16x32_f16 v[108:111], v[188:191], v[204:207], v[108:111]\n"
      "s_setprio 0\n"
      "buffer_load_dwordx4 v[136:139], v248, s[20:23], s42 offen offset:2048 sc0 sc1\n"
      "buffer_load_dwordx4 v[152:155], v248, s[20:23], s43 offen offset:2048 sc0 sc1\n"
      "s_waitcnt lgkmcnt(0)\n"
      "ds_read_b128 v[200:203], v251 offset:21504\n"
      "ds_read_b128 v[204:207], v251 offset:29696\n"
      "v_mfma_f32_16x16x32_f16 v[16:19], v[160:163], v[192:195], v[16:19]\n"
      "s_setprio 1\n"
      "v_mfma_f32_16x16x32_f16 v[16:19], v[176:179], v[196:199], v[16:19]\n"
      "v_mfma_f32_16x16x32_f16 v[48:51], v[164:167], v[192:195], v[48:51]\n"
      "v_mfma_f32_16x16x32_f16 v[48:51], v[180:183], v[196:199], v[48:51]\n"
      "v_mfma_f32_16x16x32_f16 v[80:83], v[168:171], v[192:195], v[80:83]\n"
      "v_mfma_f32_16x16x32_f16 v[80:83], v[184:187], v[196:199], v[80:83]\n"
      "v_mfma_f32_16x16x32_f16 v[112:115], v[172:175], v[192:195], v[112:115]\n"
      "v_mfma_f32_16x16x32_f16 v[112:115], v[188:191], v[196:199], v[112:115]\n"
      "buffer_load_dwordx4 v[140:143], v248, s[20:23], s42 offen offset:3072 sc0 sc1\n"
      "buffer_load_dwordx4 v[156:159], v248, s[20:23], s43 offen offset:3072 sc0 sc1\n"
      "s_waitcnt lgkmcnt(0)\n"
      "ds_read_b128 v[192:195], v251 offset:22528\n"
      "ds_read_b128 v[196:199], v251 offset:30720\n"
      "v_mfma_f32_16x16x32_f16 v[20:23], v[160:163], v[200:203], v[20:23]\n"
      "v_mfma_f32_16x16x32_f16 v[20:23], v[176:179], v[204:207], v[20:23]\n"
      "v_mfma_f32_16x16x32_f16 v[52:55], v[164:167], v[200:203], v[52:55]\n"
      "v_mfma_f32_16x16x32_f16 v[52:55], v[180:183], v[204:207], v[52:55]\n"
      "v_mfma_f32_16x16x32_f16 v[84:87], v[168:171], v[200:203], v[84:87]\n"
      "v_mfma_f32_16x16x32_f16 v[84:87], v[184:187], v[204:207], v[84:87]\n"
      "v_mfma_f32_16x16x32_f16 v[116:119], v[172:175], v[200:203], v[116:119]\n"
      "v_mfma_f32_16x16x32_f16 v[116:119], v[188:191], v[204:207], v[116:119]\n"
      "s_setprio 0\n"
      "s_waitcnt vmcnt(8)\n"
      "v_cvt_pk_f16_f32 v240, v208, v212\n"
      "v_cvt_pk_f16_f32 v241, v216, v220\n"
      "ds_write_b64 v252, v[240:241] offset:0\n"
      "v_cvt_pk_f16_f32 v242, v209, v213\n"
      "v_cvt_pk_f16_f32 v243, v217, v221\n"
      "ds_write_b64 v252, v[242:243] offset:1024\n"
      "v_cvt_pk_f16_f32 v244, v210, v214\n"
      "v_cvt_pk_f16_f32 v245, v218, v222\n"
      "ds_write_b64 v252, v[244:245] offset:2048\n"
      "v_cvt_pk_f16_f32 v246, v211, v215\n"
      "v_cvt_pk_f16_f32 v247, v219, v223\n"
      "ds_write_b64 v252, v[246:247] offset:3072\n"
      "s_and_b32 s45, s44, -2\n"
      "s_xor_b32 s45, s45, 524288\n"
      "s_add_u32 s46, s45, 65536\n"
      "buffer_load_dwordx4 v[208:211], v249, s[8:11], s45 offen\n"
      "buffer_load_dwordx4 v[212:215], v250, s[8:11], s45 offen\n"
      "buffer_load_dwordx4 v[216:219], v249, s[8:11], s46 offen\n"
      "buffer_load_dwordx4 v[220:223], v250, s[8:11], s46 offen\n"
      "s_waitcnt lgkmcnt(4)\n"
      "ds_read_b128 v[200:203], v251 offset:23552\n"
      "ds_read_b128 v[204:207], v251 offset:31744\n"
      "v_mfma_f32_16x16x32_f16 v[24:27], v[160:163], v[192:195], v[24:27]\n"
      "s_setprio 1\n"
      "v_mfma_f32_16x16x32_f16 v[24:27], v[176:179], v[196:199], v[24:27]\n"
      "v_mfma_f32_16x16x32_f16 v[56:59], v[164:167], v[192:195], v[56:59]\n"
      "v_mfma_f32_16x16x32_f16 v[56:59], v[180:183], v[196:199], v[56:59]\n"
      "v_mfma_f32_16x16x32_f16 v[88:91], v[168:171], v[192:195], v[88:91]\n"
      "v_mfma_f32_16x16x32_f16 v[88:91], v[184:187], v[196:199], v[88:91]\n"
      "v_mfma_f32_16x16x32_f16 v[120:123], v[172:175], v[192:195], v[120:123]\n"
      "v_mfma_f32_16x16x32_f16 v[120:123], v[188:191], v[196:199], v[120:123]\n"
      "s_waitcnt lgkmcnt(0)\n"
      "v_add_u32 v251, 0x8000, v251\n"
      "v_and_b32 v251, 0x1ffff, v251\n"
      "ds_read_b128 v[192:195], v251 offset:0\n"
      "ds_read_b128 v[196:199], v251 offset:8192\n"
      "v_mfma_f32_16x16x32_f16 v[28:31], v[160:163], v[200:203], v[28:31]\n"
      "v_mfma_f32_16x16x32_f16 v[28:31], v[176:179], v[204:207], v[28:31]\n"
      "v_mfma_f32_16x16x32_f16 v[60:63], v[164:167], v[200:203], v[60:63]\n"
      "v_mfma_f32_16x16x32_f16 v[60:63], v[180:183], v[204:207], v[60:63]\n"
      "v_mfma_f32_16x16x32_f16 v[92:95], v[168:171], v[200:203], v[92:95]\n"
      "v_mfma_f32_16x16x32_f16 v[92:95], v[184:187], v[204:207], v[92:95]\n"
      "v_mfma_f32_16x16x32_f16 v[124:127], v[172:175], v[200:203], v[124:127]\n"
      "v_mfma_f32_16x16x32_f16 v[124:127], v[188:191], v[204:207], v[124:127]\n"
      "s_setprio 0\n"
      "s_add_u32 s47, s47, 1\n"
      "s_add_u32 s31, s31, 3\n"
      "s_cmp_lt_u32 s47, 8\n"
      "s_cselect_b32 s31, s31, s56\n"
      "s_branch .Lnext_%=\n"
      ".Ldbl_%=:\n"
      "s_add_u32 s54, s54, 1\n"
      "v_readlane_b32 s55, v254, s54\n"
      "s_waitcnt lgkmcnt(0)\n"
      "ds_read_b128 v[200:203], v251 offset:1024\n"
      "ds_read_b128 v[204:207], v251 offset:9216\n"
      "s_waitcnt vmcnt(10)\n"
      "v_mfma_f32_16x16x32_f16 v[0:3], v[128:131], v[192:195], v[0:3]\n"
      "s_setprio 1\n"
      "v_mfma_f32_16x16x32_f16 v[0:3], v[144:147], v[196:199], v[0:3]\n"
      "s_waitcnt vmcnt(8)\n"
      "v_mfma_f32_16x16x32_f16 v[32:35], v[132:135], v[192:195], v[32:35]\n"
      "v_mfma_f32_16x16x32_f16 v[32:35], v[148:151], v[196:199], v[32:35]\n"
      "s_waitcnt vmcnt(6)\n"
      "v_mfma_f32_16x16x32_f16 v[64:67], v[136:139], v[192:195], v[64:67]\n"
      "v_mfma_f32_16x16x32_f16 v[64:67], v[152:155], v[196:199], v[64:67]\n"
      "s_waitcnt vmcnt(4)\n"
      "v_mfma_f32_16x16x32_f16 v[96:99], v[140:143], v[192:195], v[96:99]\n"
      "v_mfma_f32_16x16x32_f16 v[96:99], v[156:159], v[196:199], v[96:99]\n"
      "s_add_u32 s42, s42, 65536\n"
      "s_add_u32 s43, s43, 65536\n"
      "buffer_load_dwordx4 v[160:163], v248, s[20:23], s42 offen offset:0 sc0 sc1\n"
      "buffer_load_dwordx4 v[176:179], v248, s[20:23], s43 offen offset:0 sc0 sc1\n"
      "buffer_load_dwordx4 v[164:167], v248, s[20:23], s42 offen offset:1024 sc0 sc1\n"
      "buffer_load_dwordx4 v[180:183], v248, s[20:23], s43 offen offset:1024 sc0 sc1\n"
      "s_waitcnt lgkmcnt(0)\n"
      "ds_read_b128 v[192:195], v251 offset:2048\n"
      "ds_read_b128 v[196:199], v251 offset:10240\n"
      "v_mfma_f32_16x16x32_f16 v[4:7], v[128:131], v[200:203], v[4:7]\n"
      "v_mfma_f32_16x16x32_f16 v[4:7], v[144:147], v[204:207], v[4:7]\n"
      "v_mfma_f32_16x16x32_f16 v[36:39], v[132:135], v[200:203], v[36:39]\n"
      "v_mfma_f32_16x16x32_f16 v[36:39], v[148:151], v[204:207], v[36:39]\n"
      "v_mfma_f32_16x16x32_f16 v[68:71], v[136:139], v[200:203], v[68:71]\n"
      "v_mfma_f32_16x16x32_f16 v[68:71], v[152:155], v[204:207], v[68:71]\n"
      "v_mfma_f32_16x16x32_f16 v[100:103], v[140:143], v[200:203], v[100:103]\n"
      "v_mfma_f32_16x16x32_f16 v[100:103], v[156:159], v[204:207], v[100:103]\n"
      "s_setprio 0\n"
      "buffer_load_dwordx4 v[168:171], v248, s[20:23], s42 offen offset:2048 sc0 sc1\n"
      "buffer_load_dwordx4 v[184:187], v248, s[20:23], s43 offen offset:2048 sc0 sc1\n"
      "s_waitcnt lgkmcnt(0)\n"
      "ds_read_b128 v[200:203], v251 offset:3072\n"
      "ds_read_b128 v[204:207], v251 offset:11264\n"
      "v_mfma_f32_16x16x32_f16 v[8:11], v[128:131], v[192:195], v[8:11]\n"
      "s_setprio 1\n"
      "v_mfma_f32_16x16x32_f16 v[8:11], v[144:147], v[196:199], v[8:11]\n"
      "v_mfma_f32_16x16x32_f16 v[40:43], v[132:135], v[192:195], v[40:43]\n"
      "v_mfma_f32_16x16x32_f16 v[40:43], v[148:151], v[196:199], v[40:43]\n"
      "v_mfma_f32_16x16x32_f16 v[72:75], v[136:139], v[192:195], v[72:75]\n"
      "v_mfma_f32_16x16x32_f16 v[72:75], v[152:155], v[196:199], v[72:75]\n"
      "v_mfma_f32_16x16x32_f16 v[104:107], v[140:143], v[192:195], v[104:107]\n"
      "v_mfma_f32_16x16x32_f16 v[104:107], v[156:159], v[196:199], v[104:107]\n"
      "buffer_load_dwordx4 v[172:175], v248, s[20:23], s42 offen offset:3072 sc0 sc1\n"
      "buffer_load_dwordx4 v[188:191], v248, s[20:23], s43 offen offset:3072 sc0 sc1\n"
      "s_waitcnt vmcnt(8)\n"
      "v_cvt_pk_f16_f32 v240, v208, v212\n"
      "v_cvt_pk_f16_f32 v241, v216, v220\n"
      "ds_write_b64 v252, v[240:241] offset:8\n"
      "v_cvt_pk_f16_f32 v242, v209, v213\n"
      "v_cvt_pk_f16_f32 v243, v217, v221\n"
      "ds_write_b64 v252, v[242:243] offset:1032\n"
      "v_cvt_pk_f16_f32 v244, v210, v214\n"
      "v_cvt_pk_f16_f32 v245, v218, v222\n"
      "ds_write_b64 v252, v[244:245] offset:2056\n"
      "v_cvt_pk_f16_f32 v246, v211, v215\n"
      "v_cvt_pk_f16_f32 v247, v219, v223\n"
      "ds_write_b64 v252, v[246:247] offset:3080\n"
      "v_add_u32 v252, 0x8000, v252\n"
      "v_and_b32 v252, 0x1ffff, v252\n"
      "s_waitcnt lgkmcnt(4)\n"
      "ds_read_b128 v[192:195], v251 offset:4096\n"
      "ds_read_b128 v[196:199], v251 offset:12288\n"
      "v_mfma_f32_16x16x32_f16 v[12:15], v[128:131], v[200:203], v[12:15]\n"
      "v_mfma_f32_16x16x32_f16 v[12:15], v[144:147], v[204:207], v[12:15]\n"
      "v_mfma_f32_16x16x32_f16 v[44:47], v[132:135], v[200:203], v[44:47]\n"
      "v_mfma_f32_16x16x32_f16 v[44:47], v[148:151], v[204:207], v[44:47]\n"
      "v_mfma_f32_16x16x32_f16 v[76:79], v[136:139], v[200:203], v[76:79]\n"
      "v_mfma_f32_16x16x32_f16 v[76:79], v[152:155], v[204:207], v[76:79]\n"
      "v_mfma_f32_16x16x32_f16 v[108:111], v[140:143], v[200:203], v[108:111]\n"
      "v_mfma_f32_16x16x32_f16 v[108:111], v[156:159], v[204:207], v[108:111]\n"
      "s_setprio 0\n"
      "s_cmp_eq_u32 s28, 4\n"
      "s_cbranch_scc1 .Lso32_%=\n"
      ".Lsob33_%=:\n"
      "s_and_b32 s45, s44, -2\n"
      "s_add_u32 s46, s45, 65536\n"
      "buffer_load_dwordx4 v[208:211], v249, s[8:11], s45 offen\n"
      "buffer_load_dwordx4 v[212:215], v250, s[8:11], s45 offen\n"
      "buffer_load_dwordx4 v[216:219], v249, s[8:11], s46 offen\n"
      "buffer_load_dwordx4 v[220:223], v250, s[8:11], s46 offen\n"
      "s_and_b32 s45, s55, -2\n"
      "s_add_u32 s46, s45, 65536\n"
      "buffer_load_dwordx4 v[224:227], v249, s[8:11], s45 offen\n"
      "buffer_load_dwordx4 v[228:231], v250, s[8:11], s45 offen\n"
      "buffer_load_dwordx4 v[232:235], v249, s[8:11], s46 offen\n"
      "buffer_load_dwordx4 v[236:239], v250, s[8:11], s46 offen\n"
      "s_waitcnt lgkmcnt(0)\n"
      "ds_read_b128 v[200:203], v251 offset:5120\n"
      "ds_read_b128 v[204:207], v251 offset:13312\n"
      "v_mfma_f32_16x16x32_f16 v[16:19], v[128:131], v[192:195], v[16:19]\n"
      "s_setprio 1\n"
      "v_mfma_f32_16x16x32_f16 v[16:19], v[144:147], v[196:199], v[16:19]\n"
      "v_mfma_f32_16x16x32_f16 v[48:51], v[132:135], v[192:195], v[48:51]\n"
      "v_mfma_f32_16x16x32_f16 v[48:51], v[148:151], v[196:199], v[48:51]\n"
      "v_mfma_f32_16x16x32_f16 v[80:83], v[136:139], v[192:195], v[80:83]\n"
      "v_mfma_f32_16x16x32_f16 v[80:83], v[152:155], v[196:199], v[80:83]\n"
      "v_mfma_f32_16x16x32_f16 v[112:115], v[140:143], v[192:195], v[112:115]\n"
      "v_mfma_f32_16x16x32_f16 v[112:115], v[156:159], v[196:199], v[112:115]\n"
      "s_waitcnt lgkmcnt(0)\n"
      "ds_read_b128 v[192:195], v251 offset:6144\n"
      "ds_read_b128 v[196:199], v251 offset:14336\n"
      "v_mfma_f32_16x16x32_f16 v[20:23], v[128:131], v[200:203], v[20:23]\n"
      "v_mfma_f32_16x16x32_f16 v[20:23], v[144:147], v[204:207], v[20:23]\n"
      "v_mfma_f32_16x16x32_f16 v[52:55], v[132:135], v[200:203], v[52:55]\n"
      "v_mfma_f32_16x16x32_f16 v[52:55], v[148:151], v[204:207], v[52:55]\n"
      "v_mfma_f32_16x16x32_f16 v[84:87], v[136:139], v[200:203], v[84:87]\n"
      "v_mfma_f32_16x16x32_f16 v[84:87], v[152:155], v[204:207], v[84:87]\n"
      "v_mfma_f32_16x16x32_f16 v[116:119], v[140:143], v[200:203], v[116:119]\n"
      "v_mfma_f32_16x16x32_f16 v[116:119], v[156:159], v[204:207], v[116:119]\n"
      "s_setprio 0\n"
      "s_waitcnt lgkmcnt(0)\n"
      "ds_read_b128 v[200:203], v251 offset:7168\n"
      "ds_read_b128 v[204:207], v251 offset:15360\n"
      "v_mfma_f32_16x16x32_f16 v[24:27], v[128:131], v[192:195], v[24:27]\n"
      "s_setprio 1\n"
      "v_mfma_f32_16x16x32_f16 v[24:27], v[144:147], v[196:199], v[24:27]\n"
      "v_mfma_f32_16x16x32_f16 v[56:59], v[132:135], v[192:195], v[56:59]\n"
      "v_mfma_f32_16x16x32_f16 v[56:59], v[148:151], v[196:199], v[56:59]\n"
      "v_mfma_f32_16x16x32_f16 v[88:91], v[136:139], v[192:195], v[88:91]\n"
      "v_mfma_f32_16x16x32_f16 v[88:91], v[152:155], v[196:199], v[88:91]\n"
      "v_mfma_f32_16x16x32_f16 v[120:123], v[140:143], v[192:195], v[120:123]\n"
      "v_mfma_f32_16x16x32_f16 v[120:123], v[156:159], v[196:199], v[120:123]\n"
      "s_waitcnt lgkmcnt(0)\n"
      "ds_read_b128 v[192:195], v251 offset:16384\n"
      "ds_read_b128 v[196:199], v251 offset:24576\n"
      "v_mfma_f32_16x16x32_f16 v[28:31], v[128:131], v[200:203], v[28:31]\n"
      "v_mfma_f32_16x16x32_f16 v[28:31], v[144:147], v[204:207], v[28:31]\n"
      "v_mfma_f32_16x16x32_f16 v[60:63], v[132:135], v[200:203], v[60:63]\n"
      "v_mfma_f32_16x16x32_f16 v[60:63], v[148:151], v[204:207], v[60:63]\n"
      "v_mfma_f32_16x16x32_f16 v[92:95], v[136:139], v[200:203], v[92:95]\n"
      "v_mfma_f32_16x16x32_f16 v[92:95], v[152:155], v[204:207], v[92:95]\n"
      "v_mfma_f32_16x16x32_f16 v[124:127], v[140:143], v[200:203], v[124:127]\n"
      "v_mfma_f32_16x16x32_f16 v[124:127], v[156:159], v[204:207], v[124:127]\n"
      "s_setprio 0\n"
      "s_waitcnt lgkmcnt(0)\n"
      "ds_read_b128 v[200:203], v251 offset:17408\n"
      "ds_read_b128 v[204:207], v251 offset:25600\n"
      "s_waitcnt vmcnt(14)\n"
      "v_mfma_f32_16x16x32_f16 v[0:3], v[160:163], v[192:195], v[0:3]\n"
      "s_setprio 1\n"
      "v_mfma_f32_16x16x32_f16 v[0:3], v[176:179], v[196:199], v[0:3]\n"
      "s_waitcnt vmcnt(12)\n"
      "v_mfma_f32_16x16x32_f16 v[32:35], v[164:167], v[192:195], v[32:35]\n"
      "v_mfma_f32_16x16x32_f16 v[32:35], v[180:183], v[196:199], v[32:35]\n"
      "s_waitcnt vmcnt(10)\n"
      "v_mfma_f32_16x16x32_f16 v[64:67], v[168:171], v[192:195], v[64:67]\n"
      "v_mfma_f32_16x16x32_f16 v[64:67], v[184:187], v[196:199], v[64:67]\n"
      "s_waitcnt vmcnt(8)\n"
      "v_mfma_f32_16x16x32_f16 v[96:99], v[172:175], v[192:195], v[96:99]\n"
      "v_mfma_f32_16x16x32_f16 v[96:99], v[188:191], v[196:199], v[96:99]\n"
      "s_waitcnt lgkmcnt(0)\n"
      "ds_read_b128 v[192:195], v251 offset:18432\n"
      "ds_read_b128 v[196:199], v251 offset:26624\n"
      "v_mfma_f32_16x16x32_f16 v[4:7], v[160:163], v[200:203], v[4:7]\n"
      "v_mfma_f32_16x16x32_f16 v[4:7], v[176:179], v[204:207], v[4:7]\n"
      "v_mfma_f32_16x16x32_f16 v[36:39], v[164:167], v[200:203], v[36:39]\n"
      "v_mfma_f32_16x16x32_f16 v[36:39], v[180:183], v[204:207], v[36:39]\n"
      "v_mfma_f32_16x16x32_f16 v[68:71], v[168:171], v[200:203], v[68:71]\n"
      "v_mfma_f32_16x16x32_f16 v[68:71], v[184:187], v[204:207], v[68:71]\n"
      "v_mfma_f32_16x16x32_f16 v[100:103], v[172:175], v[200:203], v[100:103]\n"
      "v_mfma_f32_16x16x32_f16 v[100:103], v[188:191], v[204:207], v[100:103]\n"
      "s_setprio 0\n"
      "s_cmp_eq_u32 s28, 4\n"
      "s_cbranch_scc1 .Lsvc35_%=\n"
      ".Lsvcb36_%=:\n"
      "s_barrier\n"
      "s_sub_u32 s60, s41, 0\n"
      "s_cmp_lt_u32 s60, 10\n"
      "s_cbranch_scc1 .Lth41_%=\n"
      ".Lthb42_%=:\n"
      "s_add_u32 s42, s42, 65536\n"
      "s_add_u32 s43, s43, 65536\n"
      "buffer_load_dwordx4 v[128:131], v248, s[20:23], s42 offen offset:0 sc0 sc1\n"
      "buffer_load_dwordx4 v[144:147], v248, s[20:23], s43 offen offset:0 sc0 sc1\n"
      "s_waitcnt lgkmcnt(0)\n"
      "ds_read_b128 v[200:203], v251 offset:19456\n"
      "ds_read_b128 v[204:207], v251 offset:27648\n"
      "v_mfma_f32_16x16x32_f16 v[8:11], v[160:163], v[192:195], v[8:11]\n"
      "s_setprio 1\n"
      "v_mfma_f32_16x16x32_f16 v[8:11], v[176:179], v[196:199], v[8:11]\n"
      "v_mfma_f32_16x16x32_f16 v[40:43], v[164:167], v[192:195], v[40:43]\n"
      "v_mfma_f32_16x16x32_f16 v[40:43], v[180:183], v[196:199], v[40:43]\n"
      "v_mfma_f32_16x16x32_f16 v[72:75], v[168:171], v[192:195], v[72:75]\n"
      "v_mfma_f32_16x16x32_f16 v[72:75], v[184:187], v[196:199], v[72:75]\n"
      "v_mfma_f32_16x16x32_f16 v[104:107], v[172:175], v[192:195], v[104:107]\n"
      "v_mfma_f32_16x16x32_f16 v[104:107], v[188:191], v[196:199], v[104:107]\n"
      "buffer_load_dwordx4 v[132:135], v248, s[20:23], s42 offen offset:1024 sc0 sc1\n"
      "buffer_load_dwordx4 v[148:151], v248, s[20:23], s43 offen offset:1024 sc0 sc1\n"
      "s_waitcnt lgkmcnt(0)\n"
      "ds_read_b128 v[192:195], v251 offset:20480\n"
      "ds_read_b128 v[196:199], v251 offset:28672\n"
      "v_mfma_f32_16x16x32_f16 v[12:15], v[160:163], v[200:203], v[12:15]\n"
      "v_mfma_f32_16x16x32_f16 v[12:15], v[176:179], v[204:207], v[12:15]\n"
      "v_mfma_f32_16x16x32_f16 v[44:47], v[164:167], v[200:203], v[44:47]\n"
      "v_mfma_f32_16x16x32_f16 v[44:47], v[180:183], v[204:207], v[44:47]\n"
      "v_mfma_f32_16x16x32_f16 v[76:79], v[168:171], v[200:203], v[76:79]\n"
      "v_mfma_f32_16x16x32_f16 v[76:79], v[184:187], v[204:207], v[76:79]\n"
      "v_mfma_f32_16x16x32_f16 v[108:111], v[172:175], v[200:203], v[108:111]\n"
      "v_mfma_f32_16x16x32_f16 v[108:111], v[188:191], v[204:207], v[108:111]\n"
      "s_setprio 0\n"
      "buffer_load_dwordx4 v[136:139], v248, s[20:23], s42 offen offset:2048 sc0 sc1\n"
      "buffer_load_dwordx4 v[152:155], v248, s[20:23], s43 offen offset:2048 sc0 sc1\n"
      "s_waitcnt lgkmcnt(0)\n"
      "ds_read_b128 v[200:203], v251 offset:21504\n"
      "ds_read_b128 v[204:207], v251 offset:29696\n"
      "v_mfma_f32_16x16x32_f16 v[16:19], v[160:163], v[192:195], v[16:19]\n"
      "s_setprio 1\n"
      "v_mfma_f32_16x16x32_f16 v[16:19], v[176:179], v[196:199], v[16:19]\n"
      "v_mfma_f32_16x16x32_f16 v[48:51], v[164:167], v[192:195], v[48:51]\n"
      "v_mfma_f32_16x16x32_f16 v[48:51], v[180:183], v[196:199], v[48:51]\n"
      "v_mfma_f32_16x16x32_f16 v[80:83], v[168:171], v[192:195], v[80:83]\n"
      "v_mfma_f32_16x16x32_f16 v[80:83], v[184:187], v[196:199], v[80:83]\n"
      "v_mfma_f32_16x16x32_f16 v[112:115], v[172:175], v[192:195], v[112:115]\n"
      "v_mfma_f32_16x16x32_f16 v[112:115], v[188:191], v[196:199], v[112:115]\n"
      "buffer_load_dwordx4 v[140:143], v248, s[20:23], s42 offen offset:3072 sc0 sc1\n"
      "buffer_load_dwordx4 v[156:159], v248, s[20:23], s43 offen offset:3072 sc0 sc1\n"
      "s_waitcnt lgkmcnt(0)\n"
      "ds_read_b128 v[192:195], v251 offset:22528\n"
      "ds_read_b128 v[196:199], v251 offset:30720\n"
      "v_mfma_f32_16x16x32_f16 v[20:23], v[160:163], v[200:203], v[20:23]\n"
      "v_mfma_f32_16x16x32_f16 v[20:23], v[176:179], v[204:207], v[20:23]\n"
      "v_mfma_f32_16x16x32_f16 v[52:55], v[164:167], v[200:203], v[52:55]\n"
      "v_mfma_f32_16x16x32_f16 v[52:55], v[180:183], v[204:207], v[52:55]\n"
      "v_mfma_f32_16x16x32_f16 v[84:87], v[168:171], v[200:203], v[84:87]\n"
      "v_mfma_f32_16x16x32_f16 v[84:87], v[184:187], v[204:207], v[84:87]\n"
      "v_mfma_f32_16x16x32_f16 v[116:119], v[172:175], v[200:203], v[116:119]\n"
      "v_mfma_f32_16x16x32_f16 v[116:119], v[188:191], v[204:207], v[116:119]\n"
      "s_setprio 0\n"
      "s_waitcnt vmcnt(12)\n"
      "v_cvt_pk_f16_f32 v240, v208, v212\n"
      "v_cvt_pk_f16_f32 v241, v216, v220\n"
      "ds_write_b64 v252, v[240:241] offset:0\n"
      "v_cvt_pk_f16_f32 v242, v209, v213\n"
      "v_cvt_pk_f16_f32 v243, v217, v221\n"
      "ds_write_b64 v252, v[242:243] offset:1024\n"
      "v_cvt_pk_f16_f32 v244, v210, v214\n"
      "v_cvt_pk_f16_f32 v245, v218, v222\n"
      "ds_write_b64 v252, v[244:245] offset:2048\n"
      "v_cvt_pk_f16_f32 v246, v211, v215\n"
      "v_cvt_pk_f16_f32 v247, v219, v223\n"
      "ds_write_b64 v252, v[246:247] offset:3072\n"
      "v_add_u32 v253, 0x8000, v252\n"
      "v_and_b32 v253, 0x1ffff, v253\n"
      "s_waitcnt vmcnt(8)\n"
      "v_cvt_pk_f16_f32 v240, v224, v228\n"
      "v_cvt_pk_f16_f32 v241, v232, v236\n"
      "ds_write_b64 v253, v[240:241] offset:0\n"
      "v_cvt_pk_f16_f32 v242, v225, v229\n"
      "v_cvt_pk_f16_f32 v243, v233, v237\n"
      "ds_write_b64 v253, v[242:243] offset:1024\n"
      "v_cvt_pk_f16_f32 v244, v226, v230\n"
      "v_cvt_pk_f16_f32 v245, v234, v238\n"
      "ds_write_b64 v253, v[244:245] offset:2048\n"
      "v_cvt_pk_f16_f32 v246, v227, v231\n"
      "v_cvt_pk_f16_f32 v247, v235, v239\n"
      "ds_write_b64 v253, v[246:247] offset:3072\n"
      "s_and_b32 s45, s44, -2\n"
      "s_xor_b32 s45, s45, 524288\n"
      "s_add_u32 s46, s45, 65536\n"
      "buffer_load_dwordx4 v[224:227], v249, s[8:11], s45 offen\n"
      "buffer_load_dwordx4 v[228:231], v250, s[8:11], s45 offen\n"
      "buffer_load_dwordx4 v[232:235], v249, s[8:11], s46 offen\n"
      "buffer_load_dwordx4 v[236:239], v250, s[8:11], s46 offen\n"
      "s_and_b32 s45, s55, -2\n"
      "s_xor_b32 s45, s45, 524288\n"
      "s_add_u32 s46, s45, 65536\n"
      "buffer_load_dwordx4 v[208:211], v249, s[8:11], s45 offen\n"
      "buffer_load_dwordx4 v[212:215], v250, s[8:11], s45 offen\n"
      "buffer_load_dwordx4 v[216:219], v249, s[8:11], s46 offen\n"
      "buffer_load_dwordx4 v[220:223], v250, s[8:11], s46 offen\n"
      "s_waitcnt lgkmcnt(8)\n"
      "ds_read_b128 v[200:203], v251 offset:23552\n"
      "ds_read_b128 v[204:207], v251 offset:31744\n"
      "v_mfma_f32_16x16x32_f16 v[24:27], v[160:163], v[192:195], v[24:27]\n"
      "s_setprio 1\n"
      "v_mfma_f32_16x16x32_f16 v[24:27], v[176:179], v[196:199], v[24:27]\n"
      "v_mfma_f32_16x16x32_f16 v[56:59], v[164:167], v[192:195], v[56:59]\n"
      "v_mfma_f32_16x16x32_f16 v[56:59], v[180:183], v[196:199], v[56:59]\n"
      "v_mfma_f32_16x16x32_f16 v[88:91], v[168:171], v[192:195], v[88:91]\n"
      "v_mfma_f32_16x16x32_f16 v[88:91], v[184:187], v[196:199], v[88:91]\n"
      "v_mfma_f32_16x16x32_f16 v[120:123], v[172:175], v[192:195], v[120:123]\n"
      "v_mfma_f32_16x16x32_f16 v[120:123], v[188:191], v[196:199], v[120:123]\n"
      "s_waitcnt lgkmcnt(0)\n"
      "v_add_u32 v251, 0x8000, v251\n"
      "v_and_b32 v251, 0x1ffff, v251\n"
      "ds_read_b128 v[192:195], v251 offset:0\n"
      "ds_read_b128 v[196:199], v251 offset:8192\n"
      "v_mfma_f32_16x16x32_f16 v[28:31], v[160:163], v[200:203], v[28:31]\n"
      "v_mfma_f32_16x16x32_f16 v[28:31], v[176:179], v[204:207], v[28:31]\n"
      "v_mfma_f32_16x16x32_f16 v[60:63], v[164:167], v[200:203], v[60:63]\n"
      "v_mfma_f32_16x16x32_f16 v[60:63], v[180:183], v[204:207], v[60:63]\n"
      "v_mfma_f32_16x16x32_f16 v[92:95], v[168:171], v[200:203], v[92:95]\n"
      "v_mfma_f32_16x16x32_f16 v[92:95], v[184:187], v[204:207], v[92:95]\n"
      "v_mfma_f32_16x16x32_f16 v[124:127], v[172:175], v[200:203], v[124:127]\n"
      "v_mfma_f32_16x16x32_f16 v[124:127], v[188:191], v[204:207], v[124:127]\n"
      "s_setprio 0\n"
      "s_waitcnt vmcnt(4)\n"
      "v_cvt_pk_f16_f32 v240, v224, v228\n"
      "v_cvt_pk_f16_f32 v241, v232, v236\n"
      "ds_write_b64 v252, v[240:241] offset:8\n"
      "v_cvt_pk_f16_f32 v242, v225, v229\n"
      "v_cvt_pk_f16_f32 v243, v233, v237\n"
      "ds_write_b64 v252, v[242:243] offset:1032\n"
      "v_cvt_pk_f16_f32 v244, v226, v230\n"
      "v_cvt_pk_f16_f32 v245, v234, v238\n"
      "ds_write_b64 v252, v[244:245] offset:2056\n"
      "v_cvt_pk_f16_f32 v246, v227, v231\n"
      "v_cvt_pk_f16_f32 v247, v235, v239\n"
      "ds_write_b64 v252, v[246:247] offset:3080\n"
      "v_add_u32 v252, 0x8000, v252\n"
      "v_and_b32 v252, 0x1ffff, v252\n"
      "s_waitcnt lgkmcnt(0)\n"
      "s_mov_b32 s31, 0xffff\n"
      "s_branch .Lnext_%=\n"
      ".Le01_%=:\n"
      "v_cvt_pk_f16_f32 v232, v224, v225\n"
      "v_cvt_pk_f16_f32 v233, v226, v227\n"
      "v_cvt_pk_f16_f32 v234, v228, v229\n"
      "v_cvt_pk_f16_f32 v235, v230, v231\n"
      "ds_write2_b64 v237, v[232:233], v[234:235] offset1:16\n"
      "ds_read_b128 v[232:235], v238\n"
      "v_readlane_b32 s62, v254, s47\n"
      "s_add_u32 s60, s47, 8\n"
      "v_readlane_b32 s63, v254, s60\n"
      "s_add_u32 s60, s47, 24\n"
      "v_readlane_b32 s61, v254, s60\n"
      "s_add_u32 s60, s47, 17\n"
      "v_readlane_b32 s64, v254, s60\n"
      "s_nop 0\n"
      "s_add_u32 s61, s61, s30\n"
      "s_add_u32 s62, s62, s61\n"
      "s_addc_u32 s63, s63, 0\n"
      "s_cmp_lt_u32 s47, 7\n"
      "s_cselect_b32 s64, s64, 0x80000000\n"
      "s_add_u32 s65, s64, 0x8000\n"
      "s_waitcnt lgkmcnt(0)\n"
      "global_store_dwordx4 v248, v[232:235], s[62:63] sc0 sc1\n"
      "buffer_load_dwordx4 v[224:227], v236, s[12:15], s64 offen nt\n"
      "buffer_load_dwordx4 v[228:231], v236, s[12:15], s65 offen nt\n"
      "s_add_u32 s47, s47, 1\n"
      "s_add_u32 s31, s31, 3\n"
      "s_branch .Le0b2_%=\n"
      ".Lso3_%=:\n"
      "s_cmp_ge_u32 s41, 14\n"
      "s_cselect_b32 s36, 8, 6\n"
      "s_cmp_lg_u32 s32, s41\n"
      "s_cbranch_scc1 .Lnp5_%=\n"
      "v_readlane_b32 s62, v254, s48\n"
      "s_add_u32 s60, s48, 8\n"
      "v_readlane_b32 s63, v254, s60\n"
      "s_lshl_b32 s61, s48, 11\n"
      "s_add_u32 s61, s61, s35\n"
      "v_mov_b32 v253, s29\n"
      "s_add_u32 s62, s62, s61\n"
      "s_addc_u32 s63, s63, 0\n"
      "s_add_u32 s48, s48, 1\n"
      "s_add_u32 s32, s32, 3\n"
      "s_cmp_lt_u32 s48, 8\n"
      "s_cselect_b32 s32, s32, 0xffff\n"
      "s_mov_b64 exec, 1\n"
      "global_store_dword v248, v253, s[62:63] sc0 sc1\n"
      "s_mov_b64 exec, -1\n"
      ".Lnp5_%=:\n"
      "s_lshl_b32 s66, s41, 2\n"
      "s_add_u32 s66, s66, s36\n"
      "s_add_u32 s66, s66, 0\n"
      "s_cmp_ge_u32 s41, 22\n"
      "s_cselect_b32 s66, 128, s66\n"
      "s_cmp_lt_u32 s33, s66\n"
      "s_cselect_b32 s49, 1, 0\n"
      "s_cbranch_scc0 .Lsob4_%=\n"
      "s_lshl_b32 s60, s33, 7\n"
      "s_add_u32 s60, s60, s34\n"
      "buffer_load_dword v253, v248, s[16:19], s60 offen sc0 sc1\n"
      "s_branch .Lsob4_%=\n"
      ".Lso6_%=:\n"
      "s_cmp_ge_u32 s41, 14\n"
      "s_cselect_b32 s36, 8, 6\n"
      "s_cmp_lg_u32 s32, s41\n"
      "s_cbranch_scc1 .Lnp8_%=\n"
      "v_readlane_b32 s62, v254, s48\n"
      "s_add_u32 s60, s48, 8\n"
      "v_readlane_b32 s63, v254, s60\n"
      "s_lshl_b32 s61, s48, 11\n"
      "s_add_u32 s61, s61, s35\n"
      "v_mov_b32 v253, s29\n"
      "s_add_u32 s62, s62, s61\n"
      "s_addc_u32 s63, s63, 0\n"
      "s_add_u32 s48, s48, 1\n"
      "s_add_u32 s32, s32, 3\n"
      "s_cmp_lt_u32 s48, 8\n"
      "s_cselect_b32 s32, s32, 0xffff\n"
      "s_mov_b64 exec, 1\n"
      "global_store_dword v248, v253, s[62:63] sc0 sc1\n"
      "s_mov_b64 exec, -1\n"
      ".Lnp8_%=:\n"
      "s_lshl_b32 s66, s41, 2\n"
      "s_add_u32 s66, s66, s36\n"
      "s_add_u32 s66, s66, 0\n"
      "s_cmp_ge_u32 s41, 22\n"
      "s_cselect_b32 s66, 128, s66\n"
      "s_cmp_lt_u32 s33, s66\n"
      "s_cselect_b32 s49, 1, 0\n"
      "s_cbranch_scc0 .Lsob7_%=\n"
      "s_lshl_b32 s60, s33, 7\n"
      "s_add_u32 s60, s60, s34\n"
      "buffer_load_dword v253, v248, s[16:19], s60 offen sc0 sc1\n"
      "s_branch .Lsob7_%=\n"
      ".Lth15_%=:\n"
      "s_sleep 8\n"
      "s_branch .Lthb16_%=\n"
      ".Lsvc9_%=:\n"
      "s_waitcnt vmcnt(4)\n"
      "s_lshl_b32 s67, s41, 2\n"
      "s_cmp_eq_u32 s49, 0\n"
      "s_cbranch_scc1 .Lnf11_%=\n"
      "v_cmp_le_u32 vcc, s29, v253\n"
      "s_nop 0\n"
      "s_ff0_i32_b64 s60, vcc\n"
      "s_min_u32 s60, s60, 64\n"
      "s_lshr_b32 s60, s60, 3\n"
      "s_add_u32 s33, s33, s60\n"
      ".Lnf11_%=:\n"
      "s_add_u32 s67, s67, s36\n"
      "s_cmp_eq_u32 s41, s37\n"
      "s_cselect_b32 s66, 128, s67\n"
      "s_cmp_ge_u32 s33, s66\n"
      "s_cbranch_scc1 .Lsvcb10_%=\n"
      "s_memrealtime s[62:63]\n"
      ".Lsp14_%=:\n"
      "s_lshl_b32 s60, s33, 7\n"
      "s_add_u32 s60, s60, s34\n"
      "buffer_load_dword v253, v248, s[16:19], s60 offen sc0 sc1\n"
      "s_waitcnt vmcnt(0) lgkmcnt(0)\n"
      "v_cmp_le_u32 vcc, s29, v253\n"
      "s_nop 0\n"
      "s_ff0_i32_b64 s60, vcc\n"
      "s_min_u32 s60, s60, 64\n"
      "s_lshr_b32 s60, s60, 3\n"
      "s_add_u32 s33, s33, s60\n"
      "s_cmp_ge_u32 s33, s66\n"
      "s_cbranch_scc1 .Lgot13_%=\n"
      "s_memrealtime s[64:65]\n"
      "s_waitcnt lgkmcnt(0)\n"
      "s_sub_u32 s64, s64, s62\n"
      "s_cmp_lt_u32 s64, 20000\n"
      "s_cbranch_scc1 .Lsl17_%=\n"
      "s_mov_b32 s61, 0x20000\n"
      "buffer_load_dword v253, off, s[16:19], s61 sc1\n"
      "s_waitcnt vmcnt(0)\n"
      "v_readfirstlane_b32 s60, v253\n"
      "s_cmp_gt_u32 s64, 100000000\n"
      "s_cbranch_scc1 .Lto18_%=\n"
      "s_cmp_eq_u32 s60, s29\n"
      "s_cbranch_scc1 .Lto18_%=\n"
      ".Lsl17_%=:\n"
      "s_sleep 1\n"
      "s_branch .Lsp14_%=\n"
      ".Lto18_%=:\n"
      "v_mov_b32 v253, s29\n"
      "buffer_store_dword v253, off, s[16:19], s61 sc1\n"
      "s_waitcnt vmcnt(0)\n"
      "s_movk_i32 s33, 0x80\n"
      ".Lgot13_%=:\n"
      "s_branch .Lsvcb10_%=\n"
      ".Lso19_%=:\n"
      "s_cmp_ge_u32 s41, 14\n"
      "s_cselect_b32 s36, 8, 6\n"
      "s_cmp_lg_u32 s32, s41\n"
      "s_cbranch_scc1 .Lnp21_%=\n"
      "v_readlane_b32 s62, v254, s48\n"
      "s_add_u32 s60, s48, 8\n"
      "v_readlane_b32 s63, v254, s60\n"
      "s_lshl_b32 s61, s48, 11\n"
      "s_add_u32 s61, s61, s35\n"
      "v_mov_b32 v253, s29\n"
      "s_add_u32 s62, s62, s61\n"
      "s_addc_u32 s63, s63, 0\n"
      "s_add_u32 s48, s48, 1\n"
      "s_add_u32 s32, s32, 3\n"
      "s_cmp_lt_u32 s48, 8\n"
      "s_cselect_b32 s32, s32, 0xffff\n"
      "s_mov_b64 exec, 1\n"
      "global_store_dword v248, v253, s[62:63] sc0 sc1\n"
      "s_mov_b64 exec, -1\n"
      ".Lnp21_%=:\n"
      "s_lshl_b32 s66, s41, 2\n"
      "s_add_u32 s66, s66, s36\n"
      "s_add_u32 s66, s66, 0\n"
      "s_cmp_ge_u32 s41, 22\n"
      "s_cselect_b32 s66, 128, s66\n"
      "s_cmp_lt_u32 s33, s66\n"
      "s_cselect_b32 s49, 1, 0\n"
      "s_cbranch_scc0 .Lsob20_%=\n"
      "s_lshl_b32 s60, s33, 7\n"
      "s_add_u32 s60, s60, s34\n"
      "buffer_load_dword v253, v248, s[16:19], s60 offen sc0 sc1\n"
      "s_branch .Lsob20_%=\n"
      ".Lth28_%=:\n"
      "s_sleep 8\n"
      "s_branch .Lthb29_%=\n"
      ".Lsvc22_%=:\n"
      "s_waitcnt vmcnt(4)\n"
      "s_lshl_b32 s67, s41, 2\n"
      "s_cmp_eq_u32 s49, 0\n"
      "s_cbranch_scc1 .Lnf24_%=\n"
      "v_cmp_le_u32 vcc, s29, v253\n"
      "s_nop 0\n"
      "s_ff0_i32_b64 s60, vcc\n"
      "s_min_u32 s60, s60, 64\n"
      "s_lshr_b32 s60, s60, 3\n"
      "s_add_u32 s33, s33, s60\n"
      ".Lnf24_%=:\n"
      "s_add_u32 s67, s67, s36\n"
      "s_cmp_eq_u32 s41, s37\n"
      "s_cselect_b32 s66, 128, s67\n"
      "s_cmp_ge_u32 s33, s66\n"
      "s_cbranch_scc1 .Lsvcb23_%=\n"
      "s_memrealtime s[62:63]\n"
      ".Lsp27_%=:\n"
      "s_lshl_b32 s60, s33, 7\n"
      "s_add_u32 s60, s60, s34\n"
      "buffer_load_dword v253, v248, s[16:19], s60 offen sc0 sc1\n"
      "s_waitcnt vmcnt(0) lgkmcnt(0)\n"
      "v_cmp_le_u32 vcc, s29, v253\n"
      "s_nop 0\n"
      "s_ff0_i32_b64 s60, vcc\n"
      "s_min_u32 s60, s60, 64\n"
      "s_lshr_b32 s60, s60, 3\n"
      "s_add_u32 s33, s33, s60\n"
      "s_cmp_ge_u32 s33, s66\n"
      "s_cbranch_scc1 .Lgot26_%=\n"
      "s_memrealtime s[64:65]\n"
      "s_waitcnt lgkmcnt(0)\n"
      "s_sub_u32 s64, s64, s62\n"
      "s_cmp_lt_u32 s64, 20000\n"
      "s_cbranch_scc1 .Lsl30_%=\n"
      "s_mov_b32 s61, 0x20000\n"
      "buffer_load_dword v253, off, s[16:19], s61 sc1\n"
      "s_waitcnt vmcnt(0)\n"
      "v_readfirstlane_b32 s60, v253\n"
      "s_cmp_gt_u32 s64, 100000000\n"
      "s_cbranch_scc1 .Lto31_%=\n"
      "s_cmp_eq_u32 s60, s29\n"
      "s_cbranch_scc1 .Lto31_%=\n"
      ".Lsl30_%=:\n"
      "s_sleep 1\n"
      "s_branch .Lsp27_%=\n"
      ".Lto31_%=:\n"
      "v_mov_b32 v253, s29\n"
      "buffer_store_dword v253, off, s[16:19], s61 sc1\n"
      "s_waitcnt vmcnt(0)\n"
      "s_movk_i32 s33, 0x80\n"
      ".Lgot26_%=:\n"
      "s_branch .Lsvcb23_%=\n"
      ".Lso32_%=:\n"
      "s_cmp_ge_u32 s41, 14\n"
      "s_cselect_b32 s36, 8, 6\n"
      "s_cmp_lg_u32 s32, s41\n"
      "s_cbranch_scc1 .Lnp34_%=\n"
      "v_readlane_b32 s62, v254, s48\n"
      "s_add_u32 s60, s48, 8\n"
      "v_readlane_b32 s63, v254, s60\n"
      "s_lshl_b32 s61, s48, 11\n"
      "s_add_u32 s61, s61, s35\n"
      "v_mov_b32 v253, s29\n"
      "s_add_u32 s62, s62, s61\n"
      "s_addc_u32 s63, s63, 0\n"
      "s_add_u32 s48, s48, 1\n"
      "s_add_u32 s32, s32, 3\n"
      "s_cmp_lt_u32 s48, 8\n"
      "s_cselect_b32 s32, s32, 0xffff\n"
      "s_mov_b64 exec, 1\n"
      "global_store_dword v248, v253, s[62:63] sc0 sc1\n"
      "s_mov_b64 exec, -1\n"
      ".Lnp34_%=:\n"
      "s_lshl_b32 s66, s41, 2\n"
      "s_add_u32 s66, s66, s36\n"
      "s_add_u32 s66, s66, 0\n"
      "s_cmp_ge_u32 s41, 22\n"
      "s_cselect_b32 s66, 128, s66\n"
      "s_cmp_lt_u32 s33, s66\n"
      "s_cselect_b32 s49, 1, 0\n"
      "s_cbranch_scc0 .Lsob33_%=\n"
      "s_lshl_b32 s60, s33, 7\n"
      "s_add_u32 s60, s60, s34\n"
      "buffer_load_dword v253, v248, s[16:19], s60 offen sc0 sc1\n"
      "s_branch .Lsob33_%=\n"
      ".Lth41_%=:\n"
      "s_sleep 8\n"
      "s_branch .Lthb42_%=\n"
      ".Lsvc35_%=:\n"
      "s_waitcnt vmcnt(8)\n"
      "s_lshl_b32 s67, s41, 2\n"
      "s_cmp_eq_u32 s49, 0\n"
      "s_cbranch_scc1 .Lnf37_%=\n"
      "v_cmp_le_u32 vcc, s29, v253\n"
      "s_nop 0\n"
      "s_ff0_i32_b64 s60, vcc\n"
      "s_min_u32 s60, s60, 64\n"
      "s_lshr_b32 s60, s60, 3\n"
      "s_add_u32 s33, s33, s60\n"
      ".Lnf37_%=:\n"
      "s_add_u32 s67, s67, s36\n"
      "s_cmp_eq_u32 s41, s37\n"
      "s_cselect_b32 s66, 128, s67\n"
      "s_cmp_ge_u32 s33, s66\n"
      "s_cbranch_scc1 .Lsvcb36_%=\n"
      "s_memrealtime s[62:63]\n"
      ".Lsp40_%=:\n"
      "s_lshl_b32 s60, s33, 7\n"
      "s_add_u32 s60, s60, s34\n"
      "buffer_load_dword v253, v248, s[16:19], s60 offen sc0 sc1\n"
      "s_waitcnt vmcnt(0) lgkmcnt(0)\n"
      "v_cmp_le_u32 vcc, s29, v253\n"
      "s_nop 0\n"
      "s_ff0_i32_b64 s60, vcc\n"
      "s_min_u32 s60, s60, 64\n"
      "s_lshr_b32 s60, s60, 3\n"
      "s_add_u32 s33, s33, s60\n"
      "s_cmp_ge_u32 s33, s66\n"
      "s_cbranch_scc1 .Lgot39_%=\n"
      "s_memrealtime s[64:65]\n"
      "s_waitcnt lgkmcnt(0)\n"
      "s_sub_u32 s64, s64, s62\n"
      "s_cmp_lt_u32 s64, 20000\n"
      "s_cbranch_scc1 .Lsl43_%=\n"
      "s_mov_b32 s61, 0x20000\n"
      "buffer_load_dword v253, off, s[16:19], s61 sc1\n"
      "s_waitcnt vmcnt(0)\n"
      "v_readfirstlane_b32 s60, v253\n"
      "s_cmp_gt_u32 s64, 100000000\n"
      "s_cbranch_scc1 .Lto44_%=\n"
      "s_cmp_eq_u32 s60, s29\n"
      "s_cbranch_scc1 .Lto44_%=\n"
      ".Lsl43_%=:\n"
      "s_sleep 1\n"
      "s_branch .Lsp40_%=\n"
      ".Lto44_%=:\n"
      "v_mov_b32 v253, s29\n"
      "buffer_store_dword v253, off, s[16:19], s61 sc1\n"
      "s_waitcnt vmcnt(0)\n"
      "s_movk_i32 s33, 0x80\n"
      ".Lgot39_%=:\n"
      "s_branch .Lsvcb36_%=\n"
      ".Lxw47_%=:\n"
      "s_memrealtime s[62:63]\n"
      ".Lxh49_%=:\n"
      "buffer_load_dword v253, off, s[16:19], s51 sc1\n"
      "s_waitcnt vmcnt(0) lgkmcnt(0)\n"
      "v_readfirstlane_b32 s60, v253\n"
      "s_nop 0\n"
      "s_cmp_ge_u32 s60, s29\n"
      "s_cbranch_scc1 .Lxwb48_%=\n"
      "s_memrealtime s[64:65]\n"
      "s_waitcnt lgkmcnt(0)\n"
      "s_sub_u32 s64, s64, s62\n"
      "s_cmp_lt_u32 s64, 20000\n"
      "s_cbranch_scc1 .Lsl50_%=\n"
      "s_mov_b32 s61, 0x20000\n"
      "buffer_load_dword v253, off, s[16:19], s61 sc1\n"
      "s_waitcnt vmcnt(0)\n"
      "v_readfirstlane_b32 s60, v253\n"
      "s_cmp_gt_u32 s64, 100000000\n"
      "s_cbranch_scc1 .Lto51_%=\n"
      "s_cmp_eq_u32 s60, s29\n"
      "s_cbranch_scc1 .Lto51_%=\n"
      ".Lsl50_%=:\n"
      "s_sleep 1\n"
      "s_branch .Lxh49_%=\n"
      ".Lto51_%=:\n"
      "v_mov_b32 v253, s29\n"
      "buffer_store_dword v253, off, s[16:19], s61 sc1\n"
      "s_waitcnt vmcnt(0)\n"
      "s_branch .Lxwb48_%=\n"
      ".Lnan52_%=:\n"
      "v_mov_b32 v160, 0x7fc00000\n"
      "v_mov_b32 v161, 0x7fc00000\n"
      "v_mov_b32 v162, 0x7fc00000\n"
      "v_mov_b32 v163, 0x7fc00000\n"
      "s_add_u32 s60, s38, 0\n"
      "buffer_store_dwordx4 v[160:163], v236, s[24:27], s60 offen offset:0\n"
      "buffer_store_dwordx4 v[160:163], v236, s[24:27], s60 offen offset:256\n"
      "s_add_u32 s60, s38, 32768\n"
      "buffer_store_dwordx4 v[160:163], v236, s[24:27], s60 offen offset:0\n"
      "buffer_store_dwordx4 v[160:163], v236, s[24:27], s60 offen offset:256\n"
      "s_add_u32 s60, s38, 65536\n"
      "buffer_store_dwordx4 v[160:163], v236, s[24:27], s60 offen offset:0\n"
      "buffer_store_dwordx4 v[160:163], v236, s[24:27], s60 offen offset:256\n"
      "s_add_u32 s60, s38, 98304\n"
      "buffer_store_dwordx4 v[160:163], v236, s[24:27], s60 offen offset:0\n"
      "buffer_store_dwordx4 v[160:163], v236, s[24:27], s60 offen offset:256\n"
      "s_add_u32 s60, s38, 524288\n"
      "buffer_store_dwordx4 v[160:163], v236, s[24:27], s60 offen offset:0\n"
      "buffer_store_dwordx4 v[160:163], v236, s[24:27], s60 offen offset:256\n"
      "s_add_u32 s60, s38, 557056\n"
      "buffer_store_dwordx4 v[160:163], v236, s[24:27], s60 offen offset:0\n"
      "buffer_store_dwordx4 v[160:163], v236, s[24:27], s60 offen offset:256\n"
      "s_add_u32 s60, s38, 589824\n"
      "buffer_store_dwordx4 v[160:163], v236, s[24:27], s60 offen offset:0\n"
      "buffer_store_dwordx4 v[160:163], v236, s[24:27], s60 offen offset:256\n"
      "s_add_u32 s60, s38, 622592\n"
      "buffer_store_dwordx4 v[160:163], v236, s[24:27], s60 offen offset:0\n"
      "buffer_store_dwordx4 v[160:163], v236, s[24:27], s60 offen offset:256\n"
      "s_branch .Lend_%=\n"
      ".Lend_%=:\n"
      : "+{v[224:227]}"(XR[0]), "+{v[228:231]}"(XR[1]), "+{v251}"(rdo), "+{v252}"(wro), "+{v255}"(vst), "+{v236}"(vx), "+{v237}"(vtw), "+{v238}"(vtr),
        "+{s31}"(evE), "+{s32}"(fe), "+{s33}"(rdy), "+{s50}"(slow), "+{s47}"(q0)
      : "{v248}"(va), "{v249}"(vw), "{v254}"(tab),
        "{s[8:11]}"(srdW), "{s[12:15]}"(srdX), "{s[16:19]}"(srdF), "{s[20:23]}"(srdA), "{s[24:27]}"(srdC),
        "{s28}"(w), "{s29}"(call), "{s30}"(parofs), "{s34}"(fb), "{s35}"(fl0), "{s37}"(lastS), "{s38}"(c_me), "{s39}"(c_pt), "{s40}"(xf_me)
      : "memory", "scc", "vcc",
        "v0", "v1", "v2", "v3", "v4", "v5", "v6", "v7", "v8", "v9", "v10", "v11", "v12", "v13", "v14", "v15", "v16", "v17", "v18", "v19", "v20", "v21", "v22", "v23",
        "v24", "v25", "v26", "v27", "v28", "v29", "v30", "v31", "v32", "v33", "v34", "v35", "v36", "v37", "v38", "v39", "v40", "v41", "v42", "v43", "v44", "v45", "v46", "v47",
        "v48", "v49", "v50", "v51", "v52", "v53", "v54", "v55", "v56", "v57", "v58", "v59", "v60", "v61", "v62", "v63", "v64", "v65", "v66", "v67", "v68", "v69", "v70", "v71",
        "v72", "v73", "v74", "v75", "v76", "v77", "v78", "v79", "v80", "v81", "v82", "v83", "v84", "v85", "v86", "v87", "v88", "v89", "v90", "v91", "v92", "v93", "v94", "v95",
        "v96", "v97", "v98", "v99", "v100", "v101", "v102", "v103", "v104", "v105", "v106", "v107", "v108", "v109", "v110", "v111", "v112", "v113", "v114", "v115", "v116", "v117", "v118", "v119",
        "v120", "v121", "v122", "v123", "v124", "v125", "v126", "v127", "v128", "v129", "v130", "v131", "v132", "v133", "v134", "v135", "v136", "v137", "v138", "v139", "v140", "v141", "v142", "v143",
        "v144", "v145", "v146", "v147", "v148", "v149", "v150", "v151", "v152", "v153", "v154", "v155", "v156", "v157", "v158", "v159", "v160", "v161", "v162", "v163", "v164", "v165", "v166", "v167",
        "v168", "v169", "v170", "v171", "v172", "v173", "v174", "v175", "v176", "v177", "v178", "v179", "v180", "v181", "v182", "v183", "v184", "v185", "v186", "v187", "v188", "v189", "v190", "v191",
        "v192", "v193", "v194", "v195", "v196", "v197", "v198", "v199", "v200", "v201", "v202", "v203", "v204", "v205", "v206", "v207", "v208", "v209", "v210", "v211", "v212", "v213", "v214", "v215",
        "v216", "v217", "v218", "v219", "v220", "v221", "v222", "v223", "v232", "v233", "v234", "v235", "v239", "v240", "v241", "v242", "v243", "v244", "v245", "v246", "v247", "v250", "v253", "s41",
        "s42", "s43", "s44", "s45", "s46", "s48", "s49", "s51", "s52", "s53", "s54", "s55", "s56", "s60", "s61", "s62", "s63", "s64", "s65", "s66", "s67", "s36"
      );
}

extern "C" void kernel_launch(int rank, int nranks, ncclComm_t comm, hipStream_t stream,
                              void* const* d_in, const int* in_sizes, int n_in,
                              void* d_out, int out_size, const int* in_dtypes, int out_dtype,
                              void* const* d_out_all, void* const* d_ws_all, size_t ws_bytes) {
  if (nranks != NR || n_in != 2 || (unsigned)rank >= (unsigned)NR || ws_bytes < WS_NEED ||
      in_sizes[0] != 8192 * 1024 || in_sizes[1] != 8192 * 8192 || out_size != 1024 * 8192) {
    fprintf(stderr, "a2a_gemm: unexpected problem shape\n");
    return;
  }
  Params p;
  p.x = (const float*)d_in[0]; p.W = (const float*)d_in[1]; p.C = (float*)d_out; p.ws = (char*)d_ws_all[rank]; p.rank = rank;
  for (int i = 0; i < NR; ++i) p.ws_all[i] = (char*)d_ws_all[i];
  a2a_gemm<<<256, 512, 0, stream>>>(p);
  hipError_t e = hipGetLastError();
  if (e != hipSuccess) fprintf(stderr, "a2a_gemm launch: %s\n", hipGetErrorString(e));
}
```
